```python
import numpy as np

kernel_name = "gpu_gemm_rs_m4096_k4096_n16384_bf16_silu"


ABSMAX_RTOL = 0.008

COMPUTE_DTYPE = "bfloat16"
EPILOGUE = "silu"
IN_QUANT = None
OUT_QUANT = None
IN_DTYPES = {"x": "bfloat16", "w_mat": "bfloat16"}
WORLD_SIZE = 8

def _to_bf16(x):
    u = np.ascontiguousarray(x, dtype=np.float32).view(np.uint32)
    return ((u + 0x7FFF + ((u >> 16) & 1)) & 0xFFFF0000).view(np.float32)

def setup_inputs():

    x = _to_bf16(np.random.randn(4096, 4096).astype(np.float32))
    w_mat = _to_bf16((np.random.randn(4096, 16384) / np.sqrt(4096)).astype(np.float32))
    return {"x": x, "w_mat": w_mat}

def _epilogue(y):
    return (y / (1.0 + np.exp(-np.clip(y, -60.0, 60.0)))).astype(np.float32)

def reference(x, w_mat):
    y = (x @ w_mat).astype(np.float32)
    return _epilogue(y)
```

```cpp
#include <hip/hip_runtime.h>
#include <rccl/rccl.h>
#include <cstdio>

#define HIP_CHECK(expr) do { hipError_t _e = (expr); if (_e != hipSuccess) { \
    fprintf(stderr, "HIP error %s at %s:%d\n", hipGetErrorString(_e), __FILE__, __LINE__); \
    return; } } while (0)

namespace {

constexpr int NR = 8;
constexpr int GM = 4096, GK = 4096, GN = 16384;
constexpr int KS = GK / NR;
constexpr unsigned MiB = 1u << 20;
constexpr unsigned UNIT = 32768;
constexpr unsigned X_BYTES = GM * KS * 2, W_BYTES = KS * GN * 2, OUT_BYTES = (GM / NR) * GN * 2;

constexpr unsigned EXP_SZ = 20 * MiB;
constexpr unsigned EXP_W = 4 * MiB;
constexpr unsigned OFF_EXP = 1 * MiB;
constexpr unsigned YEXP_SZ = 16 * MiB;
constexpr unsigned OFF_YEXP = OFF_EXP + 2 * EXP_SZ;
constexpr unsigned STG_SZ = 6 * MiB;
constexpr unsigned OFF_STG = OFF_YEXP + 2 * YEXP_SZ;
constexpr unsigned OFF_DIAG = OFF_STG + NR * STG_SZ;
constexpr unsigned WS_NEED = OFF_DIAG + 32 * MiB;
constexpr unsigned F_CNT = 0;
constexpr unsigned F_TMO = 32768;
constexpr unsigned F_PUB = 36864;
constexpr unsigned F_Y = 49152;
constexpr unsigned F_DG = 57344;
constexpr unsigned F_A = 65536;
constexpr unsigned F_B = 131072;
constexpr unsigned F_STAMP = 524288;

constexpr unsigned long long DEADLINE = 100000000ull;

#define LAG_O 2
#define LAG_W 5

constexpr int AUX_AGENT = 16, AUX_SYS = 17;

typedef short bf16x8 __attribute__((ext_vector_type(8)));
typedef short s16x4 __attribute__((ext_vector_type(4)));
typedef float f32x4 __attribute__((ext_vector_type(4)));
typedef unsigned u32x4 __attribute__((ext_vector_type(4)));
typedef unsigned u32x2 __attribute__((ext_vector_type(2)));
#define LAS __attribute__((address_space(3)))
typedef __amdgpu_buffer_rsrc_t rsrc_t;

struct Params {
    const void* x; const void* w; void* out;
    void* ws[NR];
    int rank;
};

__device__ __forceinline__ rsrc_t mkrs(const void* p, unsigned bytes) {
    return __builtin_amdgcn_make_buffer_rsrc((void*)p, 0, bytes, 0x00020000);
}
__device__ __forceinline__ unsigned cvt_pk_bf16(float lo, float hi) {
    unsigned r; asm("v_cvt_pk_bf16_f32 %0, %1, %2" : "=v"(r) : "v"(lo), "v"(hi)); return r;
}
__device__ __forceinline__ float silu(float y) {
    const float c = __builtin_amdgcn_fmed3f(y, -60.0f, 60.0f);
    const float e = __builtin_amdgcn_exp2f(c * -1.44269504088896341f);
    return y * __builtin_amdgcn_rcpf(1.0f + e);
}
__device__ __forceinline__ int rfl(int v) { return __builtin_amdgcn_readfirstlane(v); }
__device__ __forceinline__ int skip2(int i, int a, int b) {
    const int lo = a < b ? a : b, hi = a < b ? b : a;
    int j = i; if (j >= lo) ++j; if (hi != lo && j >= hi) ++j; return j;
}
#define WAIT_V(n) asm volatile("s_waitcnt vmcnt(" #n ")" ::: "memory")
#define WAIT_L(n) asm volatile("s_waitcnt lgkmcnt(" #n ")" ::: "memory")
#define BAR __builtin_amdgcn_s_barrier()
#define SCHED __builtin_amdgcn_sched_barrier(0)

#define STAMP_AT(slot) do { } while (0)

__device__ __forceinline__ bool give_up(rsrc_t rs_me, unsigned call, unsigned long long t0) {
    const unsigned tm = __builtin_amdgcn_raw_buffer_load_b32(rs_me, 0, F_TMO, AUX_AGENT);
    if ((unsigned)rfl((int)tm) == call || __builtin_amdgcn_s_memrealtime() - t0 > DEADLINE) {
        __builtin_amdgcn_raw_buffer_store_b32(call, rs_me, 0, F_TMO, AUX_AGENT);
        return true;
    }
    __builtin_amdgcn_s_sleep(2);
    asm volatile("" ::: "memory");
    return false;
}

__global__ __launch_bounds__(512, 2) void fused_gemm_rs(const Params p) {
    __shared__ __attribute__((aligned(16))) unsigned char smem[131072];
    LAS unsigned char* lds = (LAS unsigned char*)smem;
    const int tid = threadIdx.x, lane = tid & 63, wid = rfl(tid >> 6);
    const int bid = blockIdx.x, xcd = bid & 7, slot = bid >> 3;
    const int rank = p.rank;
    const rsrc_t rs_me = mkrs(p.ws[rank], WS_NEED);

    STAMP_AT(0);
    const unsigned call = (unsigned)rfl((int)__builtin_amdgcn_raw_buffer_load_b32(rs_me, 0, F_CNT + bid * 128, AUX_AGENT)) + 1u;
    const unsigned par = call & 1u;
    const unsigned exp_off = OFF_EXP + par * EXP_SZ;
    const unsigned yexp_off = OFF_YEXP + par * YEXP_SZ;

    {
        const rsrc_t rs_x = mkrs(p.x, X_BYTES), rs_w = mkrs(p.w, W_BYTES);
        const int nb = bid - 56;
        for (int r = 0; r < 3; ++r) {
            int u;
            if (r == 0) u = bid; else { if (nb < 0) break; u = 56 + r * 200 + nb; if (u >= 640) break; }
            const int kc = u / 80, j = u - kc * 80;
            unsigned dst; u32x4 o[4];
            if (j < 64) {
                const int dest = ((j >> 3) + rank + 1) & 7, ct = j & 7;
                dst = exp_off + EXP_W + (unsigned)((dest * 8 + kc) * 8 + ct) * UNIT;
                const unsigned so = (unsigned)(kc * 64) * (GN * 2) + (unsigned)(dest * 2048 + ct * 256) * 2;
                const unsigned vo = (unsigned)(tid >> 5) * (GN * 2) + (unsigned)(tid & 31) * 16;
                u32x4 v[4];
#pragma unroll
                for (int i = 0; i < 4; ++i) v[i] = __builtin_amdgcn_raw_buffer_load_b128(rs_w, vo, so + i * 16 * (GN * 2), 0);
                if (r) __syncthreads();
#pragma unroll
                for (int i = 0; i < 4; ++i) *(LAS u32x4*)(lds + (i * 16 + (tid >> 5)) * 576 + (tid & 31) * 16) = v[i];
                __syncthreads();
#pragma unroll
                for (int i = 0; i < 4; ++i) {
                    const int F = i * 8 + wid, bj = F >> 4, wcn = (F >> 1) & 7, kk = F & 1;
                    const int col = (wcn >> 1) * 64 + bj * 32 + (wcn & 1) * 4 + (lane & 3) * 8;
                    const int row = kk * 32 + (lane >> 4) * 8 + ((lane & 15) >> 2);
                    const s16x4 t0 = __builtin_amdgcn_ds_read_tr16_b64_v4i16((LAS s16x4*)(lds + row * 576 + col * 2));
                    const s16x4 t1 = __builtin_amdgcn_ds_read_tr16_b64_v4i16((LAS s16x4*)(lds + (row + 4) * 576 + col * 2));
                    const u32x2 a = __builtin_bit_cast(u32x2, t0), b = __builtin_bit_cast(u32x2, t1);
                    o[i] = (u32x4){a.x, a.y, b.x, b.y};
                }
#pragma unroll
                for (int i = 0; i < 4; ++i) __builtin_amdgcn_raw_buffer_store_b128(o[i], rs_me, lane * 16, dst + (i * 8 + wid) * 1024, AUX_SYS);
            } else {
                const int rt = j - 64;
                dst = exp_off + (unsigned)(kc * 16 + rt) * UNIT;
                const unsigned vo = (unsigned)(lane & 15) * (KS * 2) + (unsigned)(lane >> 4) * 16;
#pragma unroll
                for (int i = 0; i < 4; ++i) {
                    const int F = wid * 4 + i;
                    o[i] = __builtin_amdgcn_raw_buffer_load_b128(rs_x, vo, (unsigned)(rt * 256 + (F >> 1) * 16) * (KS * 2) + kc * 128 + (F & 1) * 64, 0);
                }
#pragma unroll
                for (int i = 0; i < 4; ++i) __builtin_amdgcn_raw_buffer_store_b128(o[i], rs_me, lane * 16, dst + (wid * 4 + i) * 1024, AUX_SYS);
            }
            WAIT_V(0);
            __syncthreads();
            if (wid == 0) {
                if (r == 0 && lane == 0) __builtin_amdgcn_raw_buffer_store_b32(call, rs_me, 0, F_CNT + bid * 128, AUX_AGENT);
                if (j < 64) {
                    const int dest = ((j >> 3) + rank + 1) & 7, ct = j & 7;
                    if (dest != rank && lane == 0)
                        __builtin_amdgcn_raw_buffer_store_b32(call, mkrs(p.ws[dest], WS_NEED), 0, F_PUB + (rank * 256 + kc * 24 + ct) * 4, AUX_SYS);
                    if (lane < 8)
                        __builtin_amdgcn_raw_buffer_store_b32(call, rs_me, lane * (64 * 64 * 4), F_B + ((dest * 8 + ct) * 64 + rank * 8 + kc) * 4, AUX_AGENT);
                } else {
                    const int rt = j - 64;
                    if (lane < 8)
                        __builtin_amdgcn_raw_buffer_store_b32(call, rs_me, lane * (16 * 64 * 4), F_A + (rt * 64 + rank * 8 + kc) * 4, AUX_AGENT);
                }
            } else if (j >= 64 && lane == 0) {
                const int rt = j - 64, dest = (rank + wid) & 7;
                if ((rt >> 1) != rank)
                    __builtin_amdgcn_raw_buffer_store_b32(call, mkrs(p.ws[dest], WS_NEED), 0, F_PUB + (rank * 256 + kc * 24 + 8 + rt) * 4, AUX_SYS);
            }
        }
        __syncthreads();
    }
    STAMP_AT(1);

    if (slot < 7) {
        const int src = (rank + 1 + slot) & 7, g = xcd;
        const rsrc_t rs_src = mkrs(p.ws[src], WS_NEED);
        const rsrc_t rs_out = mkrs(p.out, OUT_BYTES);
        unsigned v_src = 0, v_dst = 0, v_flg = 0, v_pub = 0;
        {
            int pos = g + 8 * lane; bool found = false;
            for (int s = 0; s < 8 + (LAG_W > LAG_O ? LAG_W : LAG_O); ++s) {
                int kc = -1, rt = -1, ct = -1;
                if (s < 8) {
                    if (!found && pos < 8) { kc = s; ct = pos; found = true; } pos -= 8;
                    if (!found && pos >= 0 && pos < 6) { kc = s; rt = 2 * skip2(pos, src, rank); found = true; } pos -= 6;
                }
                if (s >= LAG_O && s - LAG_O < 8) { if (!found && pos >= 0 && pos < 6) { kc = s - LAG_O; rt = 2 * skip2(pos, src, rank) + 1; found = true; } pos -= 6; }
                if (s >= LAG_W && s - LAG_W < 8) { if (!found && pos >= 0 && pos < 2) { kc = s - LAG_W; rt = 2 * rank + pos; found = true; } pos -= 2; }
                if (ct >= 0) {
                    v_src = exp_off + EXP_W + (unsigned)((rank * 8 + kc) * 8 + ct) * UNIT;
                    v_dst = OFF_STG + src * STG_SZ + EXP_W + (unsigned)(kc * 8 + ct) * UNIT;
                    v_flg = F_B + ((wid * 64 + rank * 8 + ct) * 64 + src * 8 + kc) * 4;
                    v_pub = F_PUB + (src * 256 + kc * 24 + ct) * 4;
                } else if (rt >= 0) {
                    v_src = exp_off + (unsigned)(kc * 16 + rt) * UNIT;
                    v_dst = OFF_STG + src * STG_SZ + (unsigned)(kc * 16 + rt) * UNIT;
                    v_flg = F_A + ((wid * 16 + rt) * 64 + src * 8 + kc) * 4;
                    v_pub = F_PUB + (src * 256 + kc * 24 + 8 + rt) * 4;
                }
            }
        }
        constexpr int NU = 22;
        const unsigned vo = (unsigned)(wid * 1024 + lane * 16);
        unsigned long long mask = 0; bool bad = false;
        unsigned prev_flag = 0;
        for (int l = 0; l < NU && !bad; ++l) {
            if (!((mask >> l) & 1ull)) {
                const unsigned long long t0 = __builtin_amdgcn_s_memrealtime();
                for (;;) {
                    const unsigned f = __builtin_amdgcn_raw_buffer_load_b32(rs_me, v_pub, 0, AUX_SYS);
                    mask = __ballot(lane < NU && f >= call);
                    if ((mask >> l) & 1ull) break;
                    if (give_up(rs_me, call, t0)) { bad = true; break; }
                }
                if (bad) break;
            }
            const unsigned so = (unsigned)__builtin_amdgcn_readlane((int)v_src, l), dofs = (unsigned)__builtin_amdgcn_readlane((int)v_dst, l);
#pragma unroll
            for (int i = 0; i < 4; ++i) {
                const u32x4 v = __builtin_amdgcn_raw_buffer_load_b128(rs_src, vo, so + i * 8192, AUX_SYS);
                WAIT_V(0);
                if (i == 0 && l > 0 && lane == 0) __builtin_amdgcn_raw_buffer_store_b32(call, rs_me, 0, prev_flag, AUX_AGENT);
                __builtin_amdgcn_raw_buffer_store_b128(v, rs_me, vo, dofs + i * 8192, AUX_AGENT);
            }
            prev_flag = (unsigned)__builtin_amdgcn_readlane((int)v_flg, l);
        }
        WAIT_V(0);
        if (!bad && lane == 0) __builtin_amdgcn_raw_buffer_store_b32(call, rs_me, 0, prev_flag, AUX_AGENT);
        STAMP_AT(2);
        unsigned todo = 0xffffu;
        const unsigned v_fy = F_Y + ((src * 8 + g) * 32 + (lane & 15)) * 4 + (lane >= 16 ? (F_DG - F_Y) : 0u);
        const unsigned v_out = (unsigned)((wid >> 2) * 128 + (g >> 2) * 64 + (wid & 3) * 16 + (lane & 15)) * (GN * 2)
                             + (unsigned)(src * 2048 + (g & 3) * 64 + (lane >> 4) * 8) * 2;
        unsigned avail = 0;
        while (todo) {
            if (!(avail & todo)) {
                const unsigned long long t0 = __builtin_amdgcn_s_memrealtime();
                for (;;) {
                    if (bad) { avail = 0xffffu; break; }
                    const unsigned f = __builtin_amdgcn_raw_buffer_load_b32(rs_me, v_fy, 0, AUX_SYS);
                    const unsigned m = (unsigned)__ballot(f >= call);
                    avail = m & (m >> 16) & 0xffffu;
                    if (avail & todo) break;
                    if (give_up(rs_me, call, t0)) bad = true;
                }
            }
            const int t = __builtin_ctz(avail & todo);
            todo &= ~(1u << t);
            const unsigned ys = yexp_off + (unsigned)(rank * 16 + t) * 131072u + g * 16384 + wid * 2048;
            const unsigned ds = OFF_DIAG + (unsigned)(src * 16 + t) * 262144u + g * 32768 + wid * 4096;
            const unsigned os = (unsigned)(t >> 3) * 256u * (GN * 2) + (t & 7) * 512;
#pragma unroll
            for (int bj = 0; bj < 2; ++bj) {
                const u32x4 y = __builtin_amdgcn_raw_buffer_load_b128(rs_src, lane * 16, ys + bj * 1024, AUX_SYS);
                const u32x4 d0 = __builtin_amdgcn_raw_buffer_load_b128(rs_me, lane * 16, ds + bj * 2048, AUX_AGENT);
                const u32x4 d1 = __builtin_amdgcn_raw_buffer_load_b128(rs_me, lane * 16, ds + bj * 2048 + 1024, AUX_AGENT);
                WAIT_V(0);
                u32x4 o;
                o.x = cvt_pk_bf16(silu(__uint_as_float(y.x << 16) + __uint_as_float(d0.x)), silu(__uint_as_float(y.x & 0xffff0000u) + __uint_as_float(d0.y)));
                o.y = cvt_pk_bf16(silu(__uint_as_float(y.y << 16) + __uint_as_float(d0.z)), silu(__uint_as_float(y.y & 0xffff0000u) + __uint_as_float(d0.w)));
                o.z = cvt_pk_bf16(silu(__uint_as_float(y.z << 16) + __uint_as_float(d1.x)), silu(__uint_as_float(y.z & 0xffff0000u) + __uint_as_float(d1.y)));
                o.w = cvt_pk_bf16(silu(__uint_as_float(y.w << 16) + __uint_as_float(d1.z)), silu(__uint_as_float(y.w & 0xffff0000u) + __uint_as_float(d1.w)));
                if (bad) o = (u32x4){0x7fc07fc0u, 0x7fc07fc0u, 0x7fc07fc0u, 0x7fc07fc0u};
                __builtin_amdgcn_raw_buffer_store_b128(o, rs_out, v_out, os + bj * 64, AUX_AGENT);
            }
        }
        STAMP_AT(3);
        return;
    }
    if (slot >= 30) return;

    const int wr = wid >> 2, wc = wid & 3;
    int nt, seglen, mode, tile0;
    unsigned v_a, v_b, v_fa, v_fb;
    int dest;
    if (slot < 23) {
        const int i = slot - 7;
        const int j = ((xcd >> 1) & 1) * 4 + (i & 3), rt2 = xcd & 1, rt = 2 * j + rt2, ct = (xcd >> 2) * 4 + (i >> 2);
        const int nsrc = (j == rank) ? 7 : 6;
        nt = 8 + 8 * nsrc; seglen = nt; mode = (j == rank) ? 0 : 1; dest = j; tile0 = rt2 * 8 + ct;
        int src, kc;
        if (lane < 8 || lane >= nt) { src = rank; kc = lane & 7; }
        else { const int q = lane - 8; kc = q / nsrc; src = skip2(q - kc * nsrc, rank, j); }
        v_a = (src == rank ? exp_off : OFF_STG + src * STG_SZ) + (unsigned)(kc * 16 + rt) * UNIT;
        v_b = (src == rank ? exp_off + (unsigned)(rank * 64) * UNIT : OFF_STG + src * STG_SZ) + EXP_W + (unsigned)(kc * 8 + ct) * UNIT;
        v_fa = F_A + ((wid * 16 + rt) * 64 + src * 8 + kc) * 4;
        v_fb = F_B + ((wid * 64 + rank * 8 + ct) * 64 + src * 8 + kc) * 4;
    } else {
        const int slab = (rank + 1 + (slot - 23)) & 7, ct = xcd;
        nt = 16; seglen = 8; mode = 2; dest = slab; tile0 = ct;
        const int kc = lane & 7, rt = 2 * rank + ((lane >> 3) & 1);
        v_a = exp_off + (unsigned)(kc * 16 + rt) * UNIT;
        v_b = exp_off + EXP_W + (unsigned)((slab * 8 + kc) * 8 + ct) * UNIT;
        v_fa = F_A + ((wid * 16 + rt) * 64 + rank * 8 + kc) * 4;
        v_fb = F_B + ((wid * 64 + slab * 8 + ct) * 64 + rank * 8 + kc) * 4;
    }
    int ready = 0; bool bad = false;
    auto ensure = [&](int need) {
        if (need > nt) need = nt;
        if (ready >= need) return;
        const unsigned long long t0 = __builtin_amdgcn_s_memrealtime();
        __builtin_amdgcn_s_waitcnt(0xC07F);
        for (;;) {
            const unsigned fa = __builtin_amdgcn_raw_buffer_load_b32(rs_me, v_fa, 0, AUX_AGENT);
            const unsigned fb = __builtin_amdgcn_raw_buffer_load_b32(rs_me, v_fb, 0, AUX_AGENT);
            const unsigned long long nm = ~__ballot(fa >= call && fb >= call);
            ready = nm ? __builtin_ctzll(nm) : 64;
            if (ready >= need) break;
            if (give_up(rs_me, call, t0)) { bad = true; ready = 64; break; }
        }
    };
    const unsigned vst = (unsigned)tid * 16u;
    const int ldsw = wid * 1024;
    const int aoff = wr * 8192 + lane * 16;
    int boff = 65536 + wc * 4096 + lane * 16; asm volatile("" : "+v"(boff));
#define SA(b, h) (((b) * 2 + (h)) * 16384)
#define SB(b, h) ((4 + (b) * 2 + (h)) * 16384)
#define STAGE(bufoff, goff) do { _Pragma("unroll") for (int _i = 0; _i < 2; ++_i) \
        __builtin_amdgcn_raw_ptr_buffer_load_lds(rs_me, (LAS void*)(lds + (bufoff) + ldsw + _i * 8192), 16, vst, (goff) + _i * 8192, 0, AUX_AGENT); } while (0)
#define LDA(dst, b, h) do { _Pragma("unroll") for (int m = 0; m < 4; ++m) _Pragma("unroll") for (int k = 0; k < 2; ++k) \
        dst[m][k] = *(const LAS bf16x8*)(lds + SA(b, h) + aoff + m * 2048 + k * 1024); } while (0)
#define LDB(dst, b, h) do { _Pragma("unroll") for (int n = 0; n < 2; ++n) _Pragma("unroll") for (int k = 0; k < 2; ++k) \
        dst[n][k] = *(const LAS bf16x8*)(lds + boff + (SB(b, h) - 65536 + n * 2048 + k * 1024)); } while (0)
#define MMA(ai, bj, At, Bt) do { __builtin_amdgcn_s_setprio(1); _Pragma("unroll") for (int m = 0; m < 4; ++m) _Pragma("unroll") for (int n = 0; n < 2; ++n) \
        _Pragma("unroll") for (int k = 0; k < 2; ++k) acc[ai][bj][m][n] = __builtin_amdgcn_mfma_f32_16x16x32_bf16(Bt[n][k], At[m][k], acc[ai][bj][m][n], 0, 0, 0); \
        __builtin_amdgcn_s_setprio(0); } while (0)
#define AOFF(t) ((unsigned)__builtin_amdgcn_readlane((int)v_a, (t)))
#define BOFF(t) ((unsigned)__builtin_amdgcn_readlane((int)v_b, (t)))
    f32x4 acc[2][2][4][2];
#pragma unroll
    for (int a = 0; a < 2; ++a)
#pragma unroll
        for (int b = 0; b < 2; ++b)
#pragma unroll
            for (int m = 0; m < 4; ++m)
#pragma unroll
                for (int n = 0; n < 2; ++n) acc[a][b][m][n] = (f32x4){0.f, 0.f, 0.f, 0.f};
    bf16x8 At[4][2], B0[2][2], B1[2][2];
    ensure(2);
    {
        const unsigned a0 = AOFF(0), b0 = BOFF(0), a1 = AOFF(1), b1 = BOFF(1);
        STAGE(SB(0, 0), b0); STAGE(SA(0, 0), a0); STAGE(SB(0, 1), b0 + 16384); STAGE(SA(0, 1), a0 + 16384);
        if (wr == 1) BAR;
        WAIT_V(4); BAR;
        STAGE(SB(1, 0), b1); STAGE(SA(1, 0), a1); STAGE(SB(1, 1), b1 + 16384);
        WAIT_V(6); BAR;
    }
    STAMP_AT(2);
    int tile = tile0;
    for (int t = 0;;) {
        const int seg_end = t + seglen;
        for (; t < seg_end; t += 2) {
            ensure(t + 4);
            const int t2 = (t + 2 < nt) ? t + 2 : 0;
            const unsigned a1 = AOFF(t + 1), a2 = AOFF(t2), b2 = BOFF(t2), a3 = AOFF(t2 + 1), b3 = BOFF(t2 + 1);
            LDB(B0, 0, 0); SCHED; LDA(At, 0, 0); STAGE(SA(1, 1), a1 + 16384);
            WAIT_L(8); BAR; WAIT_L(0); MMA(0, 0, At, B0); BAR; SCHED;
            LDB(B1, 0, 1); STAGE(SB(0, 0), b2);
            BAR; WAIT_L(0); MMA(0, 1, At, B1); BAR;
            LDA(At, 0, 1); STAGE(SA(0, 0), a2);
            BAR; WAIT_L(0); MMA(1, 0, At, B0); BAR; SCHED;
            STAGE(SB(0, 1), b2 + 16384);
            WAIT_V(6); BAR; MMA(1, 1, At, B1); BAR;
            LDB(B0, 1, 0); SCHED; LDA(At, 1, 0); STAGE(SA(0, 1), a2 + 16384);
            WAIT_L(8); BAR; WAIT_L(0); MMA(0, 0, At, B0); BAR; SCHED;
            LDB(B1, 1, 1); STAGE(SB(1, 0), b3);
            BAR; WAIT_L(0); MMA(0, 1, At, B1); BAR;
            LDA(At, 1, 1); STAGE(SA(1, 0), a3);
            BAR; WAIT_L(0); MMA(1, 0, At, B0); BAR; SCHED;
            STAGE(SB(1, 1), b3 + 16384);
            WAIT_V(6); BAR; MMA(1, 1, At, B1); BAR;
        }
        STAMP_AT(3);
        if (mode != 2) break;
        {
            int lz = lane; asm volatile("" : "+v"(lz));
            const unsigned nanm = bad ? 0x7fc00000u : 0u;
            const unsigned so = OFF_DIAG + (unsigned)(dest * 16 + tile) * 262144u + wid * 32768;
#pragma unroll
            for (int a = 0; a < 2; ++a)
#pragma unroll
                for (int m = 0; m < 4; ++m)
#pragma unroll
                    for (int b = 0; b < 2; ++b)
#pragma unroll
                        for (int n = 0; n < 2; ++n) {
                            u32x4 o = __builtin_bit_cast(u32x4, acc[a][b][m][n]); o.x |= nanm;
                            __builtin_amdgcn_raw_buffer_store_b128(o, rs_me, lz * 16 + (b * 2 + n) * 1024, so + (a * 4 + m) * 4096, AUX_AGENT);
                        }
            WAIT_V(0);
            if (lz == 0) __builtin_amdgcn_raw_buffer_store_b32(call, rs_me, 0, F_DG + ((dest * 8 + wid) * 32 + tile) * 4, AUX_AGENT);
        }
        STAMP_AT(4);
        if (t >= nt) break;
#pragma unroll
        for (int a = 0; a < 2; ++a)
#pragma unroll
            for (int b = 0; b < 2; ++b)
#pragma unroll
                for (int m = 0; m < 4; ++m)
#pragma unroll
                    for (int n = 0; n < 2; ++n) acc[a][b][m][n] = (f32x4){0.f, 0.f, 0.f, 0.f};
        tile += 8;
    }
    if (mode != 2) {
        int lz = lane; asm volatile("" : "+v"(lz));
        const unsigned nanm = bad ? 0x7fc07fc0u : 0u;
        if (mode == 0) {
            const rsrc_t rs_out = mkrs(p.out, OUT_BYTES);
            const unsigned vo = (unsigned)(lz & 15) * (GN * 2) + (unsigned)(lz >> 4) * 16;
            const unsigned so = (unsigned)((tile >> 3) * 256 + wr * 64) * (GN * 2) + (unsigned)(rank * 2048 + (tile & 7) * 256 + wc * 64) * 2;
#pragma unroll
            for (int a = 0; a < 2; ++a)
#pragma unroll
                for (int m = 0; m < 4; ++m)
#pragma unroll
                    for (int b = 0; b < 2; ++b) {
                        const f32x4 v0 = acc[a][b][m][0], v1 = acc[a][b][m][1];
                        u32x4 o; o.x = cvt_pk_bf16(silu(v0[0]), silu(v0[1])) | nanm; o.y = cvt_pk_bf16(silu(v0[2]), silu(v0[3]));
                        o.z = cvt_pk_bf16(silu(v1[0]), silu(v1[1])); o.w = cvt_pk_bf16(silu(v1[2]), silu(v1[3]));
                        __builtin_amdgcn_raw_buffer_store_b128(o, rs_out, vo + b * 64, so + (unsigned)(a * 128 + m * 16) * (GN * 2), AUX_AGENT);
                    }
        } else {
            const unsigned so = yexp_off + (unsigned)(dest * 16 + tile) * 131072u + wid * 16384;
#pragma unroll
            for (int a = 0; a < 2; ++a)
#pragma unroll
                for (int m = 0; m < 4; ++m)
#pragma unroll
                    for (int b = 0; b < 2; ++b) {
                        const f32x4 v0 = acc[a][b][m][0], v1 = acc[a][b][m][1];
                        u32x4 o; o.x = cvt_pk_bf16(v0[0], v0[1]) | nanm; o.y = cvt_pk_bf16(v0[2], v0[3]);
                        o.z = cvt_pk_bf16(v1[0], v1[1]); o.w = cvt_pk_bf16(v1[2], v1[3]);
                        __builtin_amdgcn_raw_buffer_store_b128(o, rs_me, lz * 16 + b * 1024, so + (a * 4 + m) * 2048, AUX_SYS);
                    }
            WAIT_V(0);
            if (lz == 0) __builtin_amdgcn_raw_buffer_store_b32(call, mkrs(p.ws[dest], WS_NEED), 0, F_Y + ((rank * 8 + wid) * 32 + tile) * 4, AUX_SYS);
        }
        STAMP_AT(4);
    }
    WAIT_V(0);
    if (wr == 0) BAR;
}

}

extern "C" void kernel_launch(int rank, int nranks, ncclComm_t comm,
                              hipStream_t stream,
                              void* const* d_in, const int* in_sizes, int n_in,
                              void* d_out, int out_size,
                              const int* in_dtypes, int out_dtype,
                              void* const* d_out_all,
                              void* const* d_ws_all, size_t ws_bytes) {
    (void)comm; (void)in_dtypes; (void)out_dtype; (void)d_out_all;
    if (nranks != NR || (unsigned)rank >= (unsigned)NR || n_in < 2 || ws_bytes < (size_t)WS_NEED ||
        in_sizes[0] != GM * KS || in_sizes[1] != KS * GN || out_size != (GM / NR) * GN) {
        fprintf(stderr, "kernel_launch: unexpected problem shape\n");
        return;
    }
    Params p;
    p.x = d_in[0]; p.w = d_in[1]; p.out = d_out; p.rank = rank;
    for (int i = 0; i < NR; ++i) p.ws[i] = d_ws_all[i];
    hipLaunchKernelGGL(fused_gemm_rs, dim3(256), dim3(512), 0, stream, p);
    HIP_CHECK(hipGetLastError());
}
```

```cpp
#include <hip/hip_runtime.h>
#include <rccl/rccl.h>
#include <cstdio>

#define HIP_CHECK(expr) do { hipError_t _e = (expr); if (_e != hipSuccess) { \
    fprintf(stderr, "HIP error %s at %s:%d\n", hipGetErrorString(_e), __FILE__, __LINE__); \
    return; } } while (0)

namespace {

constexpr int NR = 8;
constexpr int GM = 4096, GK = 4096, GN = 16384;
constexpr int KS = GK / NR;
constexpr unsigned MiB = 1u << 20;
constexpr unsigned UNIT = 32768;
constexpr unsigned X_BYTES = GM * KS * 2, W_BYTES = KS * GN * 2, OUT_BYTES = (GM / NR) * GN * 2;

constexpr unsigned EXP_SZ = 20 * MiB;
constexpr unsigned EXP_W = 4 * MiB;
constexpr unsigned OFF_EXP = 1 * MiB;
constexpr unsigned YEXP_SZ = 16 * MiB;
constexpr unsigned OFF_YEXP = OFF_EXP + 2 * EXP_SZ;
constexpr unsigned STG_SZ = 6 * MiB;
constexpr unsigned OFF_STG = OFF_YEXP + 2 * YEXP_SZ;
constexpr unsigned OFF_DIAG = OFF_STG + NR * STG_SZ;
constexpr unsigned WS_NEED = OFF_DIAG + 32 * MiB;
constexpr unsigned F_CNT = 0;
constexpr unsigned F_TMO = 32768;
constexpr unsigned F_PUB = 36864;
constexpr unsigned F_Y = 49152;
constexpr unsigned F_DG = 57344;
constexpr unsigned F_Q = 33792;
constexpr unsigned F_A = 65536;
constexpr unsigned F_B = 131072;
constexpr unsigned F_STAMP = 524288;

constexpr unsigned long long DEADLINE = 100000000ull;

#define PW 8
#define LAG_O 2
#define LAG_W 7

constexpr int AUX_AGENT = 16, AUX_SYS = 17;

typedef short bf16x8 __attribute__((ext_vector_type(8)));
typedef short s16x4 __attribute__((ext_vector_type(4)));
typedef float f32x4 __attribute__((ext_vector_type(4)));
typedef unsigned u32x4 __attribute__((ext_vector_type(4)));
typedef unsigned u32x2 __attribute__((ext_vector_type(2)));
#define LAS __attribute__((address_space(3)))
typedef __amdgpu_buffer_rsrc_t rsrc_t;
typedef __attribute__((address_space(1))) unsigned gu32;

struct Params {
    const void* x; const void* w; void* out;
    void* ws[NR];
    int rank;
};

__device__ __forceinline__ rsrc_t mkrs(const void* p, unsigned bytes) {
    return __builtin_amdgcn_make_buffer_rsrc((void*)p, 0, bytes, 0x00020000);
}
__device__ __forceinline__ unsigned cvt_pk_bf16(float lo, float hi) {
    typedef float f32x2 __attribute__((ext_vector_type(2)));
    typedef __bf16 bf16x2 __attribute__((ext_vector_type(2)));
    return __builtin_bit_cast(unsigned, __builtin_convertvector((f32x2){lo, hi}, bf16x2));
}
__device__ __forceinline__ float silu(float y) {
    const float c = __builtin_amdgcn_fmed3f(y, -60.0f, 60.0f);
    const float e = __builtin_amdgcn_exp2f(c * -1.44269504088896341f);
    return y * __builtin_amdgcn_rcpf(1.0f + e);
}
__device__ __forceinline__ int rfl(int v) { return __builtin_amdgcn_readfirstlane(v); }
__device__ __forceinline__ int skip2(int i, int a, int b) {
    const int lo = a < b ? a : b, hi = a < b ? b : a;
    int j = i; if (j >= lo) ++j; if (hi != lo && j >= hi) ++j; return j;
}
#define ST128(v, rs, voff, soff, aux) __builtin_amdgcn_raw_buffer_store_b128((v), (rs), (unsigned)(voff) + (unsigned)(soff), 0, (aux))
#define WAIT_V(n) asm volatile("s_waitcnt vmcnt(" #n ")" ::: "memory")
#define WAIT_L(n) asm volatile("s_waitcnt lgkmcnt(" #n ")" ::: "memory")
#define BAR __builtin_amdgcn_s_barrier()
#define SCHED __builtin_amdgcn_sched_barrier(0)

#define STAMP_AT(slot) do { } while (0)

__device__ __forceinline__ bool give_up(rsrc_t rs_me, unsigned call, unsigned long long t0) {
    const unsigned tm = __builtin_amdgcn_raw_buffer_load_b32(rs_me, 0, F_TMO, AUX_AGENT);
    if ((unsigned)rfl((int)tm) == call || __builtin_amdgcn_s_memrealtime() - t0 > DEADLINE) {
        __builtin_amdgcn_raw_buffer_store_b32(call, rs_me, 0, F_TMO, AUX_AGENT);
        return true;
    }
    __builtin_amdgcn_s_sleep(2);
    asm volatile("" ::: "memory");
    return false;
}

__global__ __launch_bounds__(512, 2) void fused_gemm_rs(const Params p) {
    __shared__ __attribute__((aligned(16))) unsigned char smem[131072];
    LAS unsigned char* lds = (LAS unsigned char*)smem;
    const int tid = threadIdx.x, lane = tid & 63, wid = rfl(tid >> 6);
    const int bid = blockIdx.x, xcd = bid & 7, slot = bid >> 3;
    const int rank = p.rank;
    const rsrc_t rs_me = mkrs(p.ws[rank], WS_NEED);

    STAMP_AT(0);
    const unsigned call = (unsigned)rfl((int)__builtin_amdgcn_raw_buffer_load_b32(rs_me, 0, F_CNT + bid * 128, AUX_AGENT)) + 1u;
    const unsigned par = call & 1u;
    const unsigned exp_off = OFF_EXP + par * EXP_SZ;
    const unsigned yexp_off = OFF_YEXP + par * YEXP_SZ;

    {
        const rsrc_t rs_x = mkrs(p.x, X_BYTES), rs_w = mkrs(p.w, W_BYTES);
        const int nb = bid - 56;
        for (int r = 0; r < 3; ++r) {
            int u;
            if (r == 0) u = bid; else { if (nb < 0) break; u = 56 + r * 200 + nb; if (u >= 640) break; }
            const int kc = u / 80, j = u - kc * 80;
            unsigned dst; u32x4 o[4];
            if (j < 64) {
                const int dest = ((j >> 3) + rank + 1) & 7, ct = j & 7;
                dst = exp_off + EXP_W + (unsigned)((dest * 8 + kc) * 8 + ct) * UNIT;
                const unsigned so = (unsigned)(kc * 64) * (GN * 2) + (unsigned)(dest * 2048 + ct * 256) * 2;
                const unsigned vo = (unsigned)(tid >> 5) * (GN * 2) + (unsigned)(tid & 31) * 16;
                u32x4 v[4];
#pragma unroll
                for (int i = 0; i < 4; ++i) v[i] = __builtin_amdgcn_raw_buffer_load_b128(rs_w, vo, so + i * 16 * (GN * 2), 0);
                if (r) __syncthreads();
#pragma unroll
                for (int i = 0; i < 4; ++i) *(LAS u32x4*)(lds + (i * 16 + (tid >> 5)) * 576 + (tid & 31) * 16) = v[i];
                __syncthreads();
#pragma unroll
                for (int i = 0; i < 4; ++i) {
                    const int F = i * 8 + wid, bj = F >> 4, wcn = (F >> 1) & 7, kk = F & 1;
                    const int col = (wcn >> 1) * 64 + bj * 32 + (wcn & 1) * 4 + (lane & 3) * 8;
                    const int row = kk * 32 + (lane >> 4) * 8 + ((lane & 15) >> 2);
                    const s16x4 t0 = __builtin_amdgcn_ds_read_tr16_b64_v4i16((LAS s16x4*)(lds + row * 576 + col * 2));
                    const s16x4 t1 = __builtin_amdgcn_ds_read_tr16_b64_v4i16((LAS s16x4*)(lds + (row + 4) * 576 + col * 2));
                    const u32x2 a = __builtin_bit_cast(u32x2, t0), b = __builtin_bit_cast(u32x2, t1);
                    o[i] = (u32x4){a.x, a.y, b.x, b.y};
                }
#pragma unroll
                for (int i = 0; i < 4; ++i) ST128(o[i], rs_me, lane * 16, dst + (i * 8 + wid) * 1024, AUX_SYS);
            } else {
                const int rt = j - 64;
                dst = exp_off + (unsigned)(kc * 16 + rt) * UNIT;
                const unsigned vo = (unsigned)(lane & 15) * (KS * 2) + (unsigned)(lane >> 4) * 16;
#pragma unroll
                for (int i = 0; i < 4; ++i) {
                    const int F = wid * 4 + i;
                    o[i] = __builtin_amdgcn_raw_buffer_load_b128(rs_x, vo, (unsigned)(rt * 256 + (F >> 1) * 16) * (KS * 2) + kc * 128 + (F & 1) * 64, 0);
                }
#pragma unroll
                for (int i = 0; i < 4; ++i) ST128(o[i], rs_me, lane * 16, dst + (wid * 4 + i) * 1024, AUX_SYS);
            }
            WAIT_V(0);
            __syncthreads();
            if (wid == 0) {
                if (r == 0 && lane == 0) __builtin_amdgcn_raw_buffer_store_b32(call, rs_me, 0, F_CNT + bid * 128, AUX_AGENT);
                if (j < 64) {
                    const int dest = ((j >> 3) + rank + 1) & 7, ct = j & 7;
                    if (dest != rank && lane == 0)
                        __builtin_amdgcn_raw_buffer_store_b32(call, mkrs(p.ws[dest], WS_NEED), 0, F_PUB + (rank * 256 + kc * 24 + ct) * 4, AUX_SYS);
                    if (lane < 8)
                        __builtin_amdgcn_raw_buffer_store_b32(call, rs_me, lane * (64 * 64 * 4), F_B + ((dest * 8 + ct) * 64 + rank * 8 + kc) * 4, AUX_AGENT);
                } else {
                    const int rt = j - 64;
                    if (lane < 8)
                        __builtin_amdgcn_raw_buffer_store_b32(call, rs_me, lane * (16 * 64 * 4), F_A + (rt * 64 + rank * 8 + kc) * 4, AUX_AGENT);
                }
            } else if (j >= 64 && lane == 0) {
                const int rt = j - 64, dest = (rank + wid) & 7;
                if ((rt >> 1) != rank)
                    __builtin_amdgcn_raw_buffer_store_b32(call, mkrs(p.ws[dest], WS_NEED), 0, F_PUB + (rank * 256 + kc * 24 + 8 + rt) * 4, AUX_SYS);
            }
        }
        __syncthreads();
    }
    STAMP_AT(1);

    if (slot < 7) {
        const int link = slot;
        const int src = (rank + 1 + link) & 7;
        const rsrc_t rs_src = mkrs(p.ws[src], WS_NEED);
        const rsrc_t rs_out = mkrs(p.out, OUT_BYTES);
        constexpr unsigned NIN = 176 * 8, NTOT = NIN + 16 * 64, NDRAW = NTOT + 8 * PW;
        if (wid >= PW) return;
        unsigned vz = 0; asm volatile("" : "+v"(vz));
        gu32* const qctr = (gu32*)((char*)p.ws[rank] + F_Q + link * 128) + vz;
        const unsigned qbase = (call - 1u) * NDRAW;
        unsigned vtick = 0;
        if (lane == 0) vtick = __hip_atomic_fetch_add(qctr, 1u, __ATOMIC_RELAXED, __HIP_MEMORY_SCOPE_AGENT);
        int code[3];
#pragma unroll
        for (int c = 0; c < 3; ++c) {
            int pos = lane + 64 * c, cd = 0; bool found = false;
            for (int s = 0; s < 8 + (LAG_W > LAG_O ? LAG_W : LAG_O); ++s) {
                if (s < 8) {
                    if (!found && pos < 8) { cd = s | 8 | (pos << 4); found = true; } pos -= 8;
                    if (!found && pos < 6) { cd = s | ((2 * skip2(pos, src, rank)) << 4); found = true; } pos -= 6;
                }
                if (s >= LAG_O && s - LAG_O < 8) { if (!found && pos < 6) { cd = (s - LAG_O) | ((2 * skip2(pos, src, rank) + 1) << 4); found = true; } pos -= 6; }
                if (s >= LAG_W && s - LAG_W < 8) { if (!found && pos < 2) { cd = (s - LAG_W) | ((2 * rank + pos) << 4); found = true; } pos -= 2; }
            }
            code[c] = found ? cd : 8;
        }
        int pub_ready = 0; bool bad = false;
        unsigned long long yav0 = 0, yav1 = 0;
        const unsigned vl16 = (unsigned)lane * 16u, v_fy = (unsigned)(lane * 4 + src * 512);
#define DRAIN() do { WAIT_V(0); SCHED; __builtin_amdgcn_s_waitcnt(0x0F70); SCHED; } while (0)
#define DRAW() do { if (lane == 0) vtick = __hip_atomic_fetch_add(qctr, 1u, __ATOMIC_RELAXED, __HIP_MEMORY_SCOPE_AGENT); } while (0)
#define TICKET() ((unsigned)rfl((int)vtick) - qbase)
#define RLOAD(off) __builtin_amdgcn_raw_buffer_load_b128(rs_src, vl16, (off), AUX_SYS)
#define LLOAD(off) __builtin_amdgcn_raw_buffer_load_b128(rs_me, vl16, (off), AUX_AGENT)
        auto in_decode = [&](unsigned t, unsigned& so, unsigned& dofs, unsigned& fl) {
            const int u = t >> 3, w = t & 7;
            if (u >= pub_ready) {
                const unsigned long long t0 = __builtin_amdgcn_s_memrealtime();
                for (;;) {
                    unsigned f[3];
#pragma unroll
                    for (int c = 0; c < 3; ++c)
                        f[c] = __builtin_amdgcn_raw_buffer_load_b32(rs_me, ((code[c] & 7) * 24 + ((code[c] & 8) ? 0 : 8) + (code[c] >> 4)) * 4, F_PUB + src * 1024, AUX_SYS);
                    const unsigned long long n0 = ~__ballot(f[0] >= call), n1 = ~__ballot(f[1] >= call), n2 = ~__ballot(f[2] >= call);
                    pub_ready = n0 ? __builtin_ctzll(n0) : n1 ? 64 + __builtin_ctzll(n1) : n2 ? 128 + __builtin_ctzll(n2) : 192;
                    if (u < pub_ready) break;
                    if (give_up(rs_me, call, t0)) { bad = true; break; }
                }
            }
            const int c0 = __builtin_amdgcn_readlane(code[0], u & 63), c1 = __builtin_amdgcn_readlane(code[1], u & 63), c2 = __builtin_amdgcn_readlane(code[2], u & 63);
            const int cd = u < 64 ? c0 : u < 128 ? c1 : c2, kc = cd & 7, idx = cd >> 4;
            if (cd & 8) {
                so = exp_off + EXP_W + (unsigned)((rank * 8 + kc) * 8 + idx) * UNIT;
                dofs = OFF_STG + src * STG_SZ + EXP_W + (unsigned)(kc * 8 + idx) * UNIT;
                fl = F_B + ((w * 64 + rank * 8 + idx) * 64 + src * 8 + kc) * 4;
            } else {
                so = exp_off + (unsigned)(kc * 16 + idx) * UNIT;
                dofs = OFF_STG + src * STG_SZ + (unsigned)(kc * 16 + idx) * UNIT;
                fl = F_A + ((w * 16 + idx) * 64 + src * 8 + kc) * 4;
            }
            so += w * 1024; dofs += w * 1024;
        };
        auto y_decode = [&](unsigned t, unsigned& ys, unsigned& ds, unsigned& os) {
            const int q = t - NIN, tl = q >> 6, g = (q >> 3) & 7, am = q & 7;
            const int bit = (tl & 7) * 8 + g;
            if (!((((tl & 8) ? yav1 : yav0) >> bit) & 1ull) && !bad) {
                const unsigned long long t0 = __builtin_amdgcn_s_memrealtime();
                for (;;) {
                    const unsigned y0 = __builtin_amdgcn_raw_buffer_load_b32(rs_me, v_fy + F_Y, 0, AUX_SYS);
                    const unsigned y1 = __builtin_amdgcn_raw_buffer_load_b32(rs_me, v_fy + F_Y + 256, 0, AUX_SYS);
                    const unsigned d0 = __builtin_amdgcn_raw_buffer_load_b32(rs_me, v_fy + F_DG, 0, AUX_AGENT);
                    const unsigned d1 = __builtin_amdgcn_raw_buffer_load_b32(rs_me, v_fy + F_DG + 256, 0, AUX_AGENT);
                    yav0 = __ballot((y0 < d0 ? y0 : d0) >= call); yav1 = __ballot((y1 < d1 ? y1 : d1) >= call);
                    if ((((tl & 8) ? yav1 : yav0) >> bit) & 1ull) break;
                    if (give_up(rs_me, call, t0)) { bad = true; break; }
                }
            }
            ys = yexp_off + (unsigned)(rank * 16 + tl) * 131072u + g * 16384 + am * 2048;
            ds = OFF_DIAG + (unsigned)(src * 16 + tl) * 262144u + g * 32768 + am * 4096;
            os = (unsigned)((tl >> 3) * 256 + (am >> 2) * 128 + (g >> 2) * 64 + (am & 3) * 16) * (GN * 2)
               + (unsigned)(src * 2048 + (tl & 7) * 256 + (g & 3) * 64) * 2;
        };
        unsigned t = TICKET();
        if (t < NIN) {
            unsigned so, dofs, fl, prev = 0;
            in_decode(t, so, dofs, fl);
            if (!bad) {
                u32x4 A = RLOAD(so), B;
                for (;;) {
                    DRAW();
                    DRAIN(); B = RLOAD(so + 8192); SCHED; ST128(A, rs_me, vl16, dofs, AUX_AGENT);
                    if (prev && lane == 0) __builtin_amdgcn_raw_buffer_store_b32(call, rs_me, 0, prev, AUX_AGENT);
                    t = TICKET();
                    DRAIN(); A = RLOAD(so + 16384); SCHED; ST128(B, rs_me, vl16, dofs + 8192, AUX_AGENT);
                    DRAIN(); B = RLOAD(so + 24576); SCHED; ST128(A, rs_me, vl16, dofs + 16384, AUX_AGENT);
                    const unsigned dlast = dofs + 24576; prev = fl;
                    bool more = t < NIN;
                    if (more) { in_decode(t, so, dofs, fl); more = !bad; }
                    DRAIN();
                    if (more) A = RLOAD(so);
                    SCHED; ST128(B, rs_me, vl16, dlast, AUX_AGENT);
                    if (!more) break;
                }
                DRAIN();
                if (lane == 0) __builtin_amdgcn_raw_buffer_store_b32(call, rs_me, 0, prev, AUX_AGENT);
            }
            while (t < NIN) { DRAW(); t = TICKET(); }
        }
        STAMP_AT(2);
        if (t < NTOT) {
            const unsigned v_out = (unsigned)(lane & 15) * (GN * 2) + (unsigned)(lane >> 4) * 16;
            auto finish = [&](const u32x4 y, const u32x4 d0, const u32x4 d1, unsigned os) {
                u32x4 o;
                o.x = cvt_pk_bf16(silu(__uint_as_float(y.x << 16) + __uint_as_float(d0.x)), silu(__uint_as_float(y.x & 0xffff0000u) + __uint_as_float(d0.y))) | (bad ? 0x7fc07fc0u : 0u);
                o.y = cvt_pk_bf16(silu(__uint_as_float(y.y << 16) + __uint_as_float(d0.z)), silu(__uint_as_float(y.y & 0xffff0000u) + __uint_as_float(d0.w)));
                o.z = cvt_pk_bf16(silu(__uint_as_float(y.z << 16) + __uint_as_float(d1.x)), silu(__uint_as_float(y.z & 0xffff0000u) + __uint_as_float(d1.y)));
                o.w = cvt_pk_bf16(silu(__uint_as_float(y.w << 16) + __uint_as_float(d1.z)), silu(__uint_as_float(y.w & 0xffff0000u) + __uint_as_float(d1.w)));
                ST128(o, rs_out, v_out, os, AUX_AGENT);
            };
            unsigned ys, ds, os;
            y_decode(t, ys, ds, os);
            u32x4 A = RLOAD(ys), a0 = LLOAD(ds), a1 = LLOAD(ds + 1024);
            DRAW();
            for (;;) {
                DRAIN();
                t = TICKET();
                const u32x4 B = RLOAD(ys + 1024), b0 = LLOAD(ds + 2048), b1 = LLOAD(ds + 3072);
                SCHED;
                finish(A, a0, a1, os);
                const unsigned ol = os + 64;
                const bool more = t < NTOT;
                if (more) y_decode(t, ys, ds, os);
                DRAIN();
                if (more) { A = RLOAD(ys); a0 = LLOAD(ds); a1 = LLOAD(ds + 1024); DRAW(); }
                SCHED;
                finish(B, b0, b1, ol);
                if (!more) break;
            }
        }
        STAMP_AT(3);
        return;
    }

    const int wr = wid >> 2, wc = wid & 3;
    int nt, seglen, mode, tile0;
    unsigned v_a, v_b, v_fa, v_fb;
    int dest;
    if (slot < 23 || slot >= 30) {
        const bool helper = slot >= 30;
        const int i = slot - 7;
        const int j = helper ? rank : ((xcd >> 1) & 1) * 4 + (i & 3), rt2 = helper ? slot - 30 : xcd & 1;
        const int rt = 2 * j + rt2, ct = helper ? xcd : (xcd >> 2) * 4 + (i >> 2);
        const bool own = (j == rank);
        const int nsrc = own ? 7 : 6;
        nt = own ? 32 : 56; seglen = nt; mode = helper ? 2 : own ? 0 : 1; dest = j; tile0 = rt2 * 8 + ct;
        const int st = own ? 2 * (lane & 31) + (helper ? 0 : 1) : lane;
        int src, kc;
        if (st < 8 || st >= 8 + 8 * nsrc) { src = rank; kc = st & 7; }
        else { const int q = st - 8; kc = q / nsrc; src = skip2(q - kc * nsrc, rank, j); }
        v_a = (src == rank ? exp_off : OFF_STG + src * STG_SZ) + (unsigned)(kc * 16 + rt) * UNIT;
        v_b = (src == rank ? exp_off + (unsigned)(rank * 64) * UNIT : OFF_STG + src * STG_SZ) + EXP_W + (unsigned)(kc * 8 + ct) * UNIT;
        v_fa = F_A + ((wid * 16 + rt) * 64 + src * 8 + kc) * 4;
        v_fb = F_B + ((wid * 64 + rank * 8 + ct) * 64 + src * 8 + kc) * 4;
    } else {
        const int slab = (rank + 1 + (slot - 23)) & 7, ct = xcd;
        nt = 16; seglen = 8; mode = 2; dest = slab; tile0 = ct;
        const int kc = lane & 7, rt = 2 * rank + ((lane >> 3) & 1);
        v_a = exp_off + (unsigned)(kc * 16 + rt) * UNIT;
        v_b = exp_off + EXP_W + (unsigned)((slab * 8 + kc) * 8 + ct) * UNIT;
        v_fa = F_A + ((wid * 16 + rt) * 64 + rank * 8 + kc) * 4;
        v_fb = F_B + ((wid * 64 + slab * 8 + ct) * 64 + rank * 8 + kc) * 4;
    }
    int ready = 0; bool bad = false;
    auto ensure = [&](int need) {
        if (need > nt) need = nt;
        if (ready >= need) return;
        const unsigned long long t0 = __builtin_amdgcn_s_memrealtime();
        __builtin_amdgcn_s_waitcnt(0xC07F);
        for (;;) {
            const unsigned fa = __builtin_amdgcn_raw_buffer_load_b32(rs_me, v_fa, 0, AUX_AGENT);
            const unsigned fb = __builtin_amdgcn_raw_buffer_load_b32(rs_me, v_fb, 0, AUX_AGENT);
            const unsigned long long nm = ~__ballot((fa < fb ? fa : fb) >= call);
            ready = nm ? __builtin_ctzll(nm) : 64;
            if (ready >= need) break;
            if (give_up(rs_me, call, t0)) { bad = true; ready = 64; break; }
        }
    };
    const unsigned vst = (unsigned)tid * 16u;
    const int ldsw = wid * 1024;
    const int aoff = wr * 8192 + lane * 16;
    int boff = 65536 + wc * 4096 + lane * 16; asm volatile("" : "+v"(boff));
#define SA(b, h) (((b) * 2 + (h)) * 16384)
#define SB(b, h) ((4 + (b) * 2 + (h)) * 16384)
#define STAGE(bufoff, goff) do { _Pragma("unroll") for (int _i = 0; _i < 2; ++_i) \
        __builtin_amdgcn_raw_ptr_buffer_load_lds(rs_me, (LAS void*)(lds + (bufoff) + ldsw + _i * 8192), 16, vst, (goff) + _i * 8192, 0, AUX_AGENT); } while (0)
#define LDA(dst, b, h) do { _Pragma("unroll") for (int m = 0; m < 4; ++m) _Pragma("unroll") for (int k = 0; k < 2; ++k) \
        dst[m][k] = *(const LAS bf16x8*)(lds + SA(b, h) + aoff + m * 2048 + k * 1024); } while (0)
#define LDB(dst, b, h) do { _Pragma("unroll") for (int n = 0; n < 2; ++n) _Pragma("unroll") for (int k = 0; k < 2; ++k) \
        dst[n][k] = *(const LAS bf16x8*)(lds + boff + (SB(b, h) - 65536 + n * 2048 + k * 1024)); } while (0)
#define MMA(ai, bj, At, Bt) do { __builtin_amdgcn_s_setprio(1); _Pragma("unroll") for (int m = 0; m < 4; ++m) _Pragma("unroll") for (int n = 0; n < 2; ++n) \
        _Pragma("unroll") for (int k = 0; k < 2; ++k) acc[ai][bj][m][n] = __builtin_amdgcn_mfma_f32_16x16x32_bf16(Bt[n][k], At[m][k], acc[ai][bj][m][n], 0, 0, 0); \
        __builtin_amdgcn_s_setprio(0); } while (0)
#define AOFF(t) ((unsigned)__builtin_amdgcn_readlane((int)v_a, (t)))
#define BOFF(t) ((unsigned)__builtin_amdgcn_readlane((int)v_b, (t)))
    f32x4 acc[2][2][4][2];
#pragma unroll
    for (int a = 0; a < 2; ++a)
#pragma unroll
        for (int b = 0; b < 2; ++b)
#pragma unroll
            for (int m = 0; m < 4; ++m)
#pragma unroll
                for (int n = 0; n < 2; ++n) acc[a][b][m][n] = (f32x4){0.f, 0.f, 0.f, 0.f};
    bf16x8 At[4][2], B0[2][2], B1[2][2];
    ensure(2);
    {
        const unsigned a0 = AOFF(0), b0 = BOFF(0), a1 = AOFF(1), b1 = BOFF(1);
        STAGE(SB(0, 0), b0); STAGE(SA(0, 0), a0); STAGE(SB(0, 1), b0 + 16384); STAGE(SA(0, 1), a0 + 16384);
        if (wr == 1) BAR;
        WAIT_V(4); BAR;
        STAGE(SB(1, 0), b1); STAGE(SA(1, 0), a1); STAGE(SB(1, 1), b1 + 16384);
        WAIT_V(6); BAR;
    }
    STAMP_AT(2);
    int tile = tile0;
    for (int t = 0;;) {
        const int seg_end = t + seglen;
        for (; t < seg_end; t += 2) {
            ensure(t + 4);
            const int t2 = (t + 2 < nt) ? t + 2 : 0;
            const unsigned a1 = AOFF(t + 1), a2 = AOFF(t2), b2 = BOFF(t2), a3 = AOFF(t2 + 1), b3 = BOFF(t2 + 1);
            LDB(B0, 0, 0); SCHED; LDA(At, 0, 0); STAGE(SA(1, 1), a1 + 16384);
            WAIT_L(8); BAR; WAIT_L(0); MMA(0, 0, At, B0); BAR; SCHED;
            LDB(B1, 0, 1); STAGE(SB(0, 0), b2);
            BAR; WAIT_L(0); MMA(0, 1, At, B1); BAR;
            LDA(At, 0, 1); STAGE(SA(0, 0), a2);
            BAR; WAIT_L(0); MMA(1, 0, At, B0); BAR; SCHED;
            STAGE(SB(0, 1), b2 + 16384);
            WAIT_V(6); BAR; MMA(1, 1, At, B1); BAR;
            LDB(B0, 1, 0); SCHED; LDA(At, 1, 0); STAGE(SA(0, 1), a2 + 16384);
            WAIT_L(8); BAR; WAIT_L(0); MMA(0, 0, At, B0); BAR; SCHED;
            LDB(B1, 1, 1); STAGE(SB(1, 0), b3);
            BAR; WAIT_L(0); MMA(0, 1, At, B1); BAR;
            LDA(At, 1, 1); STAGE(SA(1, 0), a3);
            BAR; WAIT_L(0); MMA(1, 0, At, B0); BAR; SCHED;
            STAGE(SB(1, 1), b3 + 16384);
            WAIT_V(6); BAR; MMA(1, 1, At, B1); BAR;
        }
        if (t >= nt && wr == 0) BAR;
        STAMP_AT(3);
        if (mode != 2) break;
        {
            int lz = lane; asm volatile("" : "+v"(lz));
            const unsigned nanm = bad ? 0x7fc00000u : 0u;
            const unsigned so = OFF_DIAG + (unsigned)(dest * 16 + tile) * 262144u + wid * 32768;
#pragma unroll
            for (int a = 0; a < 2; ++a)
#pragma unroll
                for (int m = 0; m < 4; ++m)
#pragma unroll
                    for (int b = 0; b < 2; ++b)
#pragma unroll
                        for (int n = 0; n < 2; ++n) {
                            u32x4 o = __builtin_bit_cast(u32x4, acc[a][b][m][n]); o.x |= nanm;
                            ST128(o, rs_me, lz * 16 + (b * 2 + n) * 1024, so + (a * 4 + m) * 4096, AUX_AGENT);
                        }
            WAIT_V(0);
            if (lz == 0) __builtin_amdgcn_raw_buffer_store_b32(call, rs_me, 0, F_DG + ((dest * 16 + tile) * 8 + wid) * 4, AUX_AGENT);
        }
        STAMP_AT(4);
        if (t >= nt) break;
#pragma unroll
        for (int a = 0; a < 2; ++a)
#pragma unroll
            for (int b = 0; b < 2; ++b)
#pragma unroll
                for (int m = 0; m < 4; ++m)
#pragma unroll
                    for (int n = 0; n < 2; ++n) acc[a][b][m][n] = (f32x4){0.f, 0.f, 0.f, 0.f};
        tile += 8;
    }
    if (mode != 2) {
        int lz = lane; asm volatile("" : "+v"(lz));
        if (mode == 0) {
            {
                const unsigned long long t0 = __builtin_amdgcn_s_memrealtime();
                while (!bad && (unsigned)rfl((int)__builtin_amdgcn_raw_buffer_load_b32(rs_me, 0, F_DG + ((rank * 16 + tile) * 8 + wid) * 4, AUX_AGENT)) < call)
                    if (give_up(rs_me, call, t0)) bad = true;
            }
            const unsigned nanm = bad ? 0x7fc07fc0u : 0u;
            const rsrc_t rs_out = mkrs(p.out, OUT_BYTES);
            const unsigned vo = (unsigned)(lz & 15) * (GN * 2) + (unsigned)(lz >> 4) * 16;
            const unsigned so = (unsigned)((tile >> 3) * 256 + wr * 64) * (GN * 2) + (unsigned)(rank * 2048 + (tile & 7) * 256 + wc * 64) * 2;
            const unsigned hs = OFF_DIAG + (unsigned)(rank * 16 + tile) * 262144u + wid * 32768;
#pragma unroll
            for (int a = 0; a < 2; ++a) {
                f32x4 h[4][2][2];
#pragma unroll
                for (int m = 0; m < 4; ++m)
#pragma unroll
                    for (int b = 0; b < 2; ++b)
#pragma unroll
                        for (int n = 0; n < 2; ++n)
                            h[m][b][n] = __builtin_bit_cast(f32x4, __builtin_amdgcn_raw_buffer_load_b128(rs_me, lz * 16 + (b * 2 + n) * 1024, hs + (a * 4 + m) * 4096, AUX_AGENT));
#pragma unroll
                for (int m = 0; m < 4; ++m)
#pragma unroll
                    for (int b = 0; b < 2; ++b) {
                        const f32x4 v0 = acc[a][b][m][0] + h[m][b][0], v1 = acc[a][b][m][1] + h[m][b][1];
                        u32x4 o; o.x = cvt_pk_bf16(silu(v0[0]), silu(v0[1])) | nanm; o.y = cvt_pk_bf16(silu(v0[2]), silu(v0[3]));
                        o.z = cvt_pk_bf16(silu(v1[0]), silu(v1[1])); o.w = cvt_pk_bf16(silu(v1[2]), silu(v1[3]));
                        ST128(o, rs_out, vo + b * 64, so + (unsigned)(a * 128 + m * 16) * (GN * 2), AUX_AGENT);
                    }
                asm volatile("" ::: "memory");
            }
        } else {
            const unsigned nanm = bad ? 0x7fc07fc0u : 0u;
            const unsigned so = yexp_off + (unsigned)(dest * 16 + tile) * 131072u + wid * 16384;
#pragma unroll
            for (int a = 0; a < 2; ++a)
#pragma unroll
                for (int m = 0; m < 4; ++m)
#pragma unroll
                    for (int b = 0; b < 2; ++b) {
                        const f32x4 v0 = acc[a][b][m][0], v1 = acc[a][b][m][1];
                        u32x4 o; o.x = cvt_pk_bf16(v0[0], v0[1]) | nanm; o.y = cvt_pk_bf16(v0[2], v0[3]);
                        o.z = cvt_pk_bf16(v1[0], v1[1]); o.w = cvt_pk_bf16(v1[2], v1[3]);
                        ST128(o, rs_me, lz * 16 + b * 1024, so + (a * 4 + m) * 2048, AUX_SYS);
                    }
            WAIT_V(0);
            if (lz == 0) __builtin_amdgcn_raw_buffer_store_b32(call, mkrs(p.ws[dest], WS_NEED), 0, F_Y + ((rank * 16 + tile) * 8 + wid) * 4, AUX_SYS);
        }
        STAMP_AT(4);
    }
    WAIT_V(0);
}

}

extern "C" void kernel_launch(int rank, int nranks, ncclComm_t comm,
                              hipStream_t stream,
                              void* const* d_in, const int* in_sizes, int n_in,
                              void* d_out, int out_size,
                              const int* in_dtypes, int out_dtype,
                              void* const* d_out_all,
                              void* const* d_ws_all, size_t ws_bytes) {
    (void)comm; (void)in_dtypes; (void)out_dtype; (void)d_out_all;
    if (nranks != NR || (unsigned)rank >= (unsigned)NR || n_in < 2 || ws_bytes < (size_t)WS_NEED ||
        in_sizes[0] != GM * KS || in_sizes[1] != KS * GN || out_size != (GM / NR) * GN) {
        fprintf(stderr, "kernel_launch: unexpected problem shape\n");
        return;
    }
    Params p;
    p.x = d_in[0]; p.w = d_in[1]; p.out = d_out; p.rank = rank;
    for (int i = 0; i < NR; ++i) p.ws[i] = d_ws_all[i];
    hipLaunchKernelGGL(fused_gemm_rs, dim3(256), dim3(512), 0, stream, p);
    HIP_CHECK(hipGetLastError());
}
```

```cpp
#include <hip/hip_runtime.h>
#include <rccl/rccl.h>
#include <cstdio>

#define HIP_CHECK(expr) do { hipError_t _e = (expr); if (_e != hipSuccess) { \
    fprintf(stderr, "HIP error %s at %s:%d\n", hipGetErrorString(_e), __FILE__, __LINE__); \
    return; } } while (0)

namespace {

constexpr int NR = 8;
constexpr int GM = 4096, GK = 4096, GN = 16384;
constexpr int KS = GK / NR;
constexpr unsigned MiB = 1u << 20;
constexpr unsigned UNIT = 32768;
constexpr unsigned X_BYTES = GM * KS * 2, W_BYTES = KS * GN * 2, OUT_BYTES = (GM / NR) * GN * 2;

constexpr unsigned EXP_SZ = 20 * MiB;
constexpr unsigned EXP_W = 4 * MiB;
constexpr unsigned OFF_EXP = 1 * MiB;
constexpr unsigned YEXP_SZ = 16 * MiB;
constexpr unsigned OFF_YEXP = OFF_EXP + 2 * EXP_SZ;
constexpr unsigned STG_SZ = 6 * MiB;
constexpr unsigned OFF_STG = OFF_YEXP + 2 * YEXP_SZ;
constexpr unsigned OFF_DIAG = OFF_STG + NR * STG_SZ;
constexpr unsigned OFF_PUSH = OFF_DIAG + 32 * MiB;
constexpr unsigned WS_NEED = OFF_PUSH + 4 * MiB;
constexpr unsigned F_CNT = 0;
constexpr unsigned F_TMO = 32768;
constexpr unsigned F_PUB = 36864;
constexpr unsigned F_Y = 49152;
constexpr unsigned F_DG = 57344;
constexpr unsigned F_Q = 33792;
constexpr unsigned F_A = 65536;
constexpr unsigned F_B = 131072;
constexpr unsigned F_STAMP = 524288;

constexpr unsigned long long DEADLINE = 100000000ull;

#define PW 8
#define NPX 3
#define LAG_O 2
#define LAG_W 7

constexpr int AUX_AGENT = 16, AUX_SYS = 17;

typedef short bf16x8 __attribute__((ext_vector_type(8)));
typedef short s16x4 __attribute__((ext_vector_type(4)));
typedef float f32x4 __attribute__((ext_vector_type(4)));
typedef unsigned u32x4 __attribute__((ext_vector_type(4)));
typedef unsigned u32x2 __attribute__((ext_vector_type(2)));
#define LAS __attribute__((address_space(3)))
typedef __amdgpu_buffer_rsrc_t rsrc_t;
typedef __attribute__((address_space(1))) unsigned gu32;

struct Params {
    const void* x; const void* w; void* out; void* me;
    int rank, pad;
    void* ws[NR];
};

__device__ __forceinline__ rsrc_t mkrs(const void* p, unsigned bytes) {
    return __builtin_amdgcn_make_buffer_rsrc((void*)p, 0, bytes, 0x00020000);
}
__device__ __forceinline__ unsigned cvt_pk_bf16(float lo, float hi) {
    typedef float f32x2 __attribute__((ext_vector_type(2)));
    typedef __bf16 bf16x2 __attribute__((ext_vector_type(2)));
    return __builtin_bit_cast(unsigned, __builtin_convertvector((f32x2){lo, hi}, bf16x2));
}
__device__ __forceinline__ float silu(float y) {
    const float c = __builtin_amdgcn_fmed3f(y, -60.0f, 60.0f);
    const float e = __builtin_amdgcn_exp2f(c * -1.44269504088896341f);
    return y * __builtin_amdgcn_rcpf(1.0f + e);
}
__device__ __forceinline__ int rfl(int v) { return __builtin_amdgcn_readfirstlane(v); }
__device__ __forceinline__ int skip2(int i, int a, int b) {
    const int lo = a < b ? a : b, hi = a < b ? b : a;
    int j = i; if (j >= lo) ++j; if (hi != lo && j >= hi) ++j; return j;
}
#define ST128(v, rs, voff, soff, aux) __builtin_amdgcn_raw_buffer_store_b128((v), (rs), (unsigned)(voff) + (unsigned)(soff), 0, (aux))
#define WAIT_V(n) asm volatile("s_waitcnt vmcnt(" #n ")" ::: "memory")
#define WAIT_L(n) asm volatile("s_waitcnt lgkmcnt(" #n ")" ::: "memory")
#define BAR __builtin_amdgcn_s_barrier()
#define SCHED __builtin_amdgcn_sched_barrier(0)

#define STAMP_AT(slot) do { } while (0)

__device__ __forceinline__ bool give_up(rsrc_t rs_me, unsigned call, unsigned long long t0) {
    const unsigned tm = __builtin_amdgcn_raw_buffer_load_b32(rs_me, 0, F_TMO, AUX_AGENT);
    if ((unsigned)rfl((int)tm) == call || __builtin_amdgcn_s_memrealtime() - t0 > DEADLINE) {
        __builtin_amdgcn_raw_buffer_store_b32(call, rs_me, 0, F_TMO, AUX_AGENT);
        return true;
    }
    __builtin_amdgcn_s_sleep(2);
    asm volatile("" ::: "memory");
    return false;
}

__global__ __launch_bounds__(512, 2) void fused_gemm_rs(const Params p) {
    __shared__ __attribute__((aligned(16))) unsigned char smem[131072];
    LAS unsigned char* lds = (LAS unsigned char*)smem;
    const int tid = threadIdx.x, lane = tid & 63, wid = rfl(tid >> 6);
    const int bid = blockIdx.x, xcd = bid & 7, slot = bid >> 3;
    const int rank = p.rank;
    const int po = xcd < NPX ? (slot - 7) * NPX + xcd : 25 * NPX + (slot - 7) * (8 - NPX) + (xcd - NPX);
    constexpr int PWV = (336 + 25 * (8 - NPX) - 1) / (25 * (8 - NPX));
    const bool pusher = slot >= 7 && xcd >= NPX && wid < PWV && (po - 25 * NPX) * PWV + wid < 336;
    const int pi = pusher ? (po - 25 * NPX) * PWV + wid : 0, pe = pi / 7;
    void* const push_ws = p.ws[pi - pe * 7];
    asm volatile("" :: "s"(p.me), "s"(p.x), "s"(p.w), "s"(rank), "s"(push_ws));
    const rsrc_t rs_me = mkrs(p.me, WS_NEED);

    STAMP_AT(0);
    const unsigned vcall = __builtin_amdgcn_raw_buffer_load_b32(rs_me, 0, F_CNT + bid * 128, AUX_AGENT);
    unsigned call = 0, exp_off = 0;

    if (slot < 7) {
        call = (unsigned)rfl((int)vcall) + 1u;
        exp_off = OFF_EXP + (call & 1u) * EXP_SZ;
        __syncthreads();
        if (tid == 0) __builtin_amdgcn_raw_buffer_store_b32(call, rs_me, 0, F_CNT + bid * 128, AUX_AGENT);
    } else {
        const rsrc_t rs_x = mkrs(p.x, X_BYTES), rs_w = mkrs(p.w, W_BYTES);
        const unsigned vx = (unsigned)(lane & 15) * (KS * 2) + (unsigned)(lane >> 4) * 16;
        const int pdest = (rank + 1 + pi - pe * 7) & 7, pw = pe / 6, pj = skip2(pe - pw * 6, rank, pdest);
        const rsrc_t rs_push = mkrs(push_ws, WS_NEED);
        if (pusher) {
            u32x4 pv[4];
#pragma unroll
            for (int i = 0; i < 4; ++i)
                pv[i] = __builtin_amdgcn_raw_buffer_load_b128(rs_x, vx, (unsigned)(pj * 512 + ((pw >> 1) + 4 * i) * 16) * (KS * 2) + (pw & 1) * 64, 0);
            SCHED;
            const unsigned d = OFF_PUSH + (unsigned)((((rfl((int)vcall) + 1) & 1) * 8 + rank) * 8 + pj) * UNIT + pw * 1024;
#pragma unroll
            for (int i = 0; i < 4; ++i) ST128(pv[i], rs_push, lane * 16, d + i * 8192, AUX_SYS);
            SCHED;
        }
        for (int r = 0; r < 4; ++r) {
            const int u = r * 200 + po;
            if (u >= 640) break;
            const int kc = u / 80, j = u - kc * 80;
            const bool isw = j < 64;
            const int dl = isw ? j >> 3 : (wid + 7) & 7, dest = (rank + 1 + dl) & 7;
            const rsrc_t rs_flag = mkrs(p.ws[dl], WS_NEED);
            const int ct = j & 7, rt = j - 64;
            u32x4 v[4];
            if (isw) {
                const unsigned so = (unsigned)(kc * 64) * (GN * 2) + (unsigned)(dest * 2048 + ct * 256) * 2;
                const unsigned vo = (unsigned)(tid >> 5) * (GN * 2) + (unsigned)(tid & 31) * 16;
#pragma unroll
                for (int i = 0; i < 4; ++i) v[i] = __builtin_amdgcn_raw_buffer_load_b128(rs_w, vo, so + i * 16 * (GN * 2), 0);
            } else {
#pragma unroll
                for (int i = 0; i < 4; ++i) {
                    const int F = wid * 4 + i;
                    v[i] = __builtin_amdgcn_raw_buffer_load_b128(rs_x, vx, (unsigned)(rt * 256 + (F >> 1) * 16) * (KS * 2) + kc * 128 + (F & 1) * 64, 0);
                }
            }
            if (r == 0) {
                SCHED;
                call = (unsigned)rfl((int)vcall) + 1u;
                exp_off = OFF_EXP + (call & 1u) * EXP_SZ;
            }
            if (isw) {
                const unsigned dst = exp_off + EXP_W + (unsigned)((dest * 8 + kc) * 8 + ct) * UNIT;
#pragma unroll
                for (int i = 0; i < 4; ++i) *(LAS u32x4*)(lds + (i * 16 + (tid >> 5)) * 576 + (tid & 31) * 16) = v[i];
                __syncthreads();
#pragma unroll
                for (int i = 0; i < 4; ++i) {
                    const int F = i * 8 + wid, bj = F >> 4, wcn = (F >> 1) & 7, kk = F & 1;
                    const int col = (wcn >> 1) * 64 + bj * 32 + (wcn & 1) * 4 + (lane & 3) * 8;
                    const int row = kk * 32 + (lane >> 4) * 8 + ((lane & 15) >> 2);
                    const s16x4 t0 = __builtin_amdgcn_ds_read_tr16_b64_v4i16((LAS s16x4*)(lds + row * 576 + col * 2));
                    const s16x4 t1 = __builtin_amdgcn_ds_read_tr16_b64_v4i16((LAS s16x4*)(lds + (row + 4) * 576 + col * 2));
                    const u32x2 a = __builtin_bit_cast(u32x2, t0), b = __builtin_bit_cast(u32x2, t1);
                    v[i] = (u32x4){a.x, a.y, b.x, b.y};
                }
#pragma unroll
                for (int i = 0; i < 4; ++i) ST128(v[i], rs_me, lane * 16, dst + (i * 8 + wid) * 1024, AUX_SYS);
            } else {
                const unsigned dst = exp_off + (unsigned)(kc * 16 + rt) * UNIT;
#pragma unroll
                for (int i = 0; i < 4; ++i) ST128(v[i], rs_me, lane * 16, dst + (wid * 4 + i) * 1024, AUX_SYS);
            }
            WAIT_V(0);
            if (r == 0 && pusher && lane == 0)
                __builtin_amdgcn_raw_buffer_store_b32(call, rs_push, 0, F_A + ((pw * 16 + 2 * pj) * 64 + rank * 8) * 4, AUX_SYS);
            __syncthreads();
            if (wid == 0) {
                if (isw) {
                    if (dest != rank && lane == 0)
                        __builtin_amdgcn_raw_buffer_store_b32(call, rs_flag, 0, F_PUB + (rank * 256 + kc * 24 + ct) * 4, AUX_SYS);
                    if (lane < 8)
                        __builtin_amdgcn_raw_buffer_store_b32(call, rs_me, lane * (64 * 64 * 4), F_B + ((dest * 8 + ct) * 64 + rank * 8 + kc) * 4, AUX_AGENT);
                } else {
                    if (lane < 8)
                        __builtin_amdgcn_raw_buffer_store_b32(call, rs_me, lane * (16 * 64 * 4), F_A + (rt * 64 + rank * 8 + kc) * 4, AUX_AGENT);
                }
                if (r == 0 && lane == 0) __builtin_amdgcn_raw_buffer_store_b32(call, rs_me, 0, F_CNT + bid * 128, AUX_AGENT);
            } else if (!isw && lane == 0) {
                if ((rt >> 1) != rank)
                    __builtin_amdgcn_raw_buffer_store_b32(call, rs_flag, 0, F_PUB + (rank * 256 + kc * 24 + 8 + rt) * 4, AUX_SYS);
            }
        }
    }
    const unsigned par = call & 1u;
    const unsigned yexp_off = OFF_YEXP + par * YEXP_SZ;
    STAMP_AT(1);

    if (slot < 7) {
        const int link = slot;
        const int src = (rank + 1 + link) & 7;
        const rsrc_t rs_src = mkrs(p.ws[link], WS_NEED);
        const rsrc_t rs_out = mkrs(p.out, OUT_BYTES);
        constexpr unsigned NIN = 170 * 8, NTOT = NIN + 16 * 64, NDRAW = NTOT + 8 * PW;
        if (wid >= PW) return;
        unsigned vz = 0; asm volatile("" : "+v"(vz));
        gu32* const qctr = (gu32*)((char*)p.me + F_Q + link * 128) + vz;
        const unsigned qbase = (call - 1u) * NDRAW;
        unsigned vtick = 0;
        if (lane == 0) vtick = __hip_atomic_fetch_add(qctr, 1u, __ATOMIC_RELAXED, __HIP_MEMORY_SCOPE_AGENT);
        int code[3];
#pragma unroll
        for (int c = 0; c < 3; ++c) {
            int pos = lane + 64 * c, cd = 0; bool found = false;
            for (int s = 0; s < 8 + (LAG_W > LAG_O ? LAG_W : LAG_O); ++s) {
                if (s < 8) {
                    if (!found && pos < 8) { cd = s | 8 | (pos << 4); found = true; } pos -= 8;
                    if (s > 0) { if (!found && pos < 6) { cd = s | ((2 * skip2(pos, src, rank)) << 4); found = true; } pos -= 6; }
                }
                if (s >= LAG_O && s - LAG_O < 8) { if (!found && pos < 6) { cd = (s - LAG_O) | ((2 * skip2(pos, src, rank) + 1) << 4); found = true; } pos -= 6; }
                if (s >= LAG_W && s - LAG_W < 8) { if (!found && pos < 2) { cd = (s - LAG_W) | ((2 * rank + pos) << 4); found = true; } pos -= 2; }
            }
            code[c] = found ? cd : 8;
        }
        int pub_ready = 0; bool bad = false;
        unsigned long long yav0 = 0, yav1 = 0;
        const unsigned vl16 = (unsigned)lane * 16u, v_fy = (unsigned)(lane * 4 + src * 512);
#define DRAIN() do { WAIT_V(0); SCHED; __builtin_amdgcn_s_waitcnt(0x0F70); SCHED; } while (0)
#define DRAW() do { if (lane == 0) vtick = __hip_atomic_fetch_add(qctr, 1u, __ATOMIC_RELAXED, __HIP_MEMORY_SCOPE_AGENT); } while (0)
#define TICKET() ((unsigned)rfl((int)vtick) - qbase)
#define RLOAD(off) __builtin_amdgcn_raw_buffer_load_b128(rs_src, vl16, (off), AUX_SYS)
#define LLOAD(off) __builtin_amdgcn_raw_buffer_load_b128(rs_me, vl16, (off), AUX_AGENT)
        auto in_decode = [&](unsigned t, unsigned& so, unsigned& dofs, unsigned& fl) {
            const int u = t >> 3, w = t & 7;
            if (u >= pub_ready) {
                const unsigned long long t0 = __builtin_amdgcn_s_memrealtime();
                for (;;) {
                    unsigned f[3];
#pragma unroll
                    for (int c = 0; c < 3; ++c)
                        f[c] = __builtin_amdgcn_raw_buffer_load_b32(rs_me, ((code[c] & 7) * 24 + ((code[c] & 8) ? 0 : 8) + (code[c] >> 4)) * 4, F_PUB + src * 1024, AUX_SYS);
                    const unsigned long long n0 = ~__ballot(f[0] >= call), n1 = ~__ballot(f[1] >= call), n2 = ~__ballot(f[2] >= call);
                    pub_ready = n0 ? __builtin_ctzll(n0) : n1 ? 64 + __builtin_ctzll(n1) : n2 ? 128 + __builtin_ctzll(n2) : 192;
                    if (u < pub_ready) break;
                    if (give_up(rs_me, call, t0)) { bad = true; break; }
                }
            }
            const int c0 = __builtin_amdgcn_readlane(code[0], u & 63), c1 = __builtin_amdgcn_readlane(code[1], u & 63), c2 = __builtin_amdgcn_readlane(code[2], u & 63);
            const int cd = u < 64 ? c0 : u < 128 ? c1 : c2, kc = cd & 7, idx = cd >> 4;
            if (cd & 8) {
                so = exp_off + EXP_W + (unsigned)((rank * 8 + kc) * 8 + idx) * UNIT;
                dofs = OFF_STG + src * STG_SZ + EXP_W + (unsigned)(kc * 8 + idx) * UNIT;
                fl = F_B + ((w * 64 + rank * 8 + idx) * 64 + src * 8 + kc) * 4;
            } else {
                so = exp_off + (unsigned)(kc * 16 + idx) * UNIT;
                dofs = OFF_STG + src * STG_SZ + (unsigned)(kc * 16 + idx) * UNIT;
                fl = F_A + ((w * 16 + idx) * 64 + src * 8 + kc) * 4;
            }
            so += w * 1024; dofs += w * 1024;
        };
        auto y_decode = [&](unsigned t, unsigned& ys, unsigned& ds, unsigned& os) {
            const int q = t - NIN, tl = q >> 6, g = (q >> 3) & 7, am = q & 7;
            const int bit = (tl & 7) * 8 + g;
            if (!((((tl & 8) ? yav1 : yav0) >> bit) & 1ull) && !bad) {
                const unsigned long long t0 = __builtin_amdgcn_s_memrealtime();
                for (;;) {
                    const unsigned y0 = __builtin_amdgcn_raw_buffer_load_b32(rs_me, v_fy + F_Y, 0, AUX_SYS);
                    const unsigned y1 = __builtin_amdgcn_raw_buffer_load_b32(rs_me, v_fy + F_Y + 256, 0, AUX_SYS);
                    const unsigned d0 = __builtin_amdgcn_raw_buffer_load_b32(rs_me, v_fy + F_DG, 0, AUX_AGENT);
                    const unsigned d1 = __builtin_amdgcn_raw_buffer_load_b32(rs_me, v_fy + F_DG + 256, 0, AUX_AGENT);
                    yav0 = __ballot((y0 < d0 ? y0 : d0) >= call); yav1 = __ballot((y1 < d1 ? y1 : d1) >= call);
                    if ((((tl & 8) ? yav1 : yav0) >> bit) & 1ull) break;
                    if (give_up(rs_me, call, t0)) { bad = true; break; }
                }
            }
            ys = yexp_off + (unsigned)(rank * 16 + tl) * 131072u + g * 16384 + am * 2048;
            ds = OFF_DIAG + (unsigned)(src * 16 + tl) * 262144u + g * 32768 + am * 4096;
            os = (unsigned)((tl >> 3) * 256 + (am >> 2) * 128 + (g >> 2) * 64 + (am & 3) * 16) * (GN * 2)
               + (unsigned)(src * 2048 + (tl & 7) * 256 + (g & 3) * 64) * 2;
        };
        unsigned t = TICKET();
        if (t < NIN) {
            unsigned so, dofs, fl, prev = 0;
            in_decode(t, so, dofs, fl);
            STAMP_AT(5);
            if (!bad) {
                u32x4 A = RLOAD(so), B;
                for (;;) {
                    DRAW();
                    DRAIN(); B = RLOAD(so + 8192); SCHED; ST128(A, rs_me, vl16, dofs, AUX_AGENT);
                    if (prev && lane == 0) __builtin_amdgcn_raw_buffer_store_b32(call, rs_me, 0, prev, AUX_AGENT);
                    t = TICKET();
                    DRAIN(); A = RLOAD(so + 16384); SCHED; ST128(B, rs_me, vl16, dofs + 8192, AUX_AGENT);
                    DRAIN(); B = RLOAD(so + 24576); SCHED; ST128(A, rs_me, vl16, dofs + 16384, AUX_AGENT);
                    const unsigned dlast = dofs + 24576; prev = fl;
                    bool more = t < NIN;
                    if (more) { in_decode(t, so, dofs, fl); more = !bad; }
                    DRAIN();
                    if (more) A = RLOAD(so);
                    SCHED; ST128(B, rs_me, vl16, dlast, AUX_AGENT);
                    if (!more) break;
                }
                DRAIN();
                if (lane == 0) __builtin_amdgcn_raw_buffer_store_b32(call, rs_me, 0, prev, AUX_AGENT);
            }
            while (t < NIN) { DRAW(); t = TICKET(); }
        }
        STAMP_AT(2);
        if (t < NTOT) {
            const unsigned v_out = (unsigned)(lane & 15) * (GN * 2) + (unsigned)(lane >> 4) * 16;
            auto finish = [&](const u32x4 y, const u32x4 d0, const u32x4 d1, unsigned os) {
                u32x4 o;
                o.x = cvt_pk_bf16(silu(__uint_as_float(y.x << 16) + __uint_as_float(d0.x)), silu(__uint_as_float(y.x & 0xffff0000u) + __uint_as_float(d0.y))) | (bad ? 0x7fc07fc0u : 0u);
                o.y = cvt_pk_bf16(silu(__uint_as_float(y.y << 16) + __uint_as_float(d0.z)), silu(__uint_as_float(y.y & 0xffff0000u) + __uint_as_float(d0.w)));
                o.z = cvt_pk_bf16(silu(__uint_as_float(y.z << 16) + __uint_as_float(d1.x)), silu(__uint_as_float(y.z & 0xffff0000u) + __uint_as_float(d1.y)));
                o.w = cvt_pk_bf16(silu(__uint_as_float(y.w << 16) + __uint_as_float(d1.z)), silu(__uint_as_float(y.w & 0xffff0000u) + __uint_as_float(d1.w)));
                ST128(o, rs_out, v_out, os, AUX_AGENT);
            };
            unsigned ys, ds, os;
            y_decode(t, ys, ds, os);
            u32x4 A = RLOAD(ys), a0 = LLOAD(ds), a1 = LLOAD(ds + 1024);
            DRAW();
            for (;;) {
                DRAIN();
                t = TICKET();
                const u32x4 B = RLOAD(ys + 1024), b0 = LLOAD(ds + 2048), b1 = LLOAD(ds + 3072);
                SCHED;
                finish(A, a0, a1, os);
                const unsigned ol = os + 64;
                const bool more = t < NTOT;
                if (more) y_decode(t, ys, ds, os);
                DRAIN();
                if (more) { A = RLOAD(ys); a0 = LLOAD(ds); a1 = LLOAD(ds + 1024); DRAW(); }
                SCHED;
                finish(B, b0, b1, ol);
                if (!more) break;
            }
        }
        STAMP_AT(3);
        return;
    }

    const int wr = wid >> 2, wc = wid & 3;
    int nt, seglen, mode, tile0;
    unsigned v_a, v_b, v_fa, v_fb;
    int dest;
    if (slot < 23 || slot >= 30) {
        const bool helper = slot >= 30;
        const int i = slot - 7;
        const int j = helper ? rank : ((xcd >> 1) & 1) * 4 + (i & 3), rt2 = helper ? slot - 30 : xcd & 1;
        const int rt = 2 * j + rt2, ct = helper ? xcd : (xcd >> 2) * 4 + (i >> 2);
        const bool own = (j == rank);
        const int nsrc = own ? 7 : 6;
        nt = own ? 32 : 56; seglen = nt; mode = helper ? 2 : own ? 0 : 1; dest = j; tile0 = rt2 * 8 + ct;
        const int st = own ? 2 * (lane & 31) + (helper ? 0 : 1) : lane;
        int src, kc;
        if (st < 8 || st >= 8 + 8 * nsrc) { src = rank; kc = st & 7; }
        else { const int q = st - 8; kc = q / nsrc; src = skip2(q - kc * nsrc, rank, j); }
        v_a = (src == rank ? exp_off : OFF_STG + src * STG_SZ) + (unsigned)(kc * 16 + rt) * UNIT;
        if (src != rank && kc == 0 && rt2 == 0 && !own) v_a = OFF_PUSH + (unsigned)((par * 8 + src) * 8 + j) * UNIT;
        v_b = (src == rank ? exp_off + (unsigned)(rank * 64) * UNIT : OFF_STG + src * STG_SZ) + EXP_W + (unsigned)(kc * 8 + ct) * UNIT;
        v_fa = F_A + ((wid * 16 + rt) * 64 + src * 8 + kc) * 4;
        v_fb = F_B + ((wid * 64 + rank * 8 + ct) * 64 + src * 8 + kc) * 4;
    } else {
        const int slab = (rank + 1 + (slot - 23)) & 7, ct = xcd;
        nt = 16; seglen = 8; mode = 2; dest = slab; tile0 = ct;
        const int kc = lane & 7, rt = 2 * rank + ((lane >> 3) & 1);
        v_a = exp_off + (unsigned)(kc * 16 + rt) * UNIT;
        v_b = exp_off + EXP_W + (unsigned)((slab * 8 + kc) * 8 + ct) * UNIT;
        v_fa = F_A + ((wid * 16 + rt) * 64 + rank * 8 + kc) * 4;
        v_fb = F_B + ((wid * 64 + slab * 8 + ct) * 64 + rank * 8 + kc) * 4;
    }
    int ready = 0; bool bad = false;
    auto ensure = [&](int need) {
        if (need > nt) need = nt;
        if (ready >= need) return;
        const unsigned long long t0 = __builtin_amdgcn_s_memrealtime();
        __builtin_amdgcn_s_waitcnt(0xC07F);
        for (;;) {
            const unsigned fa = __builtin_amdgcn_raw_buffer_load_b32(rs_me, v_fa, 0, AUX_SYS);
            const unsigned fb = __builtin_amdgcn_raw_buffer_load_b32(rs_me, v_fb, 0, AUX_AGENT);
            const unsigned long long nm = ~__ballot((fa < fb ? fa : fb) >= call);
            ready = nm ? __builtin_ctzll(nm) : 64;
            if (ready >= need) break;
            if (give_up(rs_me, call, t0)) { bad = true; ready = 64; break; }
        }
    };
    const unsigned vst = (unsigned)tid * 16u;
    const int ldsw = wid * 1024;
    const int aoff = wr * 8192 + lane * 16;
    int boff = 65536 + wc * 4096 + lane * 16; asm volatile("" : "+v"(boff));
#define SA(b, h) (((b) * 2 + (h)) * 16384)
#define SB(b, h) ((4 + (b) * 2 + (h)) * 16384)
#define STAGE(bufoff, goff) do { _Pragma("unroll") for (int _i = 0; _i < 2; ++_i) \
        __builtin_amdgcn_raw_ptr_buffer_load_lds(rs_me, (LAS void*)(lds + (bufoff) + ldsw + _i * 8192), 16, vst, (goff) + _i * 8192, 0, AUX_AGENT); } while (0)
#define LDA(dst, b, h) do { _Pragma("unroll") for (int m = 0; m < 4; ++m) _Pragma("unroll") for (int k = 0; k < 2; ++k) \
        dst[m][k] = *(const LAS bf16x8*)(lds + SA(b, h) + aoff + m * 2048 + k * 1024); } while (0)
#define LDB(dst, b, h) do { _Pragma("unroll") for (int n = 0; n < 2; ++n) _Pragma("unroll") for (int k = 0; k < 2; ++k) \
        dst[n][k] = *(const LAS bf16x8*)(lds + boff + (SB(b, h) - 65536 + n * 2048 + k * 1024)); } while (0)
#define MMA(ai, bj, At, Bt) do { __builtin_amdgcn_s_setprio(1); _Pragma("unroll") for (int m = 0; m < 4; ++m) _Pragma("unroll") for (int n = 0; n < 2; ++n) \
        _Pragma("unroll") for (int k = 0; k < 2; ++k) acc[ai][bj][m][n] = __builtin_amdgcn_mfma_f32_16x16x32_bf16(Bt[n][k], At[m][k], acc[ai][bj][m][n], 0, 0, 0); \
        __builtin_amdgcn_s_setprio(0); } while (0)
#define AOFF(t) ((unsigned)__builtin_amdgcn_readlane((int)v_a, (t)))
#define BOFF(t) ((unsigned)__builtin_amdgcn_readlane((int)v_b, (t)))
    f32x4 acc[2][2][4][2];
#pragma unroll
    for (int a = 0; a < 2; ++a)
#pragma unroll
        for (int b = 0; b < 2; ++b)
#pragma unroll
            for (int m = 0; m < 4; ++m)
#pragma unroll
                for (int n = 0; n < 2; ++n) acc[a][b][m][n] = (f32x4){0.f, 0.f, 0.f, 0.f};
    bf16x8 At[4][2], B0[2][2], B1[2][2];
    ensure(2);
    {
        const unsigned a0 = AOFF(0), b0 = BOFF(0), a1 = AOFF(1), b1 = BOFF(1);
        STAGE(SB(0, 0), b0); STAGE(SA(0, 0), a0); STAGE(SB(0, 1), b0 + 16384); STAGE(SA(0, 1), a0 + 16384);
        if (wr == 1) BAR;
        WAIT_V(4); BAR;
        STAGE(SB(1, 0), b1); STAGE(SA(1, 0), a1); STAGE(SB(1, 1), b1 + 16384);
        WAIT_V(6); BAR;
    }
    STAMP_AT(2);
    int tile = tile0;
    for (int t = 0;;) {
        const int seg_end = t + seglen;
        for (; t < seg_end; t += 2) {
            ensure(t + 4);
            const int t2 = (t + 2 < nt) ? t + 2 : 0;
            const unsigned a1 = AOFF(t + 1), a2 = AOFF(t2), b2 = BOFF(t2), a3 = AOFF(t2 + 1), b3 = BOFF(t2 + 1);
            LDB(B0, 0, 0); SCHED; LDA(At, 0, 0); STAGE(SA(1, 1), a1 + 16384);
            WAIT_L(8); BAR; WAIT_L(0); MMA(0, 0, At, B0); BAR; SCHED;
            LDB(B1, 0, 1); STAGE(SB(0, 0), b2);
            BAR; WAIT_L(0); MMA(0, 1, At, B1); BAR;
            LDA(At, 0, 1); STAGE(SA(0, 0), a2);
            BAR; WAIT_L(0); MMA(1, 0, At, B0); BAR; SCHED;
            STAGE(SB(0, 1), b2 + 16384);
            WAIT_V(6); BAR; MMA(1, 1, At, B1); BAR;
            LDB(B0, 1, 0); SCHED; LDA(At, 1, 0); STAGE(SA(0, 1), a2 + 16384);
            WAIT_L(8); BAR; WAIT_L(0); MMA(0, 0, At, B0); BAR; SCHED;
            LDB(B1, 1, 1); STAGE(SB(1, 0), b3);
            BAR; WAIT_L(0); MMA(0, 1, At, B1); BAR;
            LDA(At, 1, 1); STAGE(SA(1, 0), a3);
            BAR; WAIT_L(0); MMA(1, 0, At, B0); BAR; SCHED;
            STAGE(SB(1, 1), b3 + 16384);
            WAIT_V(6); BAR; MMA(1, 1, At, B1); BAR;
        }
        if (t >= nt && wr == 0) BAR;
        STAMP_AT(3);
        if (mode != 2) break;
        {
            int lz = lane; asm volatile("" : "+v"(lz));
            const unsigned nanm = bad ? 0x7fc00000u : 0u;
            const unsigned so = OFF_DIAG + (unsigned)(dest * 16 + tile) * 262144u + wid * 32768;
#pragma unroll
            for (int a = 0; a < 2; ++a)
#pragma unroll
                for (int m = 0; m < 4; ++m)
#pragma unroll
                    for (int b = 0; b < 2; ++b)
#pragma unroll
                        for (int n = 0; n < 2; ++n) {
                            u32x4 o = __builtin_bit_cast(u32x4, acc[a][b][m][n]); o.x |= nanm;
                            ST128(o, rs_me, lz * 16 + (b * 2 + n) * 1024, so + (a * 4 + m) * 4096, AUX_AGENT);
                        }
            WAIT_V(0);
            if (lz == 0) __builtin_amdgcn_raw_buffer_store_b32(call, rs_me, 0, F_DG + ((dest * 16 + tile) * 8 + wid) * 4, AUX_AGENT);
        }
        STAMP_AT(4);
        if (t >= nt) break;
#pragma unroll
        for (int a = 0; a < 2; ++a)
#pragma unroll
            for (int b = 0; b < 2; ++b)
#pragma unroll
                for (int m = 0; m < 4; ++m)
#pragma unroll
                    for (int n = 0; n < 2; ++n) acc[a][b][m][n] = (f32x4){0.f, 0.f, 0.f, 0.f};
        tile += 8;
    }
    if (mode != 2) {
        int lz = lane; asm volatile("" : "+v"(lz));
        if (mode == 0) {
            {
                const unsigned long long t0 = __builtin_amdgcn_s_memrealtime();
                while (!bad && (unsigned)rfl((int)__builtin_amdgcn_raw_buffer_load_b32(rs_me, 0, F_DG + ((rank * 16 + tile) * 8 + wid) * 4, AUX_AGENT)) < call)
                    if (give_up(rs_me, call, t0)) bad = true;
            }
            const unsigned nanm = bad ? 0x7fc07fc0u : 0u;
            const rsrc_t rs_out = mkrs(p.out, OUT_BYTES);
            const unsigned vo = (unsigned)(lz & 15) * (GN * 2) + (unsigned)(lz >> 4) * 16;
            const unsigned so = (unsigned)((tile >> 3) * 256 + wr * 64) * (GN * 2) + (unsigned)(rank * 2048 + (tile & 7) * 256 + wc * 64) * 2;
            const unsigned hs = OFF_DIAG + (unsigned)(rank * 16 + tile) * 262144u + wid * 32768;
#pragma unroll
            for (int a = 0; a < 2; ++a) {
                f32x4 h[4][2][2];
#pragma unroll
                for (int m = 0; m < 4; ++m)
#pragma unroll
                    for (int b = 0; b < 2; ++b)
#pragma unroll
                        for (int n = 0; n < 2; ++n)
                            h[m][b][n] = __builtin_bit_cast(f32x4, __builtin_amdgcn_raw_buffer_load_b128(rs_me, lz * 16 + (b * 2 + n) * 1024, hs + (a * 4 + m) * 4096, AUX_AGENT));
#pragma unroll
                for (int m = 0; m < 4; ++m)
#pragma unroll
                    for (int b = 0; b < 2; ++b) {
                        const f32x4 v0 = acc[a][b][m][0] + h[m][b][0], v1 = acc[a][b][m][1] + h[m][b][1];
                        u32x4 o; o.x = cvt_pk_bf16(silu(v0[0]), silu(v0[1])) | nanm; o.y = cvt_pk_bf16(silu(v0[2]), silu(v0[3]));
                        o.z = cvt_pk_bf16(silu(v1[0]), silu(v1[1])); o.w = cvt_pk_bf16(silu(v1[2]), silu(v1[3]));
                        ST128(o, rs_out, vo + b * 64, so + (unsigned)(a * 128 + m * 16) * (GN * 2), AUX_AGENT);
                    }
                asm volatile("" ::: "memory");
            }
        } else {
            const unsigned nanm = bad ? 0x7fc07fc0u : 0u;
            const unsigned so = yexp_off + (unsigned)(dest * 16 + tile) * 131072u + wid * 16384;
#pragma unroll
            for (int a = 0; a < 2; ++a)
#pragma unroll
                for (int m = 0; m < 4; ++m)
#pragma unroll
                    for (int b = 0; b < 2; ++b) {
                        const f32x4 v0 = acc[a][b][m][0], v1 = acc[a][b][m][1];
                        u32x4 o; o.x = cvt_pk_bf16(v0[0], v0[1]) | nanm; o.y = cvt_pk_bf16(v0[2], v0[3]);
                        o.z = cvt_pk_bf16(v1[0], v1[1]); o.w = cvt_pk_bf16(v1[2], v1[3]);
                        ST128(o, rs_me, lz * 16 + b * 1024, so + (a * 4 + m) * 2048, AUX_SYS);
                    }
            WAIT_V(0);
            if (lz == 0) __builtin_amdgcn_raw_buffer_store_b32(call, mkrs(p.ws[(dest - rank - 1) & 7], WS_NEED), 0, F_Y + ((rank * 16 + tile) * 8 + wid) * 4, AUX_SYS);
        }
        STAMP_AT(4);
    }
    WAIT_V(0);
}

}

extern "C" void kernel_launch(int rank, int nranks, ncclComm_t comm,
                              hipStream_t stream,
                              void* const* d_in, const int* in_sizes, int n_in,
                              void* d_out, int out_size,
                              const int* in_dtypes, int out_dtype,
                              void* const* d_out_all,
                              void* const* d_ws_all, size_t ws_bytes) {
    (void)comm; (void)in_dtypes; (void)out_dtype; (void)d_out_all;
    if (nranks != NR || (unsigned)rank >= (unsigned)NR || n_in < 2 || ws_bytes < (size_t)WS_NEED ||
        in_sizes[0] != GM * KS || in_sizes[1] != KS * GN || out_size != (GM / NR) * GN) {
        fprintf(stderr, "kernel_launch: unexpected problem shape\n");
        return;
    }
    Params p;
    p.x = d_in[0]; p.w = d_in[1]; p.out = d_out; p.me = d_ws_all[rank]; p.rank = rank; p.pad = 0;
    for (int i = 0; i < NR; ++i) p.ws[i] = d_ws_all[(rank + 1 + i) & 7];
    hipLaunchKernelGGL(fused_gemm_rs, dim3(256), dim3(512), 0, stream, p);
    HIP_CHECK(hipGetLastError());
}
```

```cpp
#include <hip/hip_runtime.h>
#include <rccl/rccl.h>
#include <cstdio>

#define HIP_CHECK(expr) do { hipError_t _e = (expr); if (_e != hipSuccess) { \
    fprintf(stderr, "HIP error %s at %s:%d\n", hipGetErrorString(_e), __FILE__, __LINE__); \
    return; } } while (0)

namespace {

constexpr int NR = 8;
constexpr int GM = 4096, GK = 4096, GN = 16384;
constexpr int KS = GK / NR;
constexpr unsigned MiB = 1u << 20;
constexpr unsigned UNIT = 32768;
constexpr unsigned X_BYTES = GM * KS * 2, W_BYTES = KS * GN * 2, OUT_BYTES = (GM / NR) * GN * 2;

constexpr unsigned EXP_SZ = 20 * MiB;
constexpr unsigned EXP_W = 4 * MiB;
constexpr unsigned OFF_EXP = 1 * MiB;
constexpr unsigned YEXP_SZ = 16 * MiB;
constexpr unsigned OFF_YEXP = OFF_EXP + 2 * EXP_SZ;
constexpr unsigned STG_SZ = 6 * MiB;
constexpr unsigned OFF_STG = OFF_YEXP + 2 * YEXP_SZ;
constexpr unsigned OFF_DIAG = OFF_STG + NR * STG_SZ;
constexpr unsigned OFF_PUSH = OFF_DIAG + 32 * MiB;
constexpr unsigned WS_NEED = OFF_PUSH + 4 * MiB;
constexpr unsigned F_CNT = 0;
constexpr unsigned F_TMO = 32768;
constexpr unsigned F_PUB = 36864;
constexpr unsigned F_Y = 49152;
constexpr unsigned F_DG = 57344;
constexpr unsigned F_Q = 33792;
constexpr unsigned F_A = 65536;
constexpr unsigned F_B = 131072;
constexpr unsigned F_STAMP = 524288;

constexpr unsigned long long DEADLINE = 100000000ull;

#define PW 8
#define NPX 3
#define FR 4
#define NPUSH 5
#define LAG_O 2
#define LAG_W 6

static_assert(NPUSH >= 1 && NPUSH <= 6 && 56 * NPUSH <= 8 * 25 * (8 - NPX), "push pieces");
static_assert(FR * 200 >= 640, "every unit is published");
constexpr int AUX_AGENT = 16, AUX_SYS = 17;

typedef short bf16x8 __attribute__((ext_vector_type(8)));
typedef short s16x4 __attribute__((ext_vector_type(4)));
typedef float f32x4 __attribute__((ext_vector_type(4)));
typedef unsigned u32x4 __attribute__((ext_vector_type(4)));
typedef unsigned u32x2 __attribute__((ext_vector_type(2)));
#define LAS __attribute__((address_space(3)))
typedef __amdgpu_buffer_rsrc_t rsrc_t;
typedef __attribute__((address_space(1))) unsigned gu32;

struct Params {
    const void* x; const void* w; void* out; void* me;
    int rank, pad;
    void* ws[NR];
};

__device__ __forceinline__ rsrc_t mkrs(const void* p, unsigned bytes) {
    return __builtin_amdgcn_make_buffer_rsrc((void*)p, 0, bytes, 0x00020000);
}
__device__ __forceinline__ unsigned cvt_pk_bf16(float lo, float hi) {
    typedef float f32x2 __attribute__((ext_vector_type(2)));
    typedef __bf16 bf16x2 __attribute__((ext_vector_type(2)));
    return __builtin_bit_cast(unsigned, __builtin_convertvector((f32x2){lo, hi}, bf16x2));
}
__device__ __forceinline__ float silu(float y) {
    const float c = __builtin_amdgcn_fmed3f(y, -60.0f, 60.0f);
    const float e = __builtin_amdgcn_exp2f(c * -1.44269504088896341f);
    return y * __builtin_amdgcn_rcpf(1.0f + e);
}
__device__ __forceinline__ int rfl(int v) { return __builtin_amdgcn_readfirstlane(v); }
__device__ __forceinline__ int skip2(int i, int a, int b) {
    const int lo = a < b ? a : b, hi = a < b ? b : a;
    int j = i; if (j >= lo) ++j; if (hi != lo && j >= hi) ++j; return j;
}
#define ST128(v, rs, voff, soff, aux) __builtin_amdgcn_raw_buffer_store_b128((v), (rs), (unsigned)(voff) + (unsigned)(soff), 0, (aux))
#define WAIT_V(n) asm volatile("s_waitcnt vmcnt(" #n ")" ::: "memory")
#define WAIT_L(n) asm volatile("s_waitcnt lgkmcnt(" #n ")" ::: "memory")
#define BAR __builtin_amdgcn_s_barrier()
#define SCHED __builtin_amdgcn_sched_barrier(0)

#define STAMP_AT(slot) do { } while (0)
#define STAMP_T(t) do { } while (0)

__device__ __forceinline__ bool give_up(rsrc_t rs_me, unsigned call, unsigned long long t0) {
    const unsigned tm = __builtin_amdgcn_raw_buffer_load_b32(rs_me, 0, F_TMO, AUX_AGENT);
    if ((unsigned)rfl((int)tm) == call || __builtin_amdgcn_s_memrealtime() - t0 > DEADLINE) {
        __builtin_amdgcn_raw_buffer_store_b32(call, rs_me, 0, F_TMO, AUX_AGENT);
        return true;
    }
    __builtin_amdgcn_s_sleep(2);
    asm volatile("" ::: "memory");
    return false;
}

__global__ __launch_bounds__(512, 2) void fused_gemm_rs(const Params p) {
    __shared__ __attribute__((aligned(16))) unsigned char smem[131072];
    LAS unsigned char* lds = (LAS unsigned char*)smem;
    const int tid = threadIdx.x, lane = tid & 63, wid = rfl(tid >> 6);
    const int bid = blockIdx.x, xcd = bid & 7, slot = bid >> 3;
    const int rank = p.rank;
    const int po = xcd < NPX ? (slot - 7) * NPX + xcd : 25 * NPX + (slot - 7) * (8 - NPX) + (xcd - NPX);
    constexpr int PWV = (56 * NPUSH + 25 * (8 - NPX) - 1) / (25 * (8 - NPX));
    const bool pusher = slot >= 7 && xcd >= NPX && wid < PWV && (po - 25 * NPX) * PWV + wid < 56 * NPUSH;
    const int pi = pusher ? (po - 25 * NPX) * PWV + wid : 0, pe = pi / 7;
    void* const push_ws = p.ws[pi - pe * 7];
    asm volatile("" :: "s"(p.me), "s"(p.x), "s"(p.w), "s"(rank), "s"(push_ws));
    const rsrc_t rs_me = mkrs(p.me, WS_NEED);

    STAMP_AT(0);
    const unsigned vcall = __builtin_amdgcn_raw_buffer_load_b32(rs_me, 0, F_CNT + bid * 128, AUX_AGENT);
    unsigned call = 0, exp_off = 0;

    if (slot < 7) {
        call = (unsigned)rfl((int)vcall) + 1u;
        exp_off = OFF_EXP + (call & 1u) * EXP_SZ;
        __syncthreads();
        if (tid == 0) __builtin_amdgcn_raw_buffer_store_b32(call, rs_me, 0, F_CNT + bid * 128, AUX_AGENT);
    } else {
        const rsrc_t rs_x = mkrs(p.x, X_BYTES), rs_w = mkrs(p.w, W_BYTES);
        const unsigned vx = (unsigned)(lane & 15) * (KS * 2) + (unsigned)(lane >> 4) * 16;
        const int pdest = (rank + 1 + pi - pe * 7) & 7, pw = pe / NPUSH, pu = pe - pw * NPUSH;
        const int prt = 2 * skip2(pu, rank, pdest);
        const rsrc_t rs_push = mkrs(push_ws, WS_NEED);
        if (pusher) {
            u32x4 pv[4];
#pragma unroll
            for (int i = 0; i < 4; ++i)
                pv[i] = __builtin_amdgcn_raw_buffer_load_b128(rs_x, vx, (unsigned)(prt * 256 + ((pw >> 1) + 4 * i) * 16) * (KS * 2) + (pw & 1) * 64, 0);
            SCHED;
            const unsigned d = OFF_PUSH + (unsigned)((((rfl((int)vcall) + 1) & 1) * 8 + rank) * 8 + (prt >> 1)) * UNIT + pw * 1024;
#pragma unroll
            for (int i = 0; i < 4; ++i) ST128(pv[i], rs_push, lane * 16, d + i * 8192, AUX_SYS);
            SCHED;
        }
        for (int r = 0; r < FR; ++r) {
            const int u = xcd < NPX ? r * (25 * NPX) + po : (FR - 1) * (25 * NPX) + r * (25 * (8 - NPX)) + po;
            if (u >= 640) break;
            const int kc = u / 80, j = u - kc * 80;
            const bool isw = j < 64;
            const int dl = isw ? j >> 3 : (wid + 7) & 7, dest = (rank + 1 + dl) & 7;
            const rsrc_t rs_flag = mkrs(p.ws[dl], WS_NEED);
            const int ct = j & 7, rt = j - 64;
            u32x4 v[4];
            if (isw) {
                const unsigned so = (unsigned)(kc * 64) * (GN * 2) + (unsigned)(dest * 2048 + ct * 256) * 2;
                const unsigned vo = (unsigned)(tid >> 5) * (GN * 2) + (unsigned)(tid & 31) * 16;
#pragma unroll
                for (int i = 0; i < 4; ++i) v[i] = __builtin_amdgcn_raw_buffer_load_b128(rs_w, vo, so + i * 16 * (GN * 2), 0);
            } else {
#pragma unroll
                for (int i = 0; i < 4; ++i) {
                    const int F = wid * 4 + i;
                    v[i] = __builtin_amdgcn_raw_buffer_load_b128(rs_x, vx, (unsigned)(rt * 256 + (F >> 1) * 16) * (KS * 2) + kc * 128 + (F & 1) * 64, 0);
                }
            }
            if (r == 0) {
                SCHED;
                call = (unsigned)rfl((int)vcall) + 1u;
                exp_off = OFF_EXP + (call & 1u) * EXP_SZ;
            }
            if (isw) {
                const unsigned dst = exp_off + EXP_W + (unsigned)((dest * 8 + kc) * 8 + ct) * UNIT;
#pragma unroll
                for (int i = 0; i < 4; ++i) *(LAS u32x4*)(lds + (i * 16 + (tid >> 5)) * 576 + (tid & 31) * 16) = v[i];
                __syncthreads();
#pragma unroll
                for (int i = 0; i < 4; ++i) {
                    const int F = i * 8 + wid, bj = F >> 4, wcn = (F >> 1) & 7, kk = F & 1;
                    const int col = (wcn >> 1) * 64 + bj * 32 + (wcn & 1) * 4 + (lane & 3) * 8;
                    const int row = kk * 32 + (lane >> 4) * 8 + ((lane & 15) >> 2);
                    const s16x4 t0 = __builtin_amdgcn_ds_read_tr16_b64_v4i16((LAS s16x4*)(lds + row * 576 + col * 2));
                    const s16x4 t1 = __builtin_amdgcn_ds_read_tr16_b64_v4i16((LAS s16x4*)(lds + (row + 4) * 576 + col * 2));
                    const u32x2 a = __builtin_bit_cast(u32x2, t0), b = __builtin_bit_cast(u32x2, t1);
                    v[i] = (u32x4){a.x, a.y, b.x, b.y};
                }
#pragma unroll
                for (int i = 0; i < 4; ++i) ST128(v[i], rs_me, lane * 16, dst + (i * 8 + wid) * 1024, AUX_SYS);
            } else {
                const unsigned dst = exp_off + (unsigned)(kc * 16 + rt) * UNIT;
#pragma unroll
                for (int i = 0; i < 4; ++i) ST128(v[i], rs_me, lane * 16, dst + (wid * 4 + i) * 1024, AUX_SYS);
            }
            WAIT_V(0);
            if (r == 0 && pusher && lane == 0)
                __builtin_amdgcn_raw_buffer_store_b32(call, rs_push, 0, F_A + ((pw * 16 + prt) * 64 + rank * 8) * 4, AUX_SYS);
            __syncthreads();
            if (wid == 0) {
                if (isw) {
                    if (dest != rank && lane == 0)
                        __builtin_amdgcn_raw_buffer_store_b32(call, rs_flag, 0, F_PUB + (rank * 256 + kc * 24 + ct) * 4, AUX_SYS);
                    if (lane < 8)
                        __builtin_amdgcn_raw_buffer_store_b32(call, rs_me, lane * (64 * 64 * 4), F_B + ((dest * 8 + ct) * 64 + rank * 8 + kc) * 4, AUX_AGENT);
                } else {
                    if (lane < 8)
                        __builtin_amdgcn_raw_buffer_store_b32(call, rs_me, lane * (16 * 64 * 4), F_A + (rt * 64 + rank * 8 + kc) * 4, AUX_AGENT);
                }
                if (r == 0 && lane == 0) __builtin_amdgcn_raw_buffer_store_b32(call, rs_me, 0, F_CNT + bid * 128, AUX_AGENT);
            } else if (!isw && lane == 0) {
                if ((rt >> 1) != rank)
                    __builtin_amdgcn_raw_buffer_store_b32(call, rs_flag, 0, F_PUB + (rank * 256 + kc * 24 + 8 + rt) * 4, AUX_SYS);
            }
        }
    }
    const unsigned par = call & 1u;
    const unsigned yexp_off = OFF_YEXP + par * YEXP_SZ;
    STAMP_AT(1);

    if (slot < 7) {
        const int link = slot;
        const int src = (rank + 1 + link) & 7;
        const rsrc_t rs_src = mkrs(p.ws[link], WS_NEED);
        const rsrc_t rs_out = mkrs(p.out, OUT_BYTES);
        constexpr unsigned NIN = (176 - NPUSH) * 8, NTOT = NIN + 16 * 64, NDRAW = NTOT + 8 * PW;
        if (wid >= PW) return;
        unsigned vz = 0; asm volatile("" : "+v"(vz));
        gu32* const qctr = (gu32*)((char*)p.me + F_Q + link * 128) + vz;
        const unsigned qbase = (call - 1u) * NDRAW;
        unsigned vtick = 0;
        if (lane == 0) vtick = __hip_atomic_fetch_add(qctr, 1u, __ATOMIC_RELAXED, __HIP_MEMORY_SCOPE_AGENT);
        int code[3];
#pragma unroll
        for (int c = 0; c < 3; ++c) {
            int pos = lane + 64 * c, cd = 0; bool found = false;
            for (int s = 0; s < 8 + (LAG_W > LAG_O ? LAG_W : LAG_O); ++s) {
                if (s < 8) {
                    if (!found && pos < 8) { cd = s | 8 | (pos << 4); found = true; } pos -= 8;
                    if (s == 0) pos += NPUSH;
                    if (!found && pos < 6) { cd = s | ((2 * skip2(pos, src, rank)) << 4); found = true; } pos -= 6;
                }
                if (s >= LAG_O && s - LAG_O < 8) {
                    if (!found && pos < 6) { cd = (s - LAG_O) | ((2 * skip2(pos, src, rank) + 1) << 4); found = true; } pos -= 6;
                }
                if (s >= LAG_W && s - LAG_W < 8) { if (!found && pos < 2) { cd = (s - LAG_W) | ((2 * rank + pos) << 4); found = true; } pos -= 2; }
            }
            code[c] = found ? cd : 8;
        }
        int pub_ready = 0; bool bad = false;
        unsigned long long yav0 = 0, yav1 = 0;
        const unsigned vl16 = (unsigned)lane * 16u, v_fy = (unsigned)(lane * 4 + src * 512);
#define DRAIN() do { WAIT_V(0); SCHED; __builtin_amdgcn_s_waitcnt(0x0F70); SCHED; } while (0)
#define DRAW() do { if (lane == 0) vtick = __hip_atomic_fetch_add(qctr, 1u, __ATOMIC_RELAXED, __HIP_MEMORY_SCOPE_AGENT); } while (0)
#define TICKET() ((unsigned)rfl((int)vtick) - qbase)
#define RLOAD(off) __builtin_amdgcn_raw_buffer_load_b128(rs_src, vl16, (off), AUX_SYS)
#define LLOAD(off) __builtin_amdgcn_raw_buffer_load_b128(rs_me, vl16, (off), AUX_AGENT)
        auto in_decode = [&](unsigned t, unsigned& so, unsigned& dofs, unsigned& fl) {
            const int u = t >> 3, w = t & 7;
            if (u >= pub_ready) {
                const unsigned long long t0 = __builtin_amdgcn_s_memrealtime();
                for (;;) {
                    unsigned f[3];
#pragma unroll
                    for (int c = 0; c < 3; ++c)
                        f[c] = __builtin_amdgcn_raw_buffer_load_b32(rs_me, ((code[c] & 7) * 24 + ((code[c] & 8) ? 0 : 8) + (code[c] >> 4)) * 4, F_PUB + src * 1024, AUX_SYS);
                    const unsigned long long n0 = ~__ballot(f[0] >= call), n1 = ~__ballot(f[1] >= call), n2 = ~__ballot(f[2] >= call);
                    pub_ready = n0 ? __builtin_ctzll(n0) : n1 ? 64 + __builtin_ctzll(n1) : n2 ? 128 + __builtin_ctzll(n2) : 192;
                    if (u < pub_ready) break;
                    if (give_up(rs_me, call, t0)) { bad = true; break; }
                }
            }
            const int c0 = __builtin_amdgcn_readlane(code[0], u & 63), c1 = __builtin_amdgcn_readlane(code[1], u & 63), c2 = __builtin_amdgcn_readlane(code[2], u & 63);
            const int cd = u < 64 ? c0 : u < 128 ? c1 : c2, kc = cd & 7, idx = cd >> 4;
            if (cd & 8) {
                so = exp_off + EXP_W + (unsigned)((rank * 8 + kc) * 8 + idx) * UNIT;
                dofs = OFF_STG + src * STG_SZ + EXP_W + (unsigned)(kc * 8 + idx) * UNIT;
                fl = F_B + ((w * 64 + rank * 8 + idx) * 64 + src * 8 + kc) * 4;
            } else {
                so = exp_off + (unsigned)(kc * 16 + idx) * UNIT;
                dofs = OFF_STG + src * STG_SZ + (unsigned)(kc * 16 + idx) * UNIT;
                fl = F_A + ((w * 16 + idx) * 64 + src * 8 + kc) * 4;
            }
            so += w * 1024; dofs += w * 1024;
            STAMP_T(t);
        };
        auto y_decode = [&](unsigned t, unsigned& ys, unsigned& ds, unsigned& os) {
            const int q = t - NIN, tl = q >> 6, g = (q >> 3) & 7, am = q & 7;
            const int bit = (tl & 7) * 8 + g;
            if (!((((tl & 8) ? yav1 : yav0) >> bit) & 1ull) && !bad) {
                const unsigned long long t0 = __builtin_amdgcn_s_memrealtime();
                for (;;) {
                    const unsigned y0 = __builtin_amdgcn_raw_buffer_load_b32(rs_me, v_fy + F_Y, 0, AUX_SYS);
                    const unsigned y1 = __builtin_amdgcn_raw_buffer_load_b32(rs_me, v_fy + F_Y + 256, 0, AUX_SYS);
                    const unsigned d0 = __builtin_amdgcn_raw_buffer_load_b32(rs_me, v_fy + F_DG, 0, AUX_AGENT);
                    const unsigned d1 = __builtin_amdgcn_raw_buffer_load_b32(rs_me, v_fy + F_DG + 256, 0, AUX_AGENT);
                    yav0 = __ballot((y0 < d0 ? y0 : d0) >= call); yav1 = __ballot((y1 < d1 ? y1 : d1) >= call);
                    if ((((tl & 8) ? yav1 : yav0) >> bit) & 1ull) break;
                    if (give_up(rs_me, call, t0)) { bad = true; break; }
                }
            }
            ys = yexp_off + (unsigned)(rank * 16 + tl) * 131072u + g * 16384 + am * 2048;
            ds = OFF_DIAG + (unsigned)(src * 16 + tl) * 262144u + g * 32768 + am * 4096;
            os = (unsigned)((tl >> 3) * 256 + (am >> 2) * 128 + (g >> 2) * 64 + (am & 3) * 16) * (GN * 2)
               + (unsigned)(src * 2048 + (tl & 7) * 256 + (g & 3) * 64) * 2;
            STAMP_T(t);
        };
        unsigned t = TICKET();
        if (t < NIN) {
            unsigned so, dofs, fl, prev = 0;
            in_decode(t, so, dofs, fl);
            STAMP_AT(5);
            if (!bad) {
                u32x4 A = RLOAD(so), B;
                for (;;) {
                    DRAW();
                    DRAIN(); B = RLOAD(so + 8192); SCHED; ST128(A, rs_me, vl16, dofs, AUX_AGENT);
                    if (prev && lane == 0) __builtin_amdgcn_raw_buffer_store_b32(call, rs_me, 0, prev, AUX_AGENT);
                    t = TICKET();
                    DRAIN(); A = RLOAD(so + 16384); SCHED; ST128(B, rs_me, vl16, dofs + 8192, AUX_AGENT);
                    DRAIN(); B = RLOAD(so + 24576); SCHED; ST128(A, rs_me, vl16, dofs + 16384, AUX_AGENT);
                    const unsigned dlast = dofs + 24576; prev = fl;
                    bool more = t < NIN;
                    if (more) { in_decode(t, so, dofs, fl); more = !bad; }
                    DRAIN();
                    if (more) A = RLOAD(so);
                    SCHED; ST128(B, rs_me, vl16, dlast, AUX_AGENT);
                    if (!more) break;
                }
                DRAIN();
                if (lane == 0) __builtin_amdgcn_raw_buffer_store_b32(call, rs_me, 0, prev, AUX_AGENT);
            }
            while (t < NIN) { DRAW(); t = TICKET(); }
        }
        STAMP_AT(2);
        if (t < NTOT) {
            const unsigned v_out = (unsigned)(lane & 15) * (GN * 2) + (unsigned)(lane >> 4) * 16;
            auto finish = [&](const u32x4 y, const u32x4 d0, const u32x4 d1, unsigned os) {
                u32x4 o;
                o.x = cvt_pk_bf16(silu(__uint_as_float(y.x << 16) + __uint_as_float(d0.x)), silu(__uint_as_float(y.x & 0xffff0000u) + __uint_as_float(d0.y))) | (bad ? 0x7fc07fc0u : 0u);
                o.y = cvt_pk_bf16(silu(__uint_as_float(y.y << 16) + __uint_as_float(d0.z)), silu(__uint_as_float(y.y & 0xffff0000u) + __uint_as_float(d0.w)));
                o.z = cvt_pk_bf16(silu(__uint_as_float(y.z << 16) + __uint_as_float(d1.x)), silu(__uint_as_float(y.z & 0xffff0000u) + __uint_as_float(d1.y)));
                o.w = cvt_pk_bf16(silu(__uint_as_float(y.w << 16) + __uint_as_float(d1.z)), silu(__uint_as_float(y.w & 0xffff0000u) + __uint_as_float(d1.w)));
                ST128(o, rs_out, v_out, os, AUX_AGENT);
            };
            unsigned ys, ds, os;
            y_decode(t, ys, ds, os);
            u32x4 A = RLOAD(ys), a0 = LLOAD(ds), a1 = LLOAD(ds + 1024);
            DRAW();
            for (;;) {
                DRAIN();
                t = TICKET();
                const u32x4 B = RLOAD(ys + 1024), b0 = LLOAD(ds + 2048), b1 = LLOAD(ds + 3072);
                SCHED;
                finish(A, a0, a1, os);
                const unsigned ol = os + 64;
                const bool more = t < NTOT;
                if (more) y_decode(t, ys, ds, os);
                DRAIN();
                if (more) { A = RLOAD(ys); a0 = LLOAD(ds); a1 = LLOAD(ds + 1024); DRAW(); }
                SCHED;
                finish(B, b0, b1, ol);
                if (!more) break;
            }
        }
        STAMP_AT(3);
        return;
    }

    const int wr = wid >> 2, wc = wid & 3;
    int nt, seglen, mode, tile0;
    unsigned v_a, v_b, v_fa, v_fb;
    int dest;
    if (slot < 23 || slot >= 30) {
        const bool helper = slot >= 30;
        const int i = slot - 7;
        const int j = helper ? rank : ((xcd >> 1) & 1) * 4 + (i & 3), rt2 = helper ? slot - 30 : xcd & 1;
        const int rt = 2 * j + rt2, ct = helper ? xcd : (xcd >> 2) * 4 + (i >> 2);
        const bool own = (j == rank);
        const int nsrc = own ? 7 : 6;
        nt = own ? 32 : 56; seglen = nt; mode = helper ? 2 : own ? 0 : 1; dest = j; tile0 = rt2 * 8 + ct;
        const int st = own ? 2 * (lane & 31) + (helper ? 0 : 1) : lane;
        int src, kc;
        if (st < 8 || st >= 8 + 8 * nsrc) { src = rank; kc = st & 7; }
        else { const int q = st - 8; kc = q / nsrc; src = skip2(q - kc * nsrc, rank, j); }
        v_a = (src == rank ? exp_off : OFF_STG + src * STG_SZ) + (unsigned)(kc * 16 + rt) * UNIT;
        if (src != rank && kc == 0 && !own && rt2 == 0 && j - (j > src) - (j > rank) < NPUSH)
            v_a = OFF_PUSH + (unsigned)((par * 8 + src) * 8 + j) * UNIT;
        v_b = (src == rank ? exp_off + (unsigned)(rank * 64) * UNIT : OFF_STG + src * STG_SZ) + EXP_W + (unsigned)(kc * 8 + ct) * UNIT;
        v_fa = F_A + ((wid * 16 + rt) * 64 + src * 8 + kc) * 4;
        v_fb = F_B + ((wid * 64 + rank * 8 + ct) * 64 + src * 8 + kc) * 4;
    } else {
        const int slab = (rank + 1 + (slot - 23)) & 7, ct = xcd;
        nt = 16; seglen = 8; mode = 2; dest = slab; tile0 = ct;
        const int kc = lane & 7, rt = 2 * rank + ((lane >> 3) & 1);
        v_a = exp_off + (unsigned)(kc * 16 + rt) * UNIT;
        v_b = exp_off + EXP_W + (unsigned)((slab * 8 + kc) * 8 + ct) * UNIT;
        v_fa = F_A + ((wid * 16 + rt) * 64 + rank * 8 + kc) * 4;
        v_fb = F_B + ((wid * 64 + slab * 8 + ct) * 64 + rank * 8 + kc) * 4;
    }
    int ready = 0; bool bad = false;
    auto ensure = [&](int need) {
        if (need > nt) need = nt;
        if (ready >= need) return;
        const unsigned long long t0 = __builtin_amdgcn_s_memrealtime();
        __builtin_amdgcn_s_waitcnt(0xC07F);
        for (;;) {
            const unsigned fa = __builtin_amdgcn_raw_buffer_load_b32(rs_me, v_fa, 0, AUX_SYS);
            const unsigned fb = __builtin_amdgcn_raw_buffer_load_b32(rs_me, v_fb, 0, AUX_AGENT);
            const unsigned long long nm = ~__ballot((fa < fb ? fa : fb) >= call);
            ready = nm ? __builtin_ctzll(nm) : 64;
            if (ready >= need) break;
            if (give_up(rs_me, call, t0)) { bad = true; ready = 64; break; }
        }
    };
    const unsigned vst = (unsigned)tid * 16u;
    const int ldsw = wid * 1024;
    const int aoff = wr * 8192 + lane * 16;
    int boff = 65536 + wc * 4096 + lane * 16; asm volatile("" : "+v"(boff));
#define SA(b, h) (((b) * 2 + (h)) * 16384)
#define SB(b, h) ((4 + (b) * 2 + (h)) * 16384)
#define STAGE(bufoff, goff) do { _Pragma("unroll") for (int _i = 0; _i < 2; ++_i) \
        __builtin_amdgcn_raw_ptr_buffer_load_lds(rs_me, (LAS void*)(lds + (bufoff) + ldsw + _i * 8192), 16, vst, (goff) + _i * 8192, 0, AUX_AGENT); } while (0)
#define LDA(dst, b, h) do { _Pragma("unroll") for (int m = 0; m < 4; ++m) _Pragma("unroll") for (int k = 0; k < 2; ++k) \
        dst[m][k] = *(const LAS bf16x8*)(lds + SA(b, h) + aoff + m * 2048 + k * 1024); } while (0)
#define LDB(dst, b, h) do { _Pragma("unroll") for (int n = 0; n < 2; ++n) _Pragma("unroll") for (int k = 0; k < 2; ++k) \
        dst[n][k] = *(const LAS bf16x8*)(lds + boff + (SB(b, h) - 65536 + n * 2048 + k * 1024)); } while (0)
#define MMA(ai, bj, At, Bt) do { __builtin_amdgcn_s_setprio(1); _Pragma("unroll") for (int m = 0; m < 4; ++m) _Pragma("unroll") for (int n = 0; n < 2; ++n) \
        _Pragma("unroll") for (int k = 0; k < 2; ++k) acc[ai][bj][m][n] = __builtin_amdgcn_mfma_f32_16x16x32_bf16(Bt[n][k], At[m][k], acc[ai][bj][m][n], 0, 0, 0); \
        __builtin_amdgcn_s_setprio(0); } while (0)
#define AOFF(t) ((unsigned)__builtin_amdgcn_readlane((int)v_a, (t)))
#define BOFF(t) ((unsigned)__builtin_amdgcn_readlane((int)v_b, (t)))
    f32x4 acc[2][2][4][2];
#pragma unroll
    for (int a = 0; a < 2; ++a)
#pragma unroll
        for (int b = 0; b < 2; ++b)
#pragma unroll
            for (int m = 0; m < 4; ++m)
#pragma unroll
                for (int n = 0; n < 2; ++n) acc[a][b][m][n] = (f32x4){0.f, 0.f, 0.f, 0.f};
    bf16x8 At[4][2], B0[2][2], B1[2][2];
    ensure(2);
    {
        const unsigned a0 = AOFF(0), b0 = BOFF(0), a1 = AOFF(1), b1 = BOFF(1);
        STAGE(SB(0, 0), b0); STAGE(SA(0, 0), a0); STAGE(SB(0, 1), b0 + 16384); STAGE(SA(0, 1), a0 + 16384);
        if (wr == 1) BAR;
        WAIT_V(4); BAR;
        STAGE(SB(1, 0), b1); STAGE(SA(1, 0), a1); STAGE(SB(1, 1), b1 + 16384);
        WAIT_V(6); BAR;
    }
    STAMP_AT(2);
    int tile = tile0;
    for (int t = 0;;) {
        const int seg_end = t + seglen;
        for (; t < seg_end; t += 2) {
            ensure(t + 4);
            const int t2 = (t + 2 < nt) ? t + 2 : 0;
            const unsigned a1 = AOFF(t + 1), a2 = AOFF(t2), b2 = BOFF(t2), a3 = AOFF(t2 + 1), b3 = BOFF(t2 + 1);
            LDB(B0, 0, 0); SCHED; LDA(At, 0, 0); STAGE(SA(1, 1), a1 + 16384);
            WAIT_L(8); BAR; WAIT_L(0); MMA(0, 0, At, B0); BAR; SCHED;
            LDB(B1, 0, 1); STAGE(SB(0, 0), b2);
            BAR; WAIT_L(0); MMA(0, 1, At, B1); BAR;
            LDA(At, 0, 1); STAGE(SA(0, 0), a2);
            BAR; WAIT_L(0); MMA(1, 0, At, B0); BAR; SCHED;
            STAGE(SB(0, 1), b2 + 16384);
            WAIT_V(6); BAR; MMA(1, 1, At, B1); BAR;
            LDB(B0, 1, 0); SCHED; LDA(At, 1, 0); STAGE(SA(0, 1), a2 + 16384);
            WAIT_L(8); BAR; WAIT_L(0); MMA(0, 0, At, B0); BAR; SCHED;
            LDB(B1, 1, 1); STAGE(SB(1, 0), b3);
            BAR; WAIT_L(0); MMA(0, 1, At, B1); BAR;
            LDA(At, 1, 1); STAGE(SA(1, 0), a3);
            BAR; WAIT_L(0); MMA(1, 0, At, B0); BAR; SCHED;
            STAGE(SB(1, 1), b3 + 16384);
            WAIT_V(6); BAR; MMA(1, 1, At, B1); BAR;
        }
        if (t >= nt && wr == 0) BAR;
        STAMP_AT(3);
        if (mode != 2) break;
        {
            int lz = lane; asm volatile("" : "+v"(lz));
            const unsigned nanm = bad ? 0x7fc00000u : 0u;
            const unsigned so = OFF_DIAG + (unsigned)(dest * 16 + tile) * 262144u + wid * 32768;
#pragma unroll
            for (int a = 0; a < 2; ++a)
#pragma unroll
                for (int m = 0; m < 4; ++m)
#pragma unroll
                    for (int b = 0; b < 2; ++b)
#pragma unroll
                        for (int n = 0; n < 2; ++n) {
                            u32x4 o = __builtin_bit_cast(u32x4, acc[a][b][m][n]); o.x |= nanm;
                            ST128(o, rs_me, lz * 16 + (b * 2 + n) * 1024, so + (a * 4 + m) * 4096, AUX_AGENT);
                        }
            WAIT_V(0);
            if (lz == 0) __builtin_amdgcn_raw_buffer_store_b32(call, rs_me, 0, F_DG + ((dest * 16 + tile) * 8 + wid) * 4, AUX_AGENT);
        }
        STAMP_AT(4);
        if (t >= nt) break;
#pragma unroll
        for (int a = 0; a < 2; ++a)
#pragma unroll
            for (int b = 0; b < 2; ++b)
#pragma unroll
                for (int m = 0; m < 4; ++m)
#pragma unroll
                    for (int n = 0; n < 2; ++n) acc[a][b][m][n] = (f32x4){0.f, 0.f, 0.f, 0.f};
        tile += 8;
    }
    if (mode != 2) {
        int lz = lane; asm volatile("" : "+v"(lz));
        if (mode == 0) {
            {
                const unsigned long long t0 = __builtin_amdgcn_s_memrealtime();
                while (!bad && (unsigned)rfl((int)__builtin_amdgcn_raw_buffer_load_b32(rs_me, 0, F_DG + ((rank * 16 + tile) * 8 + wid) * 4, AUX_AGENT)) < call)
                    if (give_up(rs_me, call, t0)) bad = true;
            }
            const unsigned nanm = bad ? 0x7fc07fc0u : 0u;
            const rsrc_t rs_out = mkrs(p.out, OUT_BYTES);
            const unsigned vo = (unsigned)(lz & 15) * (GN * 2) + (unsigned)(lz >> 4) * 16;
            const unsigned so = (unsigned)((tile >> 3) * 256 + wr * 64) * (GN * 2) + (unsigned)(rank * 2048 + (tile & 7) * 256 + wc * 64) * 2;
            const unsigned hs = OFF_DIAG + (unsigned)(rank * 16 + tile) * 262144u + wid * 32768;
#pragma unroll
            for (int a = 0; a < 2; ++a) {
                f32x4 h[4][2][2];
#pragma unroll
                for (int m = 0; m < 4; ++m)
#pragma unroll
                    for (int b = 0; b < 2; ++b)
#pragma unroll
                        for (int n = 0; n < 2; ++n)
                            h[m][b][n] = __builtin_bit_cast(f32x4, __builtin_amdgcn_raw_buffer_load_b128(rs_me, lz * 16 + (b * 2 + n) * 1024, hs + (a * 4 + m) * 4096, AUX_AGENT));
#pragma unroll
                for (int m = 0; m < 4; ++m)
#pragma unroll
                    for (int b = 0; b < 2; ++b) {
                        const f32x4 v0 = acc[a][b][m][0] + h[m][b][0], v1 = acc[a][b][m][1] + h[m][b][1];
                        u32x4 o; o.x = cvt_pk_bf16(silu(v0[0]), silu(v0[1])) | nanm; o.y = cvt_pk_bf16(silu(v0[2]), silu(v0[3]));
                        o.z = cvt_pk_bf16(silu(v1[0]), silu(v1[1])); o.w = cvt_pk_bf16(silu(v1[2]), silu(v1[3]));
                        ST128(o, rs_out, vo + b * 64, so + (unsigned)(a * 128 + m * 16) * (GN * 2), AUX_AGENT);
                    }
                asm volatile("" ::: "memory");
            }
        } else {
            const unsigned nanm = bad ? 0x7fc07fc0u : 0u;
            const unsigned so = yexp_off + (unsigned)(dest * 16 + tile) * 131072u + wid * 16384;
#pragma unroll
            for (int a = 0; a < 2; ++a)
#pragma unroll
                for (int m = 0; m < 4; ++m)
#pragma unroll
                    for (int b = 0; b < 2; ++b) {
                        const f32x4 v0 = acc[a][b][m][0], v1 = acc[a][b][m][1];
                        u32x4 o; o.x = cvt_pk_bf16(v0[0], v0[1]) | nanm; o.y = cvt_pk_bf16(v0[2], v0[3]);
                        o.z = cvt_pk_bf16(v1[0], v1[1]); o.w = cvt_pk_bf16(v1[2], v1[3]);
                        ST128(o, rs_me, lz * 16 + b * 1024, so + (a * 4 + m) * 2048, AUX_SYS);
                    }
            WAIT_V(0);
            if (lz == 0) __builtin_amdgcn_raw_buffer_store_b32(call, mkrs(p.ws[(dest - rank - 1) & 7], WS_NEED), 0, F_Y + ((rank * 16 + tile) * 8 + wid) * 4, AUX_SYS);
        }
        STAMP_AT(4);
    }
    WAIT_V(0);
}

}

extern "C" void kernel_launch(int rank, int nranks, ncclComm_t comm,
                              hipStream_t stream,
                              void* const* d_in, const int* in_sizes, int n_in,
                              void* d_out, int out_size,
                              const int* in_dtypes, int out_dtype,
                              void* const* d_out_all,
                              void* const* d_ws_all, size_t ws_bytes) {
    (void)comm; (void)in_dtypes; (void)out_dtype; (void)d_out_all;
    if (nranks != NR || (unsigned)rank >= (unsigned)NR || n_in < 2 || ws_bytes < (size_t)WS_NEED ||
        in_sizes[0] != GM * KS || in_sizes[1] != KS * GN || out_size != (GM / NR) * GN) {
        fprintf(stderr, "kernel_launch: unexpected problem shape\n");
        return;
    }
    Params p;
    p.x = d_in[0]; p.w = d_in[1]; p.out = d_out; p.me = d_ws_all[rank]; p.rank = rank; p.pad = 0;
    for (int i = 0; i < NR; ++i) p.ws[i] = d_ws_all[(rank + 1 + i) & 7];
    hipLaunchKernelGGL(fused_gemm_rs, dim3(256), dim3(512), 0, stream, p);
    HIP_CHECK(hipGetLastError());
}
```

```cpp
#include <hip/hip_runtime.h>
#include <rccl/rccl.h>
#include <cstdio>

#define HIP_CHECK(expr) do { hipError_t _e = (expr); if (_e != hipSuccess) { \
    fprintf(stderr, "HIP error %s at %s:%d\n", hipGetErrorString(_e), __FILE__, __LINE__); \
    return; } } while (0)

namespace {

constexpr int NR = 8;
constexpr int GM = 4096, GK = 4096, GN = 16384;
constexpr int KS = GK / NR;
constexpr unsigned MiB = 1u << 20;
constexpr unsigned UNIT = 32768;
constexpr unsigned X_BYTES = GM * KS * 2, W_BYTES = KS * GN * 2, OUT_BYTES = (GM / NR) * GN * 2;

constexpr unsigned EXP_SZ = 20 * MiB;
constexpr unsigned EXP_W = 4 * MiB;
constexpr unsigned OFF_EXP = 2 * MiB;
constexpr unsigned YEXP_SZ = 16 * MiB;
constexpr unsigned OFF_YEXP = OFF_EXP + 2 * EXP_SZ;
constexpr unsigned STG_SZ = 6 * MiB;
constexpr unsigned OFF_STG = OFF_YEXP + 2 * YEXP_SZ;
constexpr unsigned OFF_DIAG = OFF_STG + NR * STG_SZ;
constexpr unsigned OFF_PUSH = OFF_DIAG + 32 * MiB;
constexpr unsigned WS_NEED = OFF_PUSH + 4 * MiB;
constexpr unsigned F_CNT = 0;
constexpr unsigned F_TMO = 32768;
constexpr unsigned F_PUB = 36864;
constexpr unsigned F_Y = 49152;
constexpr unsigned F_DG = 57344;
constexpr unsigned F_Q = 33792;
constexpr unsigned F_A = 65536;
constexpr unsigned F_B = 131072;
constexpr unsigned F_STAMP = 524288;

constexpr unsigned long long DEADLINE = 100000000ull;

#define PSTEP 1
#define PDELAY 64
#define PURG 56
#define LAG_O 2
#define LAG_W 6

constexpr int AUX_AGENT = 16, AUX_SYS = 17;

typedef short bf16x8 __attribute__((ext_vector_type(8)));
typedef short s16x4 __attribute__((ext_vector_type(4)));
typedef float f32x4 __attribute__((ext_vector_type(4)));
typedef unsigned u32x4 __attribute__((ext_vector_type(4)));
typedef unsigned u32x2 __attribute__((ext_vector_type(2)));
#define LAS __attribute__((address_space(3)))
typedef __amdgpu_buffer_rsrc_t rsrc_t;
typedef __attribute__((address_space(1))) unsigned gu32;

struct Params {
    const void* x; const void* w; void* out; void* me;
    int rank, pad;
    void* ws[NR];
};

__device__ __forceinline__ rsrc_t mkrs(const void* p, unsigned bytes) {
    return __builtin_amdgcn_make_buffer_rsrc((void*)p, 0, bytes, 0x00020000);
}
__device__ __forceinline__ unsigned cvt_pk_bf16(float lo, float hi) {
    typedef float f32x2 __attribute__((ext_vector_type(2)));
    typedef __bf16 bf16x2 __attribute__((ext_vector_type(2)));
    return __builtin_bit_cast(unsigned, __builtin_convertvector((f32x2){lo, hi}, bf16x2));
}
__device__ __forceinline__ float silu(float y) {
    const float c = __builtin_amdgcn_fmed3f(y, -60.0f, 60.0f);
    const float e = __builtin_amdgcn_exp2f(c * -1.44269504088896341f);
    return y * __builtin_amdgcn_rcpf(1.0f + e);
}
__device__ __forceinline__ int rfl(int v) { return __builtin_amdgcn_readfirstlane(v); }
__device__ __forceinline__ int blk_at(int me, int l, int pos) { int v = l + 1 + pos; if (v >= 7) v -= 7; return (me + 1 + v) & 7; }
#define ST128(v, rs, voff, soff, aux) __builtin_amdgcn_raw_buffer_store_b128((v), (rs), (unsigned)(voff) + (unsigned)(soff), 0, (aux))
#define WAIT_V(n) asm volatile("s_waitcnt vmcnt(" #n ")" ::: "memory")
#define WAIT_L(n) asm volatile("s_waitcnt lgkmcnt(" #n ")" ::: "memory")
#define BAR __builtin_amdgcn_s_barrier()
#define SCHED __builtin_amdgcn_sched_barrier(0)

#define STAMP_AT(slot) do { } while (0)
#define STAMP_T(t) do { } while (0)

__device__ __forceinline__ bool give_up(rsrc_t rs_me, unsigned call, unsigned long long t0) {
    const unsigned tm = __builtin_amdgcn_raw_buffer_load_b32(rs_me, 0, F_TMO, AUX_AGENT);
    if ((unsigned)rfl((int)tm) == call || __builtin_amdgcn_s_memrealtime() - t0 > DEADLINE) {
        __builtin_amdgcn_raw_buffer_store_b32(call, rs_me, 0, F_TMO, AUX_AGENT);
        return true;
    }
    __builtin_amdgcn_s_sleep(2);
    asm volatile("" ::: "memory");
    return false;
}

__global__ __launch_bounds__(512, 2) void fused_gemm_rs(const Params p) {
    __shared__ __attribute__((aligned(16))) unsigned char smem[131072];
    LAS unsigned char* lds = (LAS unsigned char*)smem;
    const int tid = threadIdx.x, lane = tid & 63, wid = rfl(tid >> 6);
    const int bid = blockIdx.x, xcd = bid & 7, slot = bid >> 3;
    const int rank = p.rank;
    void* const link_ws = p.ws[slot < 7 ? slot : 0];
    asm volatile("" :: "s"(p.me), "s"(p.x), "s"(p.w), "s"(rank), "s"(link_ws));
    const rsrc_t rs_me = mkrs(p.me, WS_NEED);

    STAMP_AT(0);
    const unsigned vcall = __builtin_amdgcn_raw_buffer_load_b32(rs_me, 0, F_CNT + bid * 128, AUX_AGENT);
    const rsrc_t rs_x = mkrs(p.x, X_BYTES);
    const unsigned vx = (unsigned)(lane & 15) * (KS * 2) + (unsigned)(lane >> 4) * 16;

    if (slot < 7) {
        const int link = slot;
        const int src = (rank + 1 + link) & 7;
        const int prt = xcd < 6 ? 2 * blk_at(src, 6 - link, xcd) : 2 * src + (xcd - 6);
        u32x4 pv[4];
#pragma unroll
        for (int i = 0; i < 4; ++i)
            pv[i] = __builtin_amdgcn_raw_buffer_load_b128(rs_x, vx, (unsigned)(prt * 256 + ((wid >> 1) + 4 * i) * 16) * (KS * 2) + (wid & 1) * 64, 0);
        SCHED;
        const unsigned call = (unsigned)rfl((int)vcall) + 1u, par = call & 1u;
        const unsigned exp_off = OFF_EXP + par * EXP_SZ, yexp_off = OFF_YEXP + par * YEXP_SZ;
        STAMP_AT(1);
        const rsrc_t rs_src = mkrs(link_ws, WS_NEED);
        const rsrc_t rs_out = mkrs(p.out, OUT_BYTES);
        constexpr unsigned NIN = 168 * 8, NTOT = NIN + 16 * 64, NDRAW = NTOT + 64;
        const unsigned vl16 = (unsigned)lane * 16u, v_fy = (unsigned)(lane * 4 + src * 512);
        const unsigned pd = OFF_PUSH + (unsigned)((par * 8 + rank) * 8 + xcd) * UNIT + wid * 1024;
        ST128(pv[0], rs_src, vl16, pd, AUX_SYS);
        if (PSTEP >= 2) ST128(pv[1], rs_src, vl16, pd + 8192, AUX_SYS);
        if (PSTEP >= 4) { ST128(pv[2], rs_src, vl16, pd + 16384, AUX_SYS); ST128(pv[3], rs_src, vl16, pd + 24576, AUX_SYS); }
        SCHED;
        BAR;
        if (tid == 0) __builtin_amdgcn_raw_buffer_store_b32(call, rs_me, 0, F_CNT + bid * 128, AUX_AGENT);
        unsigned vz = 0; asm volatile("" : "+v"(vz));
        gu32* const qctr = (gu32*)((char*)p.me + F_Q + link * 128) + vz;
        const unsigned qbase = (call - 1u) * NDRAW;
        unsigned vtick = 0;
        if (lane == 0) vtick = __hip_atomic_fetch_add(qctr, 1u, __ATOMIC_RELAXED, __HIP_MEMORY_SCOPE_AGENT);
        int code[3];
#pragma unroll
        for (int c = 0; c < 3; ++c) {
            int pos = lane + 64 * c, cd = 0; bool found = false;
            for (int s = 0; s < 8 + (LAG_W > LAG_O ? LAG_W : LAG_O); ++s) {
                if (s < 8) {
                    if (!found && pos < 8) { cd = s | 8 | (pos << 4); found = true; } pos -= 8;
                    if (s > 0) { if (!found && pos < 6) { cd = s | ((2 * blk_at(rank, link, pos)) << 4); found = true; } pos -= 6; }
                }
                if (s >= LAG_O && s - LAG_O < 8) {
                    if (!found && pos < 6) { cd = (s - LAG_O) | ((2 * blk_at(rank, link, pos) + 1) << 4); found = true; } pos -= 6;
                }
                if (s > LAG_W && s - LAG_W < 8) { if (!found && pos < 2) { cd = (s - LAG_W) | ((2 * rank + pos) << 4); found = true; } pos -= 2; }
            }
            code[c] = found ? cd : 8;
        }
        int pub_ready = 0; bool bad = false;
        unsigned long long yav0 = 0, yav1 = 0;
#define DRAIN() do { WAIT_V(0); SCHED; __builtin_amdgcn_s_waitcnt(0x0F70); SCHED; } while (0)
#define DRAW() do { if (lane == 0) vtick = __hip_atomic_fetch_add(qctr, 1u, __ATOMIC_RELAXED, __HIP_MEMORY_SCOPE_AGENT); } while (0)
#define TICKET() ((unsigned)rfl((int)vtick) - qbase)
#define RLOAD(off) __builtin_amdgcn_raw_buffer_load_b128(rs_src, vl16, (off), AUX_SYS)
#define LLOAD(off) __builtin_amdgcn_raw_buffer_load_b128(rs_me, vl16, (off), AUX_AGENT)
        auto in_decode = [&](unsigned t, unsigned& so, unsigned& dofs, unsigned& fl) {
            const int u = t >> 3, w = t & 7;
            if (u >= pub_ready) {
                const unsigned long long t0 = __builtin_amdgcn_s_memrealtime();
                for (;;) {
                    unsigned f[3];
#pragma unroll
                    for (int c = 0; c < 3; ++c)
                        f[c] = __builtin_amdgcn_raw_buffer_load_b32(rs_me, ((code[c] & 7) * 24 + ((code[c] & 8) ? 0 : 8) + (code[c] >> 4)) * 4, F_PUB + src * 1024, AUX_SYS);
                    const unsigned long long n0 = ~__ballot(f[0] >= call), n1 = ~__ballot(f[1] >= call), n2 = ~__ballot(f[2] >= call);
                    pub_ready = n0 ? __builtin_ctzll(n0) : n1 ? 64 + __builtin_ctzll(n1) : n2 ? 128 + __builtin_ctzll(n2) : 192;
                    if (u < pub_ready) break;
                    if (give_up(rs_me, call, t0)) { bad = true; break; }
                }
            }
            const int c0 = __builtin_amdgcn_readlane(code[0], u & 63), c1 = __builtin_amdgcn_readlane(code[1], u & 63), c2 = __builtin_amdgcn_readlane(code[2], u & 63);
            const int cd = u < 64 ? c0 : u < 128 ? c1 : c2, kc = cd & 7, idx = cd >> 4;
            if (cd & 8) {
                so = exp_off + EXP_W + (unsigned)((rank * 8 + kc) * 8 + idx) * UNIT;
                dofs = OFF_STG + src * STG_SZ + EXP_W + (unsigned)(kc * 8 + idx) * UNIT;
                fl = F_B + ((w * 64 + rank * 8 + idx) * 64 + src * 8 + kc) * 4;
            } else {
                so = exp_off + (unsigned)(kc * 16 + idx) * UNIT;
                dofs = OFF_STG + src * STG_SZ + (unsigned)(kc * 16 + idx) * UNIT;
                fl = F_A + ((w * 16 + idx) * 64 + src * 8 + kc) * 4;
            }
            so += w * 1024; dofs += w * 1024;
            STAMP_T(t);
        };
        auto y_decode = [&](unsigned t, unsigned& ys, unsigned& ds, unsigned& os) {
            const int q = t - NIN, tl = q >> 6, g = (q >> 3) & 7, am = q & 7;
            const int bit = (tl & 7) * 8 + g;
            if (!((((tl & 8) ? yav1 : yav0) >> bit) & 1ull) && !bad) {
                const unsigned long long t0 = __builtin_amdgcn_s_memrealtime();
                for (;;) {
                    const unsigned y0 = __builtin_amdgcn_raw_buffer_load_b32(rs_me, v_fy + F_Y, 0, AUX_SYS);
                    const unsigned y1 = __builtin_amdgcn_raw_buffer_load_b32(rs_me, v_fy + F_Y + 256, 0, AUX_SYS);
                    const unsigned d0 = __builtin_amdgcn_raw_buffer_load_b32(rs_me, v_fy + F_DG, 0, AUX_AGENT);
                    const unsigned d1 = __builtin_amdgcn_raw_buffer_load_b32(rs_me, v_fy + F_DG + 256, 0, AUX_AGENT);
                    yav0 = __ballot((y0 < d0 ? y0 : d0) >= call); yav1 = __ballot((y1 < d1 ? y1 : d1) >= call);
                    if ((((tl & 8) ? yav1 : yav0) >> bit) & 1ull) break;
                    if (give_up(rs_me, call, t0)) { bad = true; break; }
                }
            }
            ys = yexp_off + (unsigned)(rank * 16 + tl) * 131072u + g * 16384 + am * 2048;
            ds = OFF_DIAG + (unsigned)(src * 16 + tl) * 262144u + g * 32768 + am * 4096;
            os = (unsigned)((tl >> 3) * 256 + (am >> 2) * 128 + (g >> 2) * 64 + (am & 3) * 16) * (GN * 2)
               + (unsigned)(src * 2048 + (tl & 7) * 256 + (g & 3) * 64) * 2;
            STAMP_T(t);
        };
        if (PSTEP == 1) {
            DRAIN(); ST128(pv[1], rs_src, vl16, pd + 8192, AUX_SYS);
            DRAIN(); ST128(pv[2], rs_src, vl16, pd + 16384, AUX_SYS);
            DRAIN(); ST128(pv[3], rs_src, vl16, pd + 24576, AUX_SYS);
        } else if (PSTEP == 2) {
            DRAIN(); ST128(pv[2], rs_src, vl16, pd + 16384, AUX_SYS); ST128(pv[3], rs_src, vl16, pd + 24576, AUX_SYS);
        } else if (PSTEP == 3) {
            DRAIN(); ST128(pv[2], rs_src, vl16, pd + 16384, AUX_SYS);
            DRAIN(); ST128(pv[3], rs_src, vl16, pd + 24576, AUX_SYS);
        }
        unsigned t = TICKET();
        SCHED;
        unsigned pflag = F_A + ((wid * 16 + prt) * 64 + rank * 8) * 4;
#define PUSH_FLAG() do { if (pflag) { if (lane == 0) __builtin_amdgcn_raw_buffer_store_b32(call, rs_src, 0, pflag, AUX_SYS); pflag = 0; } } while (0)
        if (t < NIN) {
            unsigned so, dofs, fl, prev = 0;
            in_decode(t, so, dofs, fl);
            STAMP_AT(5);
            if (!bad) {
                u32x4 A = RLOAD(so), B;
                for (;;) {
                    DRAW();
                    DRAIN(); B = RLOAD(so + 8192); SCHED; ST128(A, rs_me, vl16, dofs, AUX_AGENT);
                    if (prev && lane == 0) __builtin_amdgcn_raw_buffer_store_b32(call, rs_me, 0, prev, AUX_AGENT);
                    PUSH_FLAG();
                    t = TICKET();
                    DRAIN(); A = RLOAD(so + 16384); SCHED; ST128(B, rs_me, vl16, dofs + 8192, AUX_AGENT);
                    DRAIN(); B = RLOAD(so + 24576); SCHED; ST128(A, rs_me, vl16, dofs + 16384, AUX_AGENT);
                    const unsigned dlast = dofs + 24576; prev = fl;
                    bool more = t < NIN;
                    if (more) { in_decode(t, so, dofs, fl); more = !bad; }
                    DRAIN();
                    if (more) A = RLOAD(so);
                    SCHED; ST128(B, rs_me, vl16, dlast, AUX_AGENT);
                    if (!more) break;
                }
                DRAIN();
                if (lane == 0) __builtin_amdgcn_raw_buffer_store_b32(call, rs_me, 0, prev, AUX_AGENT);
            }
            while (t < NIN) { DRAW(); t = TICKET(); }
        }
        if (pflag) { DRAIN(); PUSH_FLAG(); }
        STAMP_AT(2);
        if (t < NTOT) {
            const unsigned v_out = (unsigned)(lane & 15) * (GN * 2) + (unsigned)(lane >> 4) * 16;
            auto finish = [&](const u32x4 y, const u32x4 d0, const u32x4 d1, unsigned os) {
                u32x4 o;
                o.x = cvt_pk_bf16(silu(__uint_as_float(y.x << 16) + __uint_as_float(d0.x)), silu(__uint_as_float(y.x & 0xffff0000u) + __uint_as_float(d0.y))) | (bad ? 0x7fc07fc0u : 0u);
                o.y = cvt_pk_bf16(silu(__uint_as_float(y.y << 16) + __uint_as_float(d0.z)), silu(__uint_as_float(y.y & 0xffff0000u) + __uint_as_float(d0.w)));
                o.z = cvt_pk_bf16(silu(__uint_as_float(y.z << 16) + __uint_as_float(d1.x)), silu(__uint_as_float(y.z & 0xffff0000u) + __uint_as_float(d1.y)));
                o.w = cvt_pk_bf16(silu(__uint_as_float(y.w << 16) + __uint_as_float(d1.z)), silu(__uint_as_float(y.w & 0xffff0000u) + __uint_as_float(d1.w)));
                ST128(o, rs_out, v_out, os, AUX_AGENT);
            };
            unsigned ys, ds, os;
            y_decode(t, ys, ds, os);
            u32x4 A = RLOAD(ys), a0 = LLOAD(ds), a1 = LLOAD(ds + 1024);
            DRAW();
            for (;;) {
                DRAIN();
                t = TICKET();
                const u32x4 B = RLOAD(ys + 1024), b0 = LLOAD(ds + 2048), b1 = LLOAD(ds + 3072);
                SCHED;
                finish(A, a0, a1, os);
                const unsigned ol = os + 64;
                const bool more = t < NTOT;
                if (more) y_decode(t, ys, ds, os);
                DRAIN();
                if (more) { A = RLOAD(ys); a0 = LLOAD(ds); a1 = LLOAD(ds + 1024); DRAW(); }
                SCHED;
                finish(B, b0, b1, ol);
                if (!more) break;
            }
        }
        STAMP_AT(3);
        return;
    }

    unsigned call = 0, exp_off = 0;
    {
        const rsrc_t rs_w = mkrs(p.w, W_BYTES);
        if (PDELAY && bid >= 56 + PURG) __builtin_amdgcn_s_sleep(PDELAY);
        for (int r = 0; r < 4; ++r) {
            const int u = r * 200 + bid - 56;
            if (u >= 640) break;
            const int kc = u / 80, j = u - kc * 80;
            const bool isw = j < 64;
            const int dl = isw ? j >> 3 : (wid + 7) & 7, dest = (rank + 1 + dl) & 7;
            const rsrc_t rs_flag = mkrs(p.ws[dl], WS_NEED);
            const int ct = j & 7, rt = j - 64;
            u32x4 v[4];
            if (isw) {
                const unsigned so = (unsigned)(kc * 64) * (GN * 2) + (unsigned)(dest * 2048 + ct * 256) * 2;
                const unsigned vo = (unsigned)(tid >> 5) * (GN * 2) + (unsigned)(tid & 31) * 16;
#pragma unroll
                for (int i = 0; i < 4; ++i) v[i] = __builtin_amdgcn_raw_buffer_load_b128(rs_w, vo, so + i * 16 * (GN * 2), 0);
            } else {
#pragma unroll
                for (int i = 0; i < 4; ++i) {
                    const int F = wid * 4 + i;
                    v[i] = __builtin_amdgcn_raw_buffer_load_b128(rs_x, vx, (unsigned)(rt * 256 + (F >> 1) * 16) * (KS * 2) + kc * 128 + (F & 1) * 64, 0);
                }
            }
            if (r == 0) {
                SCHED;
                call = (unsigned)rfl((int)vcall) + 1u;
                exp_off = OFF_EXP + (call & 1u) * EXP_SZ;
            }
            if (isw) {
                const unsigned dst = exp_off + EXP_W + (unsigned)((dest * 8 + kc) * 8 + ct) * UNIT;
#pragma unroll
                for (int i = 0; i < 4; ++i) *(LAS u32x4*)(lds + (i * 16 + (tid >> 5)) * 576 + (tid & 31) * 16) = v[i];
                __syncthreads();
#pragma unroll
                for (int i = 0; i < 4; ++i) {
                    const int F = i * 8 + wid, bj = F >> 4, wcn = (F >> 1) & 7, kk = F & 1;
                    const int col = (wcn >> 1) * 64 + bj * 32 + (wcn & 1) * 4 + (lane & 3) * 8;
                    const int row = kk * 32 + (lane >> 4) * 8 + ((lane & 15) >> 2);
                    const s16x4 t0 = __builtin_amdgcn_ds_read_tr16_b64_v4i16((LAS s16x4*)(lds + row * 576 + col * 2));
                    const s16x4 t1 = __builtin_amdgcn_ds_read_tr16_b64_v4i16((LAS s16x4*)(lds + (row + 4) * 576 + col * 2));
                    const u32x2 a = __builtin_bit_cast(u32x2, t0), b = __builtin_bit_cast(u32x2, t1);
                    v[i] = (u32x4){a.x, a.y, b.x, b.y};
                }
#pragma unroll
                for (int i = 0; i < 4; ++i) ST128(v[i], rs_me, lane * 16, dst + (i * 8 + wid) * 1024, AUX_SYS);
            } else {
                const unsigned dst = exp_off + (unsigned)(kc * 16 + rt) * UNIT;
#pragma unroll
                for (int i = 0; i < 4; ++i) ST128(v[i], rs_me, lane * 16, dst + (wid * 4 + i) * 1024, AUX_SYS);
            }
            WAIT_V(0);
            __syncthreads();
            if (wid == 0) {
                if (isw) {
                    if (dest != rank && lane == 0)
                        __builtin_amdgcn_raw_buffer_store_b32(call, rs_flag, 0, F_PUB + (rank * 256 + kc * 24 + ct) * 4, AUX_SYS);
                    if (lane < 8)
                        __builtin_amdgcn_raw_buffer_store_b32(call, rs_me, lane * (64 * 64 * 4), F_B + ((dest * 8 + ct) * 64 + rank * 8 + kc) * 4, AUX_AGENT);
                } else {
                    if (lane < 8)
                        __builtin_amdgcn_raw_buffer_store_b32(call, rs_me, lane * (16 * 64 * 4), F_A + (rt * 64 + rank * 8 + kc) * 4, AUX_AGENT);
                }
                if (r == 0 && lane == 0) __builtin_amdgcn_raw_buffer_store_b32(call, rs_me, 0, F_CNT + bid * 128, AUX_AGENT);
            } else if (!isw && lane == 0) {
                if ((rt >> 1) != rank)
                    __builtin_amdgcn_raw_buffer_store_b32(call, rs_flag, 0, F_PUB + (rank * 256 + kc * 24 + 8 + rt) * 4, AUX_SYS);
            }
        }
    }
    const unsigned par = call & 1u;
    const unsigned yexp_off = OFF_YEXP + par * YEXP_SZ;
    STAMP_AT(1);

    const int wr = wid >> 2, wc = wid & 3;
    int nt, seglen, mode, tile0;
    unsigned v_a, v_b, v_fa, v_fb;
    int dest;
    if (slot < 23 || slot >= 30) {
        const bool helper = slot >= 30;
        const int i = slot - 7;
        const int j = helper ? rank : ((xcd >> 1) & 1) * 4 + (i & 3), rt2 = helper ? slot - 30 : xcd & 1;
        const int rt = 2 * j + rt2, ct = helper ? xcd : (xcd >> 2) * 4 + (i >> 2);
        const bool own = (j == rank);
        const int nsrc = own ? 7 : 6;
        nt = own ? 32 : 56; seglen = nt; mode = helper ? 2 : own ? 0 : 1; dest = j; tile0 = rt2 * 8 + ct;
        const int st = own ? 2 * (lane & 31) + (helper ? 0 : 1) : lane;
        int src, kc, si = 0;
        if (st < 8 || st >= 8 + 8 * nsrc) { src = rank; kc = st & 7; }
        else {
            const int q = st - 8; kc = q / nsrc; si = q - kc * nsrc;
            int l = ((j - rank - 1) & 7) - 1 - si; if (l < 0) l += 7;
            src = (rank + 1 + l) & 7;
        }
        v_a = (src == rank ? exp_off : OFF_STG + src * STG_SZ) + (unsigned)(kc * 16 + rt) * UNIT;
        if (src != rank && kc == 0 && (own || rt2 == 0))
            v_a = OFF_PUSH + (unsigned)((par * 8 + src) * 8 + (own ? 6 + rt2 : si)) * UNIT;
        v_b = (src == rank ? exp_off + (unsigned)(rank * 64) * UNIT : OFF_STG + src * STG_SZ) + EXP_W + (unsigned)(kc * 8 + ct) * UNIT;
        v_fa = F_A + ((wid * 16 + rt) * 64 + src * 8 + kc) * 4;
        v_fb = F_B + ((wid * 64 + rank * 8 + ct) * 64 + src * 8 + kc) * 4;
    } else {
        const int slab = (rank + 1 + (slot - 23)) & 7, ct = xcd;
        nt = 16; seglen = 8; mode = 2; dest = slab; tile0 = ct;
        const int kc = lane & 7, rt = 2 * rank + ((lane >> 3) & 1);
        v_a = exp_off + (unsigned)(kc * 16 + rt) * UNIT;
        v_b = exp_off + EXP_W + (unsigned)((slab * 8 + kc) * 8 + ct) * UNIT;
        v_fa = F_A + ((wid * 16 + rt) * 64 + rank * 8 + kc) * 4;
        v_fb = F_B + ((wid * 64 + slab * 8 + ct) * 64 + rank * 8 + kc) * 4;
    }
    int ready = 0; bool bad = false;
    auto ensure = [&](int need) {
        if (need > nt) need = nt;
        if (ready >= need) return;
        const unsigned long long t0 = __builtin_amdgcn_s_memrealtime();
        __builtin_amdgcn_s_waitcnt(0xC07F);
        for (;;) {
            const unsigned fa = __builtin_amdgcn_raw_buffer_load_b32(rs_me, v_fa, 0, AUX_SYS);
            const unsigned fb = __builtin_amdgcn_raw_buffer_load_b32(rs_me, v_fb, 0, AUX_AGENT);
            const unsigned long long nm = ~__ballot((fa < fb ? fa : fb) >= call);
            ready = nm ? __builtin_ctzll(nm) : 64;
            if (ready >= need) break;
            if (give_up(rs_me, call, t0)) { bad = true; ready = 64; break; }
            __builtin_amdgcn_s_sleep(4);
        }
    };
    const unsigned vst = (unsigned)tid * 16u;
    const int ldsw = wid * 1024;
    const int aoff = wr * 8192 + lane * 16;
    int boff = 65536 + wc * 4096 + lane * 16; asm volatile("" : "+v"(boff));
#define SA(b, h) (((b) * 2 + (h)) * 16384)
#define SB(b, h) ((4 + (b) * 2 + (h)) * 16384)
#define STAGE(bufoff, goff) do { _Pragma("unroll") for (int _i = 0; _i < 2; ++_i) \
        __builtin_amdgcn_raw_ptr_buffer_load_lds(rs_me, (LAS void*)(lds + (bufoff) + ldsw + _i * 8192), 16, vst, (goff) + _i * 8192, 0, AUX_AGENT); } while (0)
#define LDA(dst, b, h) do { _Pragma("unroll") for (int m = 0; m < 4; ++m) _Pragma("unroll") for (int k = 0; k < 2; ++k) \
        dst[m][k] = *(const LAS bf16x8*)(lds + SA(b, h) + aoff + m * 2048 + k * 1024); } while (0)
#define LDB(dst, b, h) do { _Pragma("unroll") for (int n = 0; n < 2; ++n) _Pragma("unroll") for (int k = 0; k < 2; ++k) \
        dst[n][k] = *(const LAS bf16x8*)(lds + boff + (SB(b, h) - 65536 + n * 2048 + k * 1024)); } while (0)
#define MMA(ai, bj, At, Bt) do { __builtin_amdgcn_s_setprio(1); _Pragma("unroll") for (int m = 0; m < 4; ++m) _Pragma("unroll") for (int n = 0; n < 2; ++n) \
        _Pragma("unroll") for (int k = 0; k < 2; ++k) acc[ai][bj][m][n] = __builtin_amdgcn_mfma_f32_16x16x32_bf16(Bt[n][k], At[m][k], acc[ai][bj][m][n], 0, 0, 0); \
        __builtin_amdgcn_s_setprio(0); } while (0)
#define AOFF(t) ((unsigned)__builtin_amdgcn_readlane((int)v_a, (t)))
#define BOFF(t) ((unsigned)__builtin_amdgcn_readlane((int)v_b, (t)))
    f32x4 acc[2][2][4][2];
#pragma unroll
    for (int a = 0; a < 2; ++a)
#pragma unroll
        for (int b = 0; b < 2; ++b)
#pragma unroll
            for (int m = 0; m < 4; ++m)
#pragma unroll
                for (int n = 0; n < 2; ++n) acc[a][b][m][n] = (f32x4){0.f, 0.f, 0.f, 0.f};
    bf16x8 At[4][2], B0[2][2], B1[2][2];
    ensure(2);
    {
        const unsigned a0 = AOFF(0), b0 = BOFF(0), a1 = AOFF(1), b1 = BOFF(1);
        STAGE(SB(0, 0), b0); STAGE(SA(0, 0), a0); STAGE(SB(0, 1), b0 + 16384); STAGE(SA(0, 1), a0 + 16384);
        if (wr == 1) BAR;
        WAIT_V(4); BAR;
        STAGE(SB(1, 0), b1); STAGE(SA(1, 0), a1); STAGE(SB(1, 1), b1 + 16384);
        WAIT_V(6); BAR;
    }
    STAMP_AT(2);
    int tile = tile0;
    for (int t = 0;;) {
        const int seg_end = t + seglen;
        for (; t < seg_end; t += 2) {
            const int t2 = (t + 2 < nt) ? t + 2 : 0;
            const unsigned a1 = AOFF(t + 1), a2 = AOFF(t2), b2 = BOFF(t2), a3 = AOFF(t2 + 1), b3 = BOFF(t2 + 1);
            LDB(B0, 0, 0); SCHED; LDA(At, 0, 0); STAGE(SA(1, 1), a1 + 16384);
            WAIT_L(8); BAR; WAIT_L(0); MMA(0, 0, At, B0); BAR; SCHED;
            ensure(t + 3);
            LDB(B1, 0, 1); STAGE(SB(0, 0), b2);
            BAR; WAIT_L(0); MMA(0, 1, At, B1); BAR;
            LDA(At, 0, 1); STAGE(SA(0, 0), a2);
            BAR; WAIT_L(0); MMA(1, 0, At, B0); BAR; SCHED;
            STAGE(SB(0, 1), b2 + 16384);
            WAIT_V(6); BAR; MMA(1, 1, At, B1); BAR;
            LDB(B0, 1, 0); SCHED; LDA(At, 1, 0); STAGE(SA(0, 1), a2 + 16384);
            WAIT_L(8); BAR; WAIT_L(0); MMA(0, 0, At, B0); BAR; SCHED;
            ensure(t + 4);
            LDB(B1, 1, 1); STAGE(SB(1, 0), b3);
            BAR; WAIT_L(0); MMA(0, 1, At, B1); BAR;
            LDA(At, 1, 1); STAGE(SA(1, 0), a3);
            BAR; WAIT_L(0); MMA(1, 0, At, B0); BAR; SCHED;
            STAGE(SB(1, 1), b3 + 16384);
            WAIT_V(6); BAR; MMA(1, 1, At, B1); BAR;
        }
        if (t >= nt && wr == 0) BAR;
        STAMP_AT(3);
        if (mode != 2) break;
        {
            int lz = lane; asm volatile("" : "+v"(lz));
            const unsigned nanm = bad ? 0x7fc00000u : 0u;
            const unsigned so = OFF_DIAG + (unsigned)(dest * 16 + tile) * 262144u + wid * 32768;
#pragma unroll
            for (int a = 0; a < 2; ++a)
#pragma unroll
                for (int m = 0; m < 4; ++m)
#pragma unroll
                    for (int b = 0; b < 2; ++b)
#pragma unroll
                        for (int n = 0; n < 2; ++n) {
                            u32x4 o = __builtin_bit_cast(u32x4, acc[a][b][m][n]); o.x |= nanm;
                            ST128(o, rs_me, lz * 16 + (b * 2 + n) * 1024, so + (a * 4 + m) * 4096, AUX_AGENT);
                        }
            WAIT_V(0);
            if (lz == 0) __builtin_amdgcn_raw_buffer_store_b32(call, rs_me, 0, F_DG + ((dest * 16 + tile) * 8 + wid) * 4, AUX_AGENT);
        }
        STAMP_AT(4);
        if (t >= nt) break;
#pragma unroll
        for (int a = 0; a < 2; ++a)
#pragma unroll
            for (int b = 0; b < 2; ++b)
#pragma unroll
                for (int m = 0; m < 4; ++m)
#pragma unroll
                    for (int n = 0; n < 2; ++n) acc[a][b][m][n] = (f32x4){0.f, 0.f, 0.f, 0.f};
        tile += 8;
    }
    if (mode != 2) {
        int lz = lane; asm volatile("" : "+v"(lz));
        if (mode == 0) {
            {
                const unsigned long long t0 = __builtin_amdgcn_s_memrealtime();
                while (!bad && (unsigned)rfl((int)__builtin_amdgcn_raw_buffer_load_b32(rs_me, 0, F_DG + ((rank * 16 + tile) * 8 + wid) * 4, AUX_AGENT)) < call)
                    if (give_up(rs_me, call, t0)) bad = true;
            }
            const unsigned nanm = bad ? 0x7fc07fc0u : 0u;
            const rsrc_t rs_out = mkrs(p.out, OUT_BYTES);
            const unsigned vo = (unsigned)(lz & 15) * (GN * 2) + (unsigned)(lz >> 4) * 16;
            const unsigned so = (unsigned)((tile >> 3) * 256 + wr * 64) * (GN * 2) + (unsigned)(rank * 2048 + (tile & 7) * 256 + wc * 64) * 2;
            const unsigned hs = OFF_DIAG + (unsigned)(rank * 16 + tile) * 262144u + wid * 32768;
#pragma unroll
            for (int a = 0; a < 2; ++a) {
                f32x4 h[4][2][2];
#pragma unroll
                for (int m = 0; m < 4; ++m)
#pragma unroll
                    for (int b = 0; b < 2; ++b)
#pragma unroll
                        for (int n = 0; n < 2; ++n)
                            h[m][b][n] = __builtin_bit_cast(f32x4, __builtin_amdgcn_raw_buffer_load_b128(rs_me, lz * 16 + (b * 2 + n) * 1024, hs + (a * 4 + m) * 4096, AUX_AGENT));
#pragma unroll
                for (int m = 0; m < 4; ++m)
#pragma unroll
                    for (int b = 0; b < 2; ++b) {
                        const f32x4 v0 = acc[a][b][m][0] + h[m][b][0], v1 = acc[a][b][m][1] + h[m][b][1];
                        u32x4 o; o.x = cvt_pk_bf16(silu(v0[0]), silu(v0[1])) | nanm; o.y = cvt_pk_bf16(silu(v0[2]), silu(v0[3]));
                        o.z = cvt_pk_bf16(silu(v1[0]), silu(v1[1])); o.w = cvt_pk_bf16(silu(v1[2]), silu(v1[3]));
                        ST128(o, rs_out, vo + b * 64, so + (unsigned)(a * 128 + m * 16) * (GN * 2), AUX_AGENT);
                    }
                asm volatile("" ::: "memory");
            }
        } else {
            const unsigned nanm = bad ? 0x7fc07fc0u : 0u;
            const unsigned so = yexp_off + (unsigned)(dest * 16 + tile) * 131072u + wid * 16384;
#pragma unroll
            for (int a = 0; a < 2; ++a)
#pragma unroll
                for (int m = 0; m < 4; ++m)
#pragma unroll
                    for (int b = 0; b < 2; ++b) {
                        const f32x4 v0 = acc[a][b][m][0], v1 = acc[a][b][m][1];
                        u32x4 o; o.x = cvt_pk_bf16(v0[0], v0[1]) | nanm; o.y = cvt_pk_bf16(v0[2], v0[3]);
                        o.z = cvt_pk_bf16(v1[0], v1[1]); o.w = cvt_pk_bf16(v1[2], v1[3]);
                        ST128(o, rs_me, lz * 16 + b * 1024, so + (a * 4 + m) * 2048, AUX_SYS);
                    }
            WAIT_V(0);
            if (lz == 0) __builtin_amdgcn_raw_buffer_store_b32(call, mkrs(p.ws[(dest - rank - 1) & 7], WS_NEED), 0, F_Y + ((rank * 16 + tile) * 8 + wid) * 4, AUX_SYS);
        }
        STAMP_AT(4);
    }
    WAIT_V(0);
}

}

extern "C" void kernel_launch(int rank, int nranks, ncclComm_t comm,
                              hipStream_t stream,
                              void* const* d_in, const int* in_sizes, int n_in,
                              void* d_out, int out_size,
                              const int* in_dtypes, int out_dtype,
                              void* const* d_out_all,
                              void* const* d_ws_all, size_t ws_bytes) {
    (void)comm; (void)in_dtypes; (void)out_dtype; (void)d_out_all;
    if (nranks != NR || (unsigned)rank >= (unsigned)NR || n_in < 2 || ws_bytes < (size_t)WS_NEED ||
        in_sizes[0] != GM * KS || in_sizes[1] != KS * GN || out_size != (GM / NR) * GN) {
        fprintf(stderr, "kernel_launch: unexpected problem shape\n");
        return;
    }
    Params p;
    p.x = d_in[0]; p.w = d_in[1]; p.out = d_out; p.me = d_ws_all[rank]; p.rank = rank; p.pad = 0;
    for (int i = 0; i < NR; ++i) p.ws[i] = d_ws_all[(rank + 1 + i) & 7];
    hipLaunchKernelGGL(fused_gemm_rs, dim3(256), dim3(512), 0, stream, p);
    HIP_CHECK(hipGetLastError());
}
```

```cpp
#include <hip/hip_runtime.h>
#include <rccl/rccl.h>
#include <cstdio>

#define HIP_CHECK(expr) do { hipError_t _e = (expr); if (_e != hipSuccess) { \
    fprintf(stderr, "HIP error %s at %s:%d\n", hipGetErrorString(_e), __FILE__, __LINE__); \
    return; } } while (0)

namespace {

constexpr int NR = 8;
constexpr int GM = 4096, GK = 4096, GN = 16384;
constexpr int KS = GK / NR;
constexpr unsigned MiB = 1u << 20;
constexpr unsigned UNIT = 32768;
constexpr unsigned X_BYTES = GM * KS * 2, W_BYTES = KS * GN * 2, OUT_BYTES = (GM / NR) * GN * 2;

constexpr unsigned EXP_SZ = 20 * MiB;
constexpr unsigned EXP_W = 4 * MiB;
#define EXP0 2
constexpr unsigned OFF_EXP = EXP0 * MiB;
constexpr unsigned YEXP_SZ = 16 * MiB;
constexpr unsigned OFF_YEXP = OFF_EXP + 2 * EXP_SZ;
constexpr unsigned STG_SZ = 6 * MiB;
constexpr unsigned OFF_STG = OFF_YEXP + 2 * YEXP_SZ;
constexpr unsigned OFF_DIAG = OFF_STG + NR * STG_SZ;
constexpr unsigned OFF_PUSH = OFF_DIAG + 32 * MiB;
constexpr unsigned WS_NEED = OFF_PUSH + 4 * MiB;
constexpr unsigned F_CNT = 0;
constexpr unsigned F_TMO = 32768;
constexpr unsigned F_PUB = 36864;
constexpr unsigned F_Y = 49152;
constexpr unsigned F_DG = 57344;
constexpr unsigned F_Q = 33792;
constexpr unsigned F_A = 65536;
constexpr unsigned F_B = 131072;
constexpr unsigned F_STAMP = 524288;

constexpr unsigned long long DEADLINE = 100000000ull;

#define PW 8
#define NPX 3
#define FR 4
#define NPUSH 5
#define LAG_O 2
#define LAG_W 6

static_assert(NPUSH >= 1 && NPUSH <= 6 && 56 * NPUSH <= 8 * 25 * (8 - NPX), "push pieces");
static_assert(FR * 200 >= 640, "every unit is published");
constexpr int AUX_AGENT = 16, AUX_SYS = 17;

typedef short bf16x8 __attribute__((ext_vector_type(8)));
typedef short s16x4 __attribute__((ext_vector_type(4)));
typedef float f32x4 __attribute__((ext_vector_type(4)));
typedef unsigned u32x4 __attribute__((ext_vector_type(4)));
typedef unsigned u32x2 __attribute__((ext_vector_type(2)));
#define LAS __attribute__((address_space(3)))
typedef __amdgpu_buffer_rsrc_t rsrc_t;
typedef __attribute__((address_space(1))) unsigned gu32;

struct Params {
    const void* x; const void* w; void* out; void* me;
    int rank, pad;
    void* ws[NR];
};

__device__ __forceinline__ rsrc_t mkrs(const void* p, unsigned bytes) {
    return __builtin_amdgcn_make_buffer_rsrc((void*)p, 0, bytes, 0x00020000);
}
__device__ __forceinline__ unsigned cvt_pk_bf16(float lo, float hi) {
    typedef float f32x2 __attribute__((ext_vector_type(2)));
    typedef __bf16 bf16x2 __attribute__((ext_vector_type(2)));
    return __builtin_bit_cast(unsigned, __builtin_convertvector((f32x2){lo, hi}, bf16x2));
}
__device__ __forceinline__ float silu(float y) {
    const float c = __builtin_amdgcn_fmed3f(y, -60.0f, 60.0f);
    const float e = __builtin_amdgcn_exp2f(c * -1.44269504088896341f);
    return y * __builtin_amdgcn_rcpf(1.0f + e);
}
__device__ __forceinline__ int rfl(int v) { return __builtin_amdgcn_readfirstlane(v); }
__device__ __forceinline__ int blk_at(int me, int l, int pos) { int v = l + 1 + pos; if (v >= 7) v -= 7; return (me + 1 + v) & 7; }
#define ST128(v, rs, voff, soff, aux) __builtin_amdgcn_raw_buffer_store_b128((v), (rs), (unsigned)(voff) + (unsigned)(soff), 0, (aux))
#define WAIT_V(n) asm volatile("s_waitcnt vmcnt(" #n ")" ::: "memory")
#define WAIT_L(n) asm volatile("s_waitcnt lgkmcnt(" #n ")" ::: "memory")
#define BAR __builtin_amdgcn_s_barrier()
#define SCHED __builtin_amdgcn_sched_barrier(0)

#define STAMP_AT(slot) do { } while (0)
#define STAMP_T(t) do { } while (0)

__device__ __forceinline__ bool give_up(rsrc_t rs_me, unsigned call, unsigned long long t0) {
    const unsigned tm = __builtin_amdgcn_raw_buffer_load_b32(rs_me, 0, F_TMO, AUX_AGENT);
    if ((unsigned)rfl((int)tm) == call || __builtin_amdgcn_s_memrealtime() - t0 > DEADLINE) {
        __builtin_amdgcn_raw_buffer_store_b32(call, rs_me, 0, F_TMO, AUX_AGENT);
        return true;
    }
    __builtin_amdgcn_s_sleep(2);
    asm volatile("" ::: "memory");
    return false;
}

__global__ __launch_bounds__(512, 2) void fused_gemm_rs(const Params p) {
    __shared__ __attribute__((aligned(16))) unsigned char smem[131072];
    LAS unsigned char* lds = (LAS unsigned char*)smem;
    const int tid = threadIdx.x, lane = tid & 63, wid = rfl(tid >> 6);
    const int bid = blockIdx.x, xcd = bid & 7, slot = bid >> 3;
    const int rank = p.rank;
    const int po = xcd < NPX ? (slot - 7) * NPX + xcd : 25 * NPX + (slot - 7) * (8 - NPX) + (xcd - NPX);
    constexpr int PWV = (56 * NPUSH + 25 * (8 - NPX) - 1) / (25 * (8 - NPX));
    const bool pusher = slot >= 7 && xcd >= NPX && wid < PWV && (po - 25 * NPX) * PWV + wid < 56 * NPUSH;
    const int pi = pusher ? (po - 25 * NPX) * PWV + wid : 0, pe = pi / 7;
    void* const push_ws = p.ws[pi - pe * 7];
    asm volatile("" :: "s"(p.me), "s"(p.x), "s"(p.w), "s"(rank), "s"(push_ws));
    const rsrc_t rs_me = mkrs(p.me, WS_NEED);

    STAMP_AT(0);
    const unsigned vcall = __builtin_amdgcn_raw_buffer_load_b32(rs_me, 0, F_CNT + bid * 128, AUX_AGENT);
    unsigned call = 0, exp_off = 0;

    if (slot < 7) {
        call = (unsigned)rfl((int)vcall) + 1u;
        exp_off = OFF_EXP + (call & 1u) * EXP_SZ;
        __syncthreads();
        if (tid == 0) __builtin_amdgcn_raw_buffer_store_b32(call, rs_me, 0, F_CNT + bid * 128, AUX_AGENT);
    } else {
        const rsrc_t rs_x = mkrs(p.x, X_BYTES), rs_w = mkrs(p.w, W_BYTES);
        const unsigned vx = (unsigned)(lane & 15) * (KS * 2) + (unsigned)(lane >> 4) * 16;
        const int pdest = (rank + 1 + pi - pe * 7) & 7, pw = pe / NPUSH, pu = pe - pw * NPUSH;
        const int prt = 2 * blk_at(pdest, (rank - pdest - 1) & 7, pu);
        const rsrc_t rs_push = mkrs(push_ws, WS_NEED);
        if (pusher) {
            u32x4 pv[4];
#pragma unroll
            for (int i = 0; i < 4; ++i)
                pv[i] = __builtin_amdgcn_raw_buffer_load_b128(rs_x, vx, (unsigned)(prt * 256 + ((pw >> 1) + 4 * i) * 16) * (KS * 2) + (pw & 1) * 64, 0);
            SCHED;
            const unsigned d = OFF_PUSH + (unsigned)((((rfl((int)vcall) + 1) & 1) * 8 + rank) * 8 + (prt >> 1)) * UNIT + pw * 1024;
#pragma unroll
            for (int i = 0; i < 4; ++i) ST128(pv[i], rs_push, lane * 16, d + i * 8192, AUX_SYS);
            SCHED;
        }
        for (int r = 0; r < FR; ++r) {
            const int u = xcd < NPX ? r * (25 * NPX) + po : (FR - 1) * (25 * NPX) + r * (25 * (8 - NPX)) + po;
            if (u >= 640) break;
            const int kc = u / 80, j = u - kc * 80;
            const bool isw = j < 64;
            const int dl = isw ? j >> 3 : (wid + 7) & 7, dest = (rank + 1 + dl) & 7;
            const rsrc_t rs_flag = mkrs(p.ws[dl], WS_NEED);
            const int ct = j & 7, rt = j - 64;
            u32x4 v[4];
            if (isw) {
                const unsigned so = (unsigned)(kc * 64) * (GN * 2) + (unsigned)(dest * 2048 + ct * 256) * 2;
                const unsigned vo = (unsigned)(tid >> 5) * (GN * 2) + (unsigned)(tid & 31) * 16;
#pragma unroll
                for (int i = 0; i < 4; ++i) v[i] = __builtin_amdgcn_raw_buffer_load_b128(rs_w, vo, so + i * 16 * (GN * 2), 0);
            } else {
#pragma unroll
                for (int i = 0; i < 4; ++i) {
                    const int F = wid * 4 + i;
                    v[i] = __builtin_amdgcn_raw_buffer_load_b128(rs_x, vx, (unsigned)(rt * 256 + (F >> 1) * 16) * (KS * 2) + kc * 128 + (F & 1) * 64, 0);
                }
            }
            if (r == 0) {
                SCHED;
                call = (unsigned)rfl((int)vcall) + 1u;
                exp_off = OFF_EXP + (call & 1u) * EXP_SZ;
            }
            if (isw) {
                const unsigned dst = exp_off + EXP_W + (unsigned)((dest * 8 + kc) * 8 + ct) * UNIT;
#pragma unroll
                for (int i = 0; i < 4; ++i) *(LAS u32x4*)(lds + (i * 16 + (tid >> 5)) * 576 + (tid & 31) * 16) = v[i];
                __syncthreads();
#pragma unroll
                for (int i = 0; i < 4; ++i) {
                    const int F = i * 8 + wid, bj = F >> 4, wcn = (F >> 1) & 7, kk = F & 1;
                    const int col = (wcn >> 1) * 64 + bj * 32 + (wcn & 1) * 4 + (lane & 3) * 8;
                    const int row = kk * 32 + (lane >> 4) * 8 + ((lane & 15) >> 2);
                    const s16x4 t0 = __builtin_amdgcn_ds_read_tr16_b64_v4i16((LAS s16x4*)(lds + row * 576 + col * 2));
                    const s16x4 t1 = __builtin_amdgcn_ds_read_tr16_b64_v4i16((LAS s16x4*)(lds + (row + 4) * 576 + col * 2));
                    const u32x2 a = __builtin_bit_cast(u32x2, t0), b = __builtin_bit_cast(u32x2, t1);
                    v[i] = (u32x4){a.x, a.y, b.x, b.y};
                }
#pragma unroll
                for (int i = 0; i < 4; ++i) ST128(v[i], rs_me, lane * 16, dst + (i * 8 + wid) * 1024, AUX_SYS);
            } else {
                const unsigned dst = exp_off + (unsigned)(kc * 16 + rt) * UNIT;
#pragma unroll
                for (int i = 0; i < 4; ++i) ST128(v[i], rs_me, lane * 16, dst + (wid * 4 + i) * 1024, AUX_SYS);
            }
            WAIT_V(0);
            if (r == 0 && pusher && lane == 0)
                __builtin_amdgcn_raw_buffer_store_b32(call, rs_push, 0, F_A + ((pw * 16 + prt) * 64 + rank * 8) * 4, AUX_SYS);
            __syncthreads();
            if (wid == 0) {
                if (isw) {
                    if (dest != rank && lane == 0)
                        __builtin_amdgcn_raw_buffer_store_b32(call, rs_flag, 0, F_PUB + (rank * 256 + kc * 24 + ct) * 4, AUX_SYS);
                    if (lane < 8)
                        __builtin_amdgcn_raw_buffer_store_b32(call, rs_me, lane * (64 * 64 * 4), F_B + ((dest * 8 + ct) * 64 + rank * 8 + kc) * 4, AUX_AGENT);
                } else {
                    if (lane < 8)
                        __builtin_amdgcn_raw_buffer_store_b32(call, rs_me, lane * (16 * 64 * 4), F_A + (rt * 64 + rank * 8 + kc) * 4, AUX_AGENT);
                }
                if (r == 0 && lane == 0) __builtin_amdgcn_raw_buffer_store_b32(call, rs_me, 0, F_CNT + bid * 128, AUX_AGENT);
            } else if (!isw && lane == 0) {
                if ((rt >> 1) != rank)
                    __builtin_amdgcn_raw_buffer_store_b32(call, rs_flag, 0, F_PUB + (rank * 256 + kc * 24 + 8 + rt) * 4, AUX_SYS);
            }
        }
    }
    const unsigned par = call & 1u;
    const unsigned yexp_off = OFF_YEXP + par * YEXP_SZ;
    STAMP_AT(1);

    if (slot < 7) {
        const int link = slot;
        const int src = (rank + 1 + link) & 7;
        const rsrc_t rs_src = mkrs(p.ws[link], WS_NEED);
        const rsrc_t rs_out = mkrs(p.out, OUT_BYTES);
        constexpr unsigned NIN = (176 - NPUSH) * 8, NTOT = NIN + 16 * 64, NDRAW = NTOT + 8 * PW;
        if (wid >= PW) return;
        unsigned vz = 0; asm volatile("" : "+v"(vz));
        gu32* const qctr = (gu32*)((char*)p.me + F_Q + link * 128) + vz;
        const unsigned qbase = (call - 1u) * NDRAW;
        unsigned vtick = 0;
        if (lane == 0) vtick = __hip_atomic_fetch_add(qctr, 1u, __ATOMIC_RELAXED, __HIP_MEMORY_SCOPE_AGENT);
        int code[3];
#pragma unroll
        for (int c = 0; c < 3; ++c) {
            int pos = lane + 64 * c, cd = 0; bool found = false;
            for (int s = 0; s < 8 + (LAG_W > LAG_O ? LAG_W : LAG_O); ++s) {
                if (s < 8) {
                    if (!found && pos < 8) { cd = s | 8 | (pos << 4); found = true; } pos -= 8;
                    if (s == 0) pos += NPUSH;
                    if (!found && pos < 6) { cd = s | ((2 * blk_at(rank, link, pos)) << 4); found = true; } pos -= 6;
                }
                if (s >= LAG_O && s - LAG_O < 8) {
                    if (!found && pos < 6) { cd = (s - LAG_O) | ((2 * blk_at(rank, link, pos) + 1) << 4); found = true; } pos -= 6;
                }
                if (s >= LAG_W && s - LAG_W < 8) { if (!found && pos < 2) { cd = (s - LAG_W) | ((2 * rank + pos) << 4); found = true; } pos -= 2; }
            }
            code[c] = found ? cd : 8;
        }
        int pub_ready = 0; bool bad = false;
        unsigned long long yav0 = 0, yav1 = 0;
        const unsigned vl16 = (unsigned)lane * 16u, v_fy = (unsigned)(lane * 4 + src * 512);
#define DRAIN() do { WAIT_V(0); SCHED; __builtin_amdgcn_s_waitcnt(0x0F70); SCHED; } while (0)
#define DRAW() do { if (lane == 0) vtick = __hip_atomic_fetch_add(qctr, 1u, __ATOMIC_RELAXED, __HIP_MEMORY_SCOPE_AGENT); } while (0)
#define TICKET() ((unsigned)rfl((int)vtick) - qbase)
#define RLOAD(off) __builtin_amdgcn_raw_buffer_load_b128(rs_src, vl16, (off), AUX_SYS)
#define LLOAD(off) __builtin_amdgcn_raw_buffer_load_b128(rs_me, vl16, (off), AUX_AGENT)
        auto in_decode = [&](unsigned t, unsigned& so, unsigned& dofs, unsigned& fl) {
            const int u = t >> 3, w = t & 7;
            if (u >= pub_ready) {
                const unsigned long long t0 = __builtin_amdgcn_s_memrealtime();
                for (;;) {
                    unsigned f[3];
#pragma unroll
                    for (int c = 0; c < 3; ++c)
                        f[c] = __builtin_amdgcn_raw_buffer_load_b32(rs_me, ((code[c] & 7) * 24 + ((code[c] & 8) ? 0 : 8) + (code[c] >> 4)) * 4, F_PUB + src * 1024, AUX_SYS);
                    const unsigned long long n0 = ~__ballot(f[0] >= call), n1 = ~__ballot(f[1] >= call), n2 = ~__ballot(f[2] >= call);
                    pub_ready = n0 ? __builtin_ctzll(n0) : n1 ? 64 + __builtin_ctzll(n1) : n2 ? 128 + __builtin_ctzll(n2) : 192;
                    if (u < pub_ready) break;
                    if (give_up(rs_me, call, t0)) { bad = true; break; }
                }
            }
            const int c0 = __builtin_amdgcn_readlane(code[0], u & 63), c1 = __builtin_amdgcn_readlane(code[1], u & 63), c2 = __builtin_amdgcn_readlane(code[2], u & 63);
            const int cd = u < 64 ? c0 : u < 128 ? c1 : c2, kc = cd & 7, idx = cd >> 4;
            if (cd & 8) {
                so = exp_off + EXP_W + (unsigned)((rank * 8 + kc) * 8 + idx) * UNIT;
                dofs = OFF_STG + src * STG_SZ + EXP_W + (unsigned)(kc * 8 + idx) * UNIT;
                fl = F_B + ((w * 64 + rank * 8 + idx) * 64 + src * 8 + kc) * 4;
            } else {
                so = exp_off + (unsigned)(kc * 16 + idx) * UNIT;
                dofs = OFF_STG + src * STG_SZ + (unsigned)(kc * 16 + idx) * UNIT;
                fl = F_A + ((w * 16 + idx) * 64 + src * 8 + kc) * 4;
            }
            so += w * 1024; dofs += w * 1024;
            STAMP_T(t);
        };
        auto y_decode = [&](unsigned t, unsigned& ys, unsigned& ds, unsigned& os) {
            const int q = t - NIN, tl = q >> 6, g = (q >> 3) & 7, am = q & 7;
            const int bit = (tl & 7) * 8 + g;
            if (!((((tl & 8) ? yav1 : yav0) >> bit) & 1ull) && !bad) {
                const unsigned long long t0 = __builtin_amdgcn_s_memrealtime();
                for (;;) {
                    const unsigned y0 = __builtin_amdgcn_raw_buffer_load_b32(rs_me, v_fy + F_Y, 0, AUX_SYS);
                    const unsigned y1 = __builtin_amdgcn_raw_buffer_load_b32(rs_me, v_fy + F_Y + 256, 0, AUX_SYS);
                    const unsigned d0 = __builtin_amdgcn_raw_buffer_load_b32(rs_me, v_fy + F_DG, 0, AUX_AGENT);
                    const unsigned d1 = __builtin_amdgcn_raw_buffer_load_b32(rs_me, v_fy + F_DG + 256, 0, AUX_AGENT);
                    yav0 = __ballot((y0 < d0 ? y0 : d0) >= call); yav1 = __ballot((y1 < d1 ? y1 : d1) >= call);
                    if ((((tl & 8) ? yav1 : yav0) >> bit) & 1ull) break;
                    if (give_up(rs_me, call, t0)) { bad = true; break; }
                }
            }
            ys = yexp_off + (unsigned)(rank * 16 + tl) * 131072u + g * 16384 + am * 2048;
            ds = OFF_DIAG + (unsigned)(src * 16 + tl) * 262144u + g * 32768 + am * 4096;
            os = (unsigned)((tl >> 3) * 256 + (am >> 2) * 128 + (g >> 2) * 64 + (am & 3) * 16) * (GN * 2)
               + (unsigned)(src * 2048 + (tl & 7) * 256 + (g & 3) * 64) * 2;
            STAMP_T(t);
        };
        unsigned t = TICKET();
        if (t < NIN) {
            unsigned so, dofs, fl, prev = 0;
            in_decode(t, so, dofs, fl);
            STAMP_AT(5);
            if (!bad) {
                u32x4 A = RLOAD(so), B;
                for (;;) {
                    DRAW();
                    DRAIN(); B = RLOAD(so + 8192); SCHED; ST128(A, rs_me, vl16, dofs, AUX_AGENT);
                    if (prev && lane == 0) __builtin_amdgcn_raw_buffer_store_b32(call, rs_me, 0, prev, AUX_AGENT);
                    t = TICKET();
                    DRAIN(); A = RLOAD(so + 16384); SCHED; ST128(B, rs_me, vl16, dofs + 8192, AUX_AGENT);
                    DRAIN(); B = RLOAD(so + 24576); SCHED; ST128(A, rs_me, vl16, dofs + 16384, AUX_AGENT);
                    const unsigned dlast = dofs + 24576; prev = fl;
                    bool more = t < NIN;
                    if (more) { in_decode(t, so, dofs, fl); more = !bad; }
                    DRAIN();
                    if (more) A = RLOAD(so);
                    SCHED; ST128(B, rs_me, vl16, dlast, AUX_AGENT);
                    if (!more) break;
                }
                DRAIN();
                if (lane == 0) __builtin_amdgcn_raw_buffer_store_b32(call, rs_me, 0, prev, AUX_AGENT);
            }
            while (t < NIN) { DRAW(); t = TICKET(); }
        }
        STAMP_AT(2);
        if (t < NTOT) {
            const unsigned v_out = (unsigned)(lane & 15) * (GN * 2) + (unsigned)(lane >> 4) * 16;
            auto finish = [&](const u32x4 y, const u32x4 d0, const u32x4 d1, unsigned os) {
                u32x4 o;
                o.x = cvt_pk_bf16(silu(__uint_as_float(y.x << 16) + __uint_as_float(d0.x)), silu(__uint_as_float(y.x & 0xffff0000u) + __uint_as_float(d0.y))) | (bad ? 0x7fc07fc0u : 0u);
                o.y = cvt_pk_bf16(silu(__uint_as_float(y.y << 16) + __uint_as_float(d0.z)), silu(__uint_as_float(y.y & 0xffff0000u) + __uint_as_float(d0.w)));
                o.z = cvt_pk_bf16(silu(__uint_as_float(y.z << 16) + __uint_as_float(d1.x)), silu(__uint_as_float(y.z & 0xffff0000u) + __uint_as_float(d1.y)));
                o.w = cvt_pk_bf16(silu(__uint_as_float(y.w << 16) + __uint_as_float(d1.z)), silu(__uint_as_float(y.w & 0xffff0000u) + __uint_as_float(d1.w)));
                ST128(o, rs_out, v_out, os, AUX_AGENT);
            };
            unsigned ys, ds, os;
            y_decode(t, ys, ds, os);
            u32x4 A = RLOAD(ys), a0 = LLOAD(ds), a1 = LLOAD(ds + 1024);
            DRAW();
            for (;;) {
                DRAIN();
                t = TICKET();
                const u32x4 B = RLOAD(ys + 1024), b0 = LLOAD(ds + 2048), b1 = LLOAD(ds + 3072);
                SCHED;
                finish(A, a0, a1, os);
                const unsigned ol = os + 64;
                const bool more = t < NTOT;
                if (more) y_decode(t, ys, ds, os);
                DRAIN();
                if (more) { A = RLOAD(ys); a0 = LLOAD(ds); a1 = LLOAD(ds + 1024); DRAW(); }
                SCHED;
                finish(B, b0, b1, ol);
                if (!more) break;
            }
        }
        STAMP_AT(3);
        return;
    }

    const int wr = wid >> 2, wc = wid & 3;
    int nt, seglen, mode, tile0;
    unsigned v_a, v_b, v_fa, v_fb;
    int dest;
    if (slot < 23 || slot >= 30) {
        const bool helper = slot >= 30;
        const int i = slot - 7;
        const int j = helper ? rank : ((xcd >> 1) & 1) * 4 + (i & 3), rt2 = helper ? slot - 30 : xcd & 1;
        const int rt = 2 * j + rt2, ct = helper ? xcd : (xcd >> 2) * 4 + (i >> 2);
        const bool own = (j == rank);
        const int nsrc = own ? 7 : 6;
        nt = own ? 32 : 56; seglen = nt; mode = helper ? 2 : own ? 0 : 1; dest = j; tile0 = rt2 * 8 + ct;
        const int st = own ? 2 * (lane & 31) + (helper ? 0 : 1) : lane;
        int src, kc, si = 0;
        if (st < 8 || st >= 8 + 8 * nsrc) { src = rank; kc = st & 7; }
        else {
            const int q = st - 8; kc = q / nsrc; si = q - kc * nsrc;
            int l = ((j - rank - 1) & 7) - 1 - si; if (l < 0) l += 7;
            src = (rank + 1 + l) & 7;
        }
        v_a = (src == rank ? exp_off : OFF_STG + src * STG_SZ) + (unsigned)(kc * 16 + rt) * UNIT;
        if (src != rank && kc == 0 && !own && rt2 == 0 && si < NPUSH)
            v_a = OFF_PUSH + (unsigned)((par * 8 + src) * 8 + j) * UNIT;
        v_b = (src == rank ? exp_off + (unsigned)(rank * 64) * UNIT : OFF_STG + src * STG_SZ) + EXP_W + (unsigned)(kc * 8 + ct) * UNIT;
        v_fa = F_A + ((wid * 16 + rt) * 64 + src * 8 + kc) * 4;
        v_fb = F_B + ((wid * 64 + rank * 8 + ct) * 64 + src * 8 + kc) * 4;
    } else {
        const int slab = (rank + 1 + (slot - 23)) & 7, ct = xcd;
        nt = 16; seglen = 8; mode = 2; dest = slab; tile0 = ct;
        const int kc = lane & 7, rt = 2 * rank + ((lane >> 3) & 1);
        v_a = exp_off + (unsigned)(kc * 16 + rt) * UNIT;
        v_b = exp_off + EXP_W + (unsigned)((slab * 8 + kc) * 8 + ct) * UNIT;
        v_fa = F_A + ((wid * 16 + rt) * 64 + rank * 8 + kc) * 4;
        v_fb = F_B + ((wid * 64 + slab * 8 + ct) * 64 + rank * 8 + kc) * 4;
    }
    int ready = 0; bool bad = false;
    auto ensure = [&](int need) {
        if (need > nt) need = nt;
        if (ready >= need) return;
        const unsigned long long t0 = __builtin_amdgcn_s_memrealtime();
        __builtin_amdgcn_s_waitcnt(0xC07F);
        for (;;) {
            const unsigned fa = __builtin_amdgcn_raw_buffer_load_b32(rs_me, v_fa, 0, AUX_SYS);
            const unsigned fb = __builtin_amdgcn_raw_buffer_load_b32(rs_me, v_fb, 0, AUX_AGENT);
            const unsigned long long nm = ~__ballot((fa < fb ? fa : fb) >= call);
            ready = nm ? __builtin_ctzll(nm) : 64;
            if (ready >= need) break;
            if (give_up(rs_me, call, t0)) { bad = true; ready = 64; break; }
            __builtin_amdgcn_s_sleep(4);
        }
    };
    const unsigned vst = (unsigned)tid * 16u;
    const int ldsw = wid * 1024;
    const int aoff = wr * 8192 + lane * 16;
    int boff = 65536 + wc * 4096 + lane * 16; asm volatile("" : "+v"(boff));
#define SA(b, h) (((b) * 2 + (h)) * 16384)
#define SB(b, h) ((4 + (b) * 2 + (h)) * 16384)
#define STAGE(bufoff, goff) do { _Pragma("unroll") for (int _i = 0; _i < 2; ++_i) \
        __builtin_amdgcn_raw_ptr_buffer_load_lds(rs_me, (LAS void*)(lds + (bufoff) + ldsw + _i * 8192), 16, vst, (goff) + _i * 8192, 0, AUX_AGENT); } while (0)
#define LDA(dst, b, h) do { _Pragma("unroll") for (int m = 0; m < 4; ++m) _Pragma("unroll") for (int k = 0; k < 2; ++k) \
        dst[m][k] = *(const LAS bf16x8*)(lds + SA(b, h) + aoff + m * 2048 + k * 1024); } while (0)
#define LDB(dst, b, h) do { _Pragma("unroll") for (int n = 0; n < 2; ++n) _Pragma("unroll") for (int k = 0; k < 2; ++k) \
        dst[n][k] = *(const LAS bf16x8*)(lds + boff + (SB(b, h) - 65536 + n * 2048 + k * 1024)); } while (0)
#define MMA(ai, bj, At, Bt) do { __builtin_amdgcn_s_setprio(1); _Pragma("unroll") for (int m = 0; m < 4; ++m) _Pragma("unroll") for (int n = 0; n < 2; ++n) \
        _Pragma("unroll") for (int k = 0; k < 2; ++k) acc[ai][bj][m][n] = __builtin_amdgcn_mfma_f32_16x16x32_bf16(Bt[n][k], At[m][k], acc[ai][bj][m][n], 0, 0, 0); \
        __builtin_amdgcn_s_setprio(0); } while (0)
#define AOFF(t) ((unsigned)__builtin_amdgcn_readlane((int)v_a, (t)))
#define BOFF(t) ((unsigned)__builtin_amdgcn_readlane((int)v_b, (t)))
    f32x4 acc[2][2][4][2];
#pragma unroll
    for (int a = 0; a < 2; ++a)
#pragma unroll
        for (int b = 0; b < 2; ++b)
#pragma unroll
            for (int m = 0; m < 4; ++m)
#pragma unroll
                for (int n = 0; n < 2; ++n) acc[a][b][m][n] = (f32x4){0.f, 0.f, 0.f, 0.f};
    bf16x8 At[4][2], B0[2][2], B1[2][2];
    ensure(2);
    {
        const unsigned a0 = AOFF(0), b0 = BOFF(0), a1 = AOFF(1), b1 = BOFF(1);
        STAGE(SB(0, 0), b0); STAGE(SA(0, 0), a0); STAGE(SB(0, 1), b0 + 16384); STAGE(SA(0, 1), a0 + 16384);
        if (wr == 1) BAR;
        WAIT_V(4); BAR;
        STAGE(SB(1, 0), b1); STAGE(SA(1, 0), a1); STAGE(SB(1, 1), b1 + 16384);
        WAIT_V(6); BAR;
    }
    STAMP_AT(2);
    int tile = tile0;
    for (int t = 0;;) {
        const int seg_end = t + seglen;
        for (; t < seg_end; t += 2) {
            const int t2 = (t + 2 < nt) ? t + 2 : 0;
            const unsigned a1 = AOFF(t + 1), a2 = AOFF(t2), b2 = BOFF(t2), a3 = AOFF(t2 + 1), b3 = BOFF(t2 + 1);
            LDB(B0, 0, 0); SCHED; LDA(At, 0, 0); STAGE(SA(1, 1), a1 + 16384);
            WAIT_L(8); BAR; WAIT_L(0); MMA(0, 0, At, B0); BAR; SCHED;
            ensure(t + 3);
            LDB(B1, 0, 1); STAGE(SB(0, 0), b2);
            BAR; WAIT_L(0); MMA(0, 1, At, B1); BAR;
            LDA(At, 0, 1); STAGE(SA(0, 0), a2);
            BAR; WAIT_L(0); MMA(1, 0, At, B0); BAR; SCHED;
            STAGE(SB(0, 1), b2 + 16384);
            WAIT_V(6); BAR; MMA(1, 1, At, B1); BAR;
            LDB(B0, 1, 0); SCHED; LDA(At, 1, 0); STAGE(SA(0, 1), a2 + 16384);
            WAIT_L(8); BAR; WAIT_L(0); MMA(0, 0, At, B0); BAR; SCHED;
            ensure(t + 4);
            LDB(B1, 1, 1); STAGE(SB(1, 0), b3);
            BAR; WAIT_L(0); MMA(0, 1, At, B1); BAR;
            LDA(At, 1, 1); STAGE(SA(1, 0), a3);
            BAR; WAIT_L(0); MMA(1, 0, At, B0); BAR; SCHED;
            STAGE(SB(1, 1), b3 + 16384);
            WAIT_V(6); BAR; MMA(1, 1, At, B1); BAR;
        }
        if (t >= nt && wr == 0) BAR;
        STAMP_AT(3);
        if (mode != 2) break;
        {
            int lz = lane; asm volatile("" : "+v"(lz));
            const unsigned nanm = bad ? 0x7fc00000u : 0u;
            const unsigned so = OFF_DIAG + (unsigned)(dest * 16 + tile) * 262144u + wid * 32768;
#pragma unroll
            for (int a = 0; a < 2; ++a)
#pragma unroll
                for (int m = 0; m < 4; ++m)
#pragma unroll
                    for (int b = 0; b < 2; ++b)
#pragma unroll
                        for (int n = 0; n < 2; ++n) {
                            u32x4 o = __builtin_bit_cast(u32x4, acc[a][b][m][n]); o.x |= nanm;
                            ST128(o, rs_me, lz * 16 + (b * 2 + n) * 1024, so + (a * 4 + m) * 4096, AUX_AGENT);
                        }
            WAIT_V(0);
            if (lz == 0) __builtin_amdgcn_raw_buffer_store_b32(call, rs_me, 0, F_DG + ((dest * 16 + tile) * 8 + wid) * 4, AUX_AGENT);
        }
        STAMP_AT(4);
        if (t >= nt) break;
#pragma unroll
        for (int a = 0; a < 2; ++a)
#pragma unroll
            for (int b = 0; b < 2; ++b)
#pragma unroll
                for (int m = 0; m < 4; ++m)
#pragma unroll
                    for (int n = 0; n < 2; ++n) acc[a][b][m][n] = (f32x4){0.f, 0.f, 0.f, 0.f};
        tile += 8;
    }
    if (mode != 2) {
        int lz = lane; asm volatile("" : "+v"(lz));
        if (mode == 0) {
            {
                const unsigned long long t0 = __builtin_amdgcn_s_memrealtime();
                while (!bad && (unsigned)rfl((int)__builtin_amdgcn_raw_buffer_load_b32(rs_me, 0, F_DG + ((rank * 16 + tile) * 8 + wid) * 4, AUX_AGENT)) < call)
                    if (give_up(rs_me, call, t0)) bad = true;
            }
            const unsigned nanm = bad ? 0x7fc07fc0u : 0u;
            const rsrc_t rs_out = mkrs(p.out, OUT_BYTES);
            const unsigned vo = (unsigned)(lz & 15) * (GN * 2) + (unsigned)(lz >> 4) * 16;
            const unsigned so = (unsigned)((tile >> 3) * 256 + wr * 64) * (GN * 2) + (unsigned)(rank * 2048 + (tile & 7) * 256 + wc * 64) * 2;
            const unsigned hs = OFF_DIAG + (unsigned)(rank * 16 + tile) * 262144u + wid * 32768;
#pragma unroll
            for (int a = 0; a < 2; ++a) {
                f32x4 h[4][2][2];
#pragma unroll
                for (int m = 0; m < 4; ++m)
#pragma unroll
                    for (int b = 0; b < 2; ++b)
#pragma unroll
                        for (int n = 0; n < 2; ++n)
                            h[m][b][n] = __builtin_bit_cast(f32x4, __builtin_amdgcn_raw_buffer_load_b128(rs_me, lz * 16 + (b * 2 + n) * 1024, hs + (a * 4 + m) * 4096, AUX_AGENT));
#pragma unroll
                for (int m = 0; m < 4; ++m)
#pragma unroll
                    for (int b = 0; b < 2; ++b) {
                        const f32x4 v0 = acc[a][b][m][0] + h[m][b][0], v1 = acc[a][b][m][1] + h[m][b][1];
                        u32x4 o; o.x = cvt_pk_bf16(silu(v0[0]), silu(v0[1])) | nanm; o.y = cvt_pk_bf16(silu(v0[2]), silu(v0[3]));
                        o.z = cvt_pk_bf16(silu(v1[0]), silu(v1[1])); o.w = cvt_pk_bf16(silu(v1[2]), silu(v1[3]));
                        ST128(o, rs_out, vo + b * 64, so + (unsigned)(a * 128 + m * 16) * (GN * 2), AUX_AGENT);
                    }
                asm volatile("" ::: "memory");
            }
        } else {
            const unsigned nanm = bad ? 0x7fc07fc0u : 0u;
            const unsigned so = yexp_off + (unsigned)(dest * 16 + tile) * 131072u + wid * 16384;
#pragma unroll
            for (int a = 0; a < 2; ++a)
#pragma unroll
                for (int m = 0; m < 4; ++m)
#pragma unroll
                    for (int b = 0; b < 2; ++b) {
                        const f32x4 v0 = acc[a][b][m][0], v1 = acc[a][b][m][1];
                        u32x4 o; o.x = cvt_pk_bf16(v0[0], v0[1]) | nanm; o.y = cvt_pk_bf16(v0[2], v0[3]);
                        o.z = cvt_pk_bf16(v1[0], v1[1]); o.w = cvt_pk_bf16(v1[2], v1[3]);
                        ST128(o, rs_me, lz * 16 + b * 1024, so + (a * 4 + m) * 2048, AUX_SYS);
                    }
            WAIT_V(0);
            if (lz == 0) __builtin_amdgcn_raw_buffer_store_b32(call, mkrs(p.ws[(dest - rank - 1) & 7], WS_NEED), 0, F_Y + ((rank * 16 + tile) * 8 + wid) * 4, AUX_SYS);
        }
        STAMP_AT(4);
    }
    WAIT_V(0);
}

}

extern "C" void kernel_launch(int rank, int nranks, ncclComm_t comm,
                              hipStream_t stream,
                              void* const* d_in, const int* in_sizes, int n_in,
                              void* d_out, int out_size,
                              const int* in_dtypes, int out_dtype,
                              void* const* d_out_all,
                              void* const* d_ws_all, size_t ws_bytes) {
    (void)comm; (void)in_dtypes; (void)out_dtype; (void)d_out_all;
    if (nranks != NR || (unsigned)rank >= (unsigned)NR || n_in < 2 || ws_bytes < (size_t)WS_NEED ||
        in_sizes[0] != GM * KS || in_sizes[1] != KS * GN || out_size != (GM / NR) * GN) {
        fprintf(stderr, "kernel_launch: unexpected problem shape\n");
        return;
    }
    Params p;
    p.x = d_in[0]; p.w = d_in[1]; p.out = d_out; p.me = d_ws_all[rank]; p.rank = rank; p.pad = 0;
    for (int i = 0; i < NR; ++i) p.ws[i] = d_ws_all[(rank + 1 + i) & 7];
    hipLaunchKernelGGL(fused_gemm_rs, dim3(256), dim3(512), 0, stream, p);
    HIP_CHECK(hipGetLastError());
}
```

```cpp
#include <hip/hip_runtime.h>
#include <rccl/rccl.h>
#include <cstdio>

#define HIP_CHECK(expr) do { hipError_t _e = (expr); if (_e != hipSuccess) { \
    fprintf(stderr, "HIP error %s at %s:%d\n", hipGetErrorString(_e), __FILE__, __LINE__); \
    return; } } while (0)

namespace {

constexpr int NR = 8;
constexpr int GM = 4096, GK = 4096, GN = 16384;
constexpr int KS = GK / NR;
constexpr unsigned MiB = 1u << 20;
constexpr unsigned UNIT = 32768;
constexpr unsigned X_BYTES = GM * KS * 2, W_BYTES = KS * GN * 2, OUT_BYTES = (GM / NR) * GN * 2;

constexpr unsigned EXP_SZ = 20 * MiB;
constexpr unsigned EXP_W = 4 * MiB;
constexpr unsigned OFF_EXP = 2 * MiB;
constexpr unsigned YEXP_SZ = 16 * MiB;
constexpr unsigned OFF_YEXP = OFF_EXP + 2 * EXP_SZ;
constexpr unsigned STG_SZ = 6 * MiB;
constexpr unsigned OFF_STG = OFF_YEXP + 2 * YEXP_SZ;
constexpr unsigned OFF_DIAG = OFF_STG + NR * STG_SZ;
constexpr unsigned OFF_PUSH = OFF_DIAG + 32 * MiB;
constexpr unsigned WS_NEED = OFF_PUSH + 4 * MiB;
constexpr unsigned F_CNT = 0;
constexpr unsigned F_TMO = 32768;
constexpr unsigned F_PUB = 36864;
constexpr unsigned F_Y = 49152;
constexpr unsigned F_DG = 57344;
constexpr unsigned F_Q = 33792;
constexpr unsigned F_A = 65536;
constexpr unsigned F_B = 131072;
constexpr unsigned F_STAMP = 524288;

constexpr unsigned long long DEADLINE = 100000000ull;

constexpr int NPX = 3;
constexpr int FR = 4;
constexpr int NPUSH = 5;
constexpr int LAG_O = 2;
constexpr int LAG_W = 6;

static_assert(NPUSH >= 1 && NPUSH <= 6 && 56 * NPUSH <= 8 * 25 * (8 - NPX), "push pieces");
static_assert(FR * 200 >= 640, "every unit is published");
constexpr int AUX_AGENT = 16, AUX_SYS = 17;

typedef short bf16x8 __attribute__((ext_vector_type(8)));
typedef short s16x4 __attribute__((ext_vector_type(4)));
typedef float f32x4 __attribute__((ext_vector_type(4)));
typedef unsigned u32x4 __attribute__((ext_vector_type(4)));
typedef unsigned u32x2 __attribute__((ext_vector_type(2)));
#define LAS __attribute__((address_space(3)))
typedef __amdgpu_buffer_rsrc_t rsrc_t;
typedef __attribute__((address_space(1))) unsigned gu32;

struct Params {
    const void* x; const void* w; void* out; void* me;
    int rank, pad;
    void* ws[NR];
};

__device__ __forceinline__ rsrc_t mkrs(const void* p, unsigned bytes) {
    return __builtin_amdgcn_make_buffer_rsrc((void*)p, 0, bytes, 0x00020000);
}
__device__ __forceinline__ unsigned cvt_pk_bf16(float lo, float hi) {
    typedef float f32x2 __attribute__((ext_vector_type(2)));
    typedef __bf16 bf16x2 __attribute__((ext_vector_type(2)));
    return __builtin_bit_cast(unsigned, __builtin_convertvector((f32x2){lo, hi}, bf16x2));
}
__device__ __forceinline__ float silu(float y) {
    const float c = __builtin_amdgcn_fmed3f(y, -60.0f, 60.0f);
    const float e = __builtin_amdgcn_exp2f(c * -1.44269504088896341f);
    return y * __builtin_amdgcn_rcpf(1.0f + e);
}
__device__ __forceinline__ int rfl(int v) { return __builtin_amdgcn_readfirstlane(v); }
__device__ __forceinline__ int blk_at(int me, int l, int pos) { int v = l + 1 + pos; if (v >= 7) v -= 7; return (me + 1 + v) & 7; }
#define ST128(v, rs, voff, soff, aux) __builtin_amdgcn_raw_buffer_store_b128((v), (rs), (unsigned)(voff) + (unsigned)(soff), 0, (aux))
#define WAIT_V(n) asm volatile("s_waitcnt vmcnt(" #n ")" ::: "memory")
#define WAIT_L(n) asm volatile("s_waitcnt lgkmcnt(" #n ")" ::: "memory")
#define BAR __builtin_amdgcn_s_barrier()
#define SCHED __builtin_amdgcn_sched_barrier(0)

#define STAMP_AT(slot) do { } while (0)
#define STAMP_T(t) do { } while (0)

__device__ __forceinline__ bool give_up(rsrc_t rs_me, unsigned call, unsigned long long t0) {
    const unsigned tm = __builtin_amdgcn_raw_buffer_load_b32(rs_me, 0, F_TMO, AUX_AGENT);
    if ((unsigned)rfl((int)tm) == call || __builtin_amdgcn_s_memrealtime() - t0 > DEADLINE) {
        __builtin_amdgcn_raw_buffer_store_b32(call, rs_me, 0, F_TMO, AUX_AGENT);
        return true;
    }
    __builtin_amdgcn_s_sleep(2);
    asm volatile("" ::: "memory");
    return false;
}

__global__ __launch_bounds__(512, 2) void fused_gemm_rs(const Params p) {
    __shared__ __attribute__((aligned(16))) unsigned char smem[131072];
    LAS unsigned char* lds = (LAS unsigned char*)smem;
    const int tid = threadIdx.x, lane = tid & 63, wid = rfl(tid >> 6);
    const int bid = blockIdx.x, xcd = bid & 7, slot = bid >> 3;
    const int rank = p.rank;
    const int po = xcd < NPX ? (slot - 7) * NPX + xcd : 25 * NPX + (slot - 7) * (8 - NPX) + (xcd - NPX);
    constexpr int PWV = (56 * NPUSH + 25 * (8 - NPX) - 1) / (25 * (8 - NPX));
    const bool pusher = slot >= 7 && xcd >= NPX && wid < PWV && (po - 25 * NPX) * PWV + wid < 56 * NPUSH;
    const int pi = pusher ? (po - 25 * NPX) * PWV + wid : 0, pe = pi / 7;
    void* const push_ws = p.ws[pi - pe * 7];
    asm volatile("" :: "s"(p.me), "s"(p.x), "s"(p.w), "s"(rank), "s"(push_ws));
    const rsrc_t rs_me = mkrs(p.me, WS_NEED);

    STAMP_AT(0);
    const unsigned vcall = __builtin_amdgcn_raw_buffer_load_b32(rs_me, 0, F_CNT + bid * 128, AUX_AGENT);
    unsigned call = 0, exp_off = 0;

    if (slot < 7) {
        call = (unsigned)rfl((int)vcall) + 1u;
        exp_off = OFF_EXP + (call & 1u) * EXP_SZ;
        __syncthreads();
        if (tid == 0) __builtin_amdgcn_raw_buffer_store_b32(call, rs_me, 0, F_CNT + bid * 128, AUX_AGENT);
    } else {
        const rsrc_t rs_x = mkrs(p.x, X_BYTES), rs_w = mkrs(p.w, W_BYTES);
        const unsigned vx = (unsigned)(lane & 15) * (KS * 2) + (unsigned)(lane >> 4) * 16;
        const int pdest = (rank + 1 + pi - pe * 7) & 7, pw = pe / NPUSH, pu = pe - pw * NPUSH;
        const int prt = 2 * blk_at(pdest, (rank - pdest - 1) & 7, pu);
        const rsrc_t rs_push = mkrs(push_ws, WS_NEED);
        if (pusher) {
            u32x4 pv[4];
#pragma unroll
            for (int i = 0; i < 4; ++i)
                pv[i] = __builtin_amdgcn_raw_buffer_load_b128(rs_x, vx, (unsigned)(prt * 256 + ((pw >> 1) + 4 * i) * 16) * (KS * 2) + (pw & 1) * 64, 0);
            SCHED;
            const unsigned d = OFF_PUSH + (unsigned)((((rfl((int)vcall) + 1) & 1) * 8 + rank) * 8 + (prt >> 1)) * UNIT + pw * 1024;
#pragma unroll
            for (int i = 0; i < 4; ++i) ST128(pv[i], rs_push, lane * 16, d + i * 8192, AUX_SYS);
            SCHED;
        }
        for (int r = 0; r < FR; ++r) {
            const int u = xcd < NPX ? r * (25 * NPX) + po : (FR - 1) * (25 * NPX) + r * (25 * (8 - NPX)) + po;
            if (u >= 640) break;
            const int kc = u / 80, j = u - kc * 80;
            const bool isw = j < 64;
            const int dl = isw ? j >> 3 : (wid + 7) & 7, dest = (rank + 1 + dl) & 7;
            const rsrc_t rs_flag = mkrs(p.ws[dl], WS_NEED);
            const int ct = j & 7, rt = j - 64;
            u32x4 v[4];
            if (isw) {
                const unsigned so = (unsigned)(kc * 64) * (GN * 2) + (unsigned)(dest * 2048 + ct * 256) * 2;
                const unsigned vo = (unsigned)(tid >> 5) * (GN * 2) + (unsigned)(tid & 31) * 16;
#pragma unroll
                for (int i = 0; i < 4; ++i) v[i] = __builtin_amdgcn_raw_buffer_load_b128(rs_w, vo, so + i * 16 * (GN * 2), 0);
            } else {
#pragma unroll
                for (int i = 0; i < 4; ++i) {
                    const int F = wid * 4 + i;
                    v[i] = __builtin_amdgcn_raw_buffer_load_b128(rs_x, vx, (unsigned)(rt * 256 + (F >> 1) * 16) * (KS * 2) + kc * 128 + (F & 1) * 64, 0);
                }
            }
            if (r == 0) {
                SCHED;
                call = (unsigned)rfl((int)vcall) + 1u;
                exp_off = OFF_EXP + (call & 1u) * EXP_SZ;
            }
            if (isw) {
                const unsigned dst = exp_off + EXP_W + (unsigned)((dest * 8 + kc) * 8 + ct) * UNIT;
#pragma unroll
                for (int i = 0; i < 4; ++i) *(LAS u32x4*)(lds + (i * 16 + (tid >> 5)) * 576 + (tid & 31) * 16) = v[i];
                __syncthreads();
#pragma unroll
                for (int i = 0; i < 4; ++i) {
                    const int F = i * 8 + wid, bj = F >> 4, wcn = (F >> 1) & 7, kk = F & 1;
                    const int col = (wcn >> 1) * 64 + bj * 32 + (wcn & 1) * 4 + (lane & 3) * 8;
                    const int row = kk * 32 + (lane >> 4) * 8 + ((lane & 15) >> 2);
                    const s16x4 t0 = __builtin_amdgcn_ds_read_tr16_b64_v4i16((LAS s16x4*)(lds + row * 576 + col * 2));
                    const s16x4 t1 = __builtin_amdgcn_ds_read_tr16_b64_v4i16((LAS s16x4*)(lds + (row + 4) * 576 + col * 2));
                    const u32x2 a = __builtin_bit_cast(u32x2, t0), b = __builtin_bit_cast(u32x2, t1);
                    v[i] = (u32x4){a.x, a.y, b.x, b.y};
                }
#pragma unroll
                for (int i = 0; i < 4; ++i) ST128(v[i], rs_me, lane * 16, dst + (i * 8 + wid) * 1024, AUX_SYS);
            } else {
                const unsigned dst = exp_off + (unsigned)(kc * 16 + rt) * UNIT;
#pragma unroll
                for (int i = 0; i < 4; ++i) ST128(v[i], rs_me, lane * 16, dst + (wid * 4 + i) * 1024, AUX_SYS);
            }
            WAIT_V(0);
            if (r == 0 && pusher && lane == 0)
                __builtin_amdgcn_raw_buffer_store_b32(call, rs_push, 0, F_A + ((pw * 16 + prt) * 64 + rank * 8) * 4, AUX_SYS);
            __syncthreads();
            if (wid == 0) {
                if (isw) {
                    if (dest != rank && lane == 0)
                        __builtin_amdgcn_raw_buffer_store_b32(call, rs_flag, 0, F_PUB + (rank * 256 + kc * 24 + ct) * 4, AUX_SYS);
                    if (lane < 8)
                        __builtin_amdgcn_raw_buffer_store_b32(call, rs_me, lane * (64 * 64 * 4), F_B + ((dest * 8 + ct) * 64 + rank * 8 + kc) * 4, AUX_AGENT);
                } else {
                    if (lane < 8)
                        __builtin_amdgcn_raw_buffer_store_b32(call, rs_me, lane * (16 * 64 * 4), F_A + (rt * 64 + rank * 8 + kc) * 4, AUX_AGENT);
                }
                if (r == 0 && lane == 0) __builtin_amdgcn_raw_buffer_store_b32(call, rs_me, 0, F_CNT + bid * 128, AUX_AGENT);
            } else if (!isw && lane == 0) {
                if ((rt >> 1) != rank)
                    __builtin_amdgcn_raw_buffer_store_b32(call, rs_flag, 0, F_PUB + (rank * 256 + kc * 24 + 8 + rt) * 4, AUX_SYS);
            }
        }
    }
    const unsigned par = call & 1u;
    const unsigned yexp_off = OFF_YEXP + par * YEXP_SZ;
    STAMP_AT(1);

    if (slot < 7) {
        const int link = slot;
        const int src = (rank + 1 + link) & 7;
        const rsrc_t rs_src = mkrs(p.ws[link], WS_NEED);
        const rsrc_t rs_out = mkrs(p.out, OUT_BYTES);
        constexpr unsigned NIN = (176 - NPUSH) * 8, NTOT = NIN + 16 * 64, NDRAW = NTOT + 64;
        unsigned vz = 0; asm volatile("" : "+v"(vz));
        gu32* const qctr = (gu32*)((char*)p.me + F_Q + link * 128) + vz;
        const unsigned qbase = (call - 1u) * NDRAW;
        unsigned vtick = 0;
        if (lane == 0) vtick = __hip_atomic_fetch_add(qctr, 1u, __ATOMIC_RELAXED, __HIP_MEMORY_SCOPE_AGENT);
        int code[3];
#pragma unroll
        for (int c = 0; c < 3; ++c) {
            int pos = lane + 64 * c, cd = 0; bool found = false;
            for (int s = 0; s < 8 + (LAG_W > LAG_O ? LAG_W : LAG_O); ++s) {
                if (s < 8) {
                    if (!found && pos < 8) { cd = s | 8 | (pos << 4); found = true; } pos -= 8;
                    if (s == 0) pos += NPUSH;
                    if (!found && pos < 6) { cd = s | ((2 * blk_at(rank, link, pos)) << 4); found = true; } pos -= 6;
                }
                if (s >= LAG_O && s - LAG_O < 8) {
                    if (!found && pos < 6) { cd = (s - LAG_O) | ((2 * blk_at(rank, link, pos) + 1) << 4); found = true; } pos -= 6;
                }
                if (s >= LAG_W && s - LAG_W < 8) { if (!found && pos < 2) { cd = (s - LAG_W) | ((2 * rank + pos) << 4); found = true; } pos -= 2; }
            }
            code[c] = found ? cd : 8;
        }
        int pub_ready = 0; bool bad = false;
        unsigned long long yav0 = 0, yav1 = 0;
        const unsigned vl16 = (unsigned)lane * 16u, v_fy = (unsigned)(lane * 4 + src * 512);
#define DRAIN() do { WAIT_V(0); SCHED; __builtin_amdgcn_s_waitcnt(0x0F70); SCHED; } while (0)
#define YWAIT() do { WAIT_V(1); SCHED; __builtin_amdgcn_s_waitcnt(0x0F71); SCHED; } while (0)
#define DRAW() do { if (lane == 0) vtick = __hip_atomic_fetch_add(qctr, 1u, __ATOMIC_RELAXED, __HIP_MEMORY_SCOPE_AGENT); } while (0)
#define TICKET() ((unsigned)rfl((int)vtick) - qbase)
#define RLOAD(off) __builtin_amdgcn_raw_buffer_load_b128(rs_src, vl16, (off), AUX_SYS)
#define LLOAD(off) __builtin_amdgcn_raw_buffer_load_b128(rs_me, vl16, (off), AUX_AGENT)
        auto in_decode = [&](unsigned t, unsigned& so, unsigned& dofs, unsigned& fl) {
            const int u = t >> 3, w = t & 7;
            if (u >= pub_ready) {
                const unsigned long long t0 = __builtin_amdgcn_s_memrealtime();
                for (;;) {
                    unsigned f[3];
#pragma unroll
                    for (int c = 0; c < 3; ++c)
                        f[c] = __builtin_amdgcn_raw_buffer_load_b32(rs_me, ((code[c] & 7) * 24 + ((code[c] & 8) ? 0 : 8) + (code[c] >> 4)) * 4, F_PUB + src * 1024, AUX_SYS);
                    const unsigned long long n0 = ~__ballot(f[0] >= call), n1 = ~__ballot(f[1] >= call), n2 = ~__ballot(f[2] >= call);
                    pub_ready = n0 ? __builtin_ctzll(n0) : n1 ? 64 + __builtin_ctzll(n1) : n2 ? 128 + __builtin_ctzll(n2) : 192;
                    if (u < pub_ready) break;
                    if (give_up(rs_me, call, t0)) { bad = true; break; }
                }
            }
            const int c0 = __builtin_amdgcn_readlane(code[0], u & 63), c1 = __builtin_amdgcn_readlane(code[1], u & 63), c2 = __builtin_amdgcn_readlane(code[2], u & 63);
            const int cd = u < 64 ? c0 : u < 128 ? c1 : c2, kc = cd & 7, idx = cd >> 4;
            if (cd & 8) {
                so = exp_off + EXP_W + (unsigned)((rank * 8 + kc) * 8 + idx) * UNIT;
                dofs = OFF_STG + src * STG_SZ + EXP_W + (unsigned)(kc * 8 + idx) * UNIT;
                fl = F_B + ((w * 64 + rank * 8 + idx) * 64 + src * 8 + kc) * 4;
            } else {
                so = exp_off + (unsigned)(kc * 16 + idx) * UNIT;
                dofs = OFF_STG + src * STG_SZ + (unsigned)(kc * 16 + idx) * UNIT;
                fl = F_A + ((w * 16 + idx) * 64 + src * 8 + kc) * 4;
            }
            so += w * 1024; dofs += w * 1024;
            STAMP_T(t);
        };
        auto y_decode = [&](unsigned t, unsigned& ys, unsigned& ds, unsigned& os) {
            const int q = t - NIN, tl = q >> 6, g = (q >> 3) & 7, am = q & 7;
            const int bit = (tl & 7) * 8 + g;
            if (!((((tl & 8) ? yav1 : yav0) >> bit) & 1ull) && !bad) {
                const unsigned long long t0 = __builtin_amdgcn_s_memrealtime();
                for (;;) {
                    const unsigned y0 = __builtin_amdgcn_raw_buffer_load_b32(rs_me, v_fy + F_Y, 0, AUX_SYS);
                    const unsigned y1 = __builtin_amdgcn_raw_buffer_load_b32(rs_me, v_fy + F_Y + 256, 0, AUX_SYS);
                    const unsigned d0 = __builtin_amdgcn_raw_buffer_load_b32(rs_me, v_fy + F_DG, 0, AUX_AGENT);
                    const unsigned d1 = __builtin_amdgcn_raw_buffer_load_b32(rs_me, v_fy + F_DG + 256, 0, AUX_AGENT);
                    yav0 = __ballot((y0 < d0 ? y0 : d0) >= call); yav1 = __ballot((y1 < d1 ? y1 : d1) >= call);
                    if ((((tl & 8) ? yav1 : yav0) >> bit) & 1ull) break;
                    if (give_up(rs_me, call, t0)) { bad = true; break; }
                }
            }
            ys = yexp_off + (unsigned)(rank * 16 + tl) * 131072u + g * 16384 + am * 2048;
            ds = OFF_DIAG + (unsigned)(src * 16 + tl) * 262144u + g * 32768 + am * 4096;
            os = (unsigned)((tl >> 3) * 256 + (am >> 2) * 128 + (g >> 2) * 64 + (am & 3) * 16) * (GN * 2)
               + (unsigned)(src * 2048 + (tl & 7) * 256 + (g & 3) * 64) * 2;
            STAMP_T(t);
        };
        unsigned t = TICKET();
        if (t < NIN) {
            unsigned so, dofs, fl, prev = 0;
            in_decode(t, so, dofs, fl);
            STAMP_AT(5);
            if (!bad) {
                u32x4 A = RLOAD(so), B;
                for (;;) {
                    DRAW();
                    DRAIN(); B = RLOAD(so + 8192); SCHED; ST128(A, rs_me, vl16, dofs, AUX_AGENT);
                    if (prev && lane == 0) __builtin_amdgcn_raw_buffer_store_b32(call, rs_me, 0, prev, AUX_AGENT);
                    t = TICKET();
                    DRAIN(); A = RLOAD(so + 16384); SCHED; ST128(B, rs_me, vl16, dofs + 8192, AUX_AGENT);
                    DRAIN(); B = RLOAD(so + 24576); SCHED; ST128(A, rs_me, vl16, dofs + 16384, AUX_AGENT);
                    const unsigned dlast = dofs + 24576; prev = fl;
                    bool more = t < NIN;
                    if (more) { in_decode(t, so, dofs, fl); more = !bad; }
                    DRAIN();
                    if (more) A = RLOAD(so);
                    SCHED; ST128(B, rs_me, vl16, dlast, AUX_AGENT);
                    if (!more) break;
                }
                DRAIN();
                if (lane == 0) __builtin_amdgcn_raw_buffer_store_b32(call, rs_me, 0, prev, AUX_AGENT);
            }
            while (t < NIN) { DRAW(); t = TICKET(); }
        }
        STAMP_AT(2);
        if (t < NTOT) {
            const unsigned v_out = (unsigned)(lane & 15) * (GN * 2) + (unsigned)(lane >> 4) * 16;
            auto finish = [&](const u32x4 y, const u32x4 d0, const u32x4 d1, unsigned os) {
                u32x4 o;
                o.x = cvt_pk_bf16(silu(__uint_as_float(y.x << 16) + __uint_as_float(d0.x)), silu(__uint_as_float(y.x & 0xffff0000u) + __uint_as_float(d0.y))) | (bad ? 0x7fc07fc0u : 0u);
                o.y = cvt_pk_bf16(silu(__uint_as_float(y.y << 16) + __uint_as_float(d0.z)), silu(__uint_as_float(y.y & 0xffff0000u) + __uint_as_float(d0.w)));
                o.z = cvt_pk_bf16(silu(__uint_as_float(y.z << 16) + __uint_as_float(d1.x)), silu(__uint_as_float(y.z & 0xffff0000u) + __uint_as_float(d1.y)));
                o.w = cvt_pk_bf16(silu(__uint_as_float(y.w << 16) + __uint_as_float(d1.z)), silu(__uint_as_float(y.w & 0xffff0000u) + __uint_as_float(d1.w)));
                ST128(o, rs_out, v_out, os, AUX_AGENT);
            };
            unsigned ys, ds, os;
            y_decode(t, ys, ds, os);
            u32x4 A = RLOAD(ys), a0 = LLOAD(ds), a1 = LLOAD(ds + 1024);
            DRAW();
            DRAIN();
            for (;;) {
                t = TICKET();
                const u32x4 B = RLOAD(ys + 1024), b0 = LLOAD(ds + 2048), b1 = LLOAD(ds + 3072);
                SCHED;
                finish(A, a0, a1, os);
                const unsigned ol = os + 64;
                const bool more = t < NTOT;
                if (more) y_decode(t, ys, ds, os);
                YWAIT();
                if (more) { A = RLOAD(ys); a0 = LLOAD(ds); a1 = LLOAD(ds + 1024); DRAW(); }
                SCHED;
                finish(B, b0, b1, ol);
                if (!more) break;
                YWAIT();
            }
        }
        STAMP_AT(3);
        return;
    }

    const int wr = wid >> 2, wc = wid & 3;
    int nt, seglen, mode, tile0;
    unsigned v_a, v_b, v_fa, v_fb;
    int dest;
    if (slot < 23 || slot >= 30) {
        const bool helper = slot >= 30;
        const int i = slot - 7;
        const int j = helper ? rank : ((xcd >> 1) & 1) * 4 + (i & 3), rt2 = helper ? slot - 30 : xcd & 1;
        const int rt = 2 * j + rt2, ct = helper ? xcd : (xcd >> 2) * 4 + (i >> 2);
        const bool own = (j == rank);
        const int nsrc = own ? 7 : 6;
        nt = own ? 32 : 56; seglen = nt; mode = helper ? 2 : own ? 0 : 1; dest = j; tile0 = rt2 * 8 + ct;
        const int st = own ? 2 * (lane & 31) + (helper ? 0 : 1) : lane;
        int src, kc, si = 0;
        if (st < 8 || st >= 8 + 8 * nsrc) { src = rank; kc = st & 7; }
        else {
            const int q = st - 8; kc = q / nsrc; si = q - kc * nsrc;
            int l = ((j - rank - 1) & 7) - 1 - si; if (l < 0) l += 7;
            src = (rank + 1 + l) & 7;
        }
        v_a = (src == rank ? exp_off : OFF_STG + src * STG_SZ) + (unsigned)(kc * 16 + rt) * UNIT;
        if (src != rank && kc == 0 && !own && rt2 == 0 && si < NPUSH)
            v_a = OFF_PUSH + (unsigned)((par * 8 + src) * 8 + j) * UNIT;
        v_b = (src == rank ? exp_off + (unsigned)(rank * 64) * UNIT : OFF_STG + src * STG_SZ) + EXP_W + (unsigned)(kc * 8 + ct) * UNIT;
        v_fa = F_A + ((wid * 16 + rt) * 64 + src * 8 + kc) * 4;
        v_fb = F_B + ((wid * 64 + rank * 8 + ct) * 64 + src * 8 + kc) * 4;
    } else {
        const int slab = (rank + 1 + (slot - 23)) & 7, ct = xcd;
        nt = 16; seglen = 8; mode = 2; dest = slab; tile0 = ct;
        const int kc = lane & 7, rt = 2 * rank + ((lane >> 3) & 1);
        v_a = exp_off + (unsigned)(kc * 16 + rt) * UNIT;
        v_b = exp_off + EXP_W + (unsigned)((slab * 8 + kc) * 8 + ct) * UNIT;
        v_fa = F_A + ((wid * 16 + rt) * 64 + rank * 8 + kc) * 4;
        v_fb = F_B + ((wid * 64 + slab * 8 + ct) * 64 + rank * 8 + kc) * 4;
    }
    int ready = 0; bool bad = false;
    auto ensure = [&](int need) {
        if (need > nt) need = nt;
        if (ready >= need) return;
        const unsigned long long t0 = __builtin_amdgcn_s_memrealtime();
        __builtin_amdgcn_s_waitcnt(0xC07F);
        for (;;) {
            const unsigned fa = __builtin_amdgcn_raw_buffer_load_b32(rs_me, v_fa, 0, AUX_SYS);
            const unsigned fb = __builtin_amdgcn_raw_buffer_load_b32(rs_me, v_fb, 0, AUX_AGENT);
            const unsigned long long nm = ~__ballot((fa < fb ? fa : fb) >= call);
            ready = nm ? __builtin_ctzll(nm) : 64;
            if (ready >= need) break;
            if (give_up(rs_me, call, t0)) { bad = true; ready = 64; break; }
            __builtin_amdgcn_s_sleep(4);
        }
    };
    const unsigned vst = (unsigned)tid * 16u;
    const int ldsw = wid * 1024;
    const int aoff = wr * 8192 + lane * 16;
    int boff = 65536 + wc * 4096 + lane * 16; asm volatile("" : "+v"(boff));
#define SA(b, h) (((b) * 2 + (h)) * 16384)
#define SB(b, h) ((4 + (b) * 2 + (h)) * 16384)
#define STAGE(bufoff, goff) do { _Pragma("unroll") for (int _i = 0; _i < 2; ++_i) \
        __builtin_amdgcn_raw_ptr_buffer_load_lds(rs_me, (LAS void*)(lds + (bufoff) + ldsw + _i * 8192), 16, vst, (goff) + _i * 8192, 0, AUX_AGENT); } while (0)
#define LDA(dst, b, h) do { _Pragma("unroll") for (int m = 0; m < 4; ++m) _Pragma("unroll") for (int k = 0; k < 2; ++k) \
        dst[m][k] = *(const LAS bf16x8*)(lds + SA(b, h) + aoff + m * 2048 + k * 1024); } while (0)
#define LDB(dst, b, h) do { _Pragma("unroll") for (int n = 0; n < 2; ++n) _Pragma("unroll") for (int k = 0; k < 2; ++k) \
        dst[n][k] = *(const LAS bf16x8*)(lds + boff + (SB(b, h) - 65536 + n * 2048 + k * 1024)); } while (0)
#define MMA(ai, bj, At, Bt) do { __builtin_amdgcn_s_setprio(1); _Pragma("unroll") for (int m = 0; m < 4; ++m) _Pragma("unroll") for (int n = 0; n < 2; ++n) \
        _Pragma("unroll") for (int k = 0; k < 2; ++k) acc[ai][bj][m][n] = __builtin_amdgcn_mfma_f32_16x16x32_bf16(Bt[n][k], At[m][k], acc[ai][bj][m][n], 0, 0, 0); \
        __builtin_amdgcn_s_setprio(0); } while (0)
#define AOFF(t) ((unsigned)__builtin_amdgcn_readlane((int)v_a, (t)))
#define BOFF(t) ((unsigned)__builtin_amdgcn_readlane((int)v_b, (t)))
    f32x4 acc[2][2][4][2];
#pragma unroll
    for (int a = 0; a < 2; ++a)
#pragma unroll
        for (int b = 0; b < 2; ++b)
#pragma unroll
            for (int m = 0; m < 4; ++m)
#pragma unroll
                for (int n = 0; n < 2; ++n) acc[a][b][m][n] = (f32x4){0.f, 0.f, 0.f, 0.f};
    bf16x8 At[4][2], B0[2][2], B1[2][2];
    ensure(2);
    {
        const unsigned a0 = AOFF(0), b0 = BOFF(0), a1 = AOFF(1), b1 = BOFF(1);
        STAGE(SB(0, 0), b0); STAGE(SA(0, 0), a0); STAGE(SB(0, 1), b0 + 16384); STAGE(SA(0, 1), a0 + 16384);
        if (wr == 1) BAR;
        WAIT_V(4); BAR;
        STAGE(SB(1, 0), b1); STAGE(SA(1, 0), a1); STAGE(SB(1, 1), b1 + 16384);
        WAIT_V(6); BAR;
    }
    STAMP_AT(2);
    int tile = tile0;
    for (int t = 0;;) {
        const int seg_end = t + seglen;
        for (; t < seg_end; t += 2) {
            const int t2 = (t + 2 < nt) ? t + 2 : 0;
            const unsigned a1 = AOFF(t + 1), a2 = AOFF(t2), b2 = BOFF(t2), a3 = AOFF(t2 + 1), b3 = BOFF(t2 + 1);
            LDB(B0, 0, 0); SCHED; LDA(At, 0, 0); STAGE(SA(1, 1), a1 + 16384);
            WAIT_L(8); BAR; WAIT_L(0); MMA(0, 0, At, B0); BAR; SCHED;
            ensure(t + 3);
            LDB(B1, 0, 1); STAGE(SB(0, 0), b2);
            BAR; WAIT_L(0); MMA(0, 1, At, B1); BAR;
            LDA(At, 0, 1); STAGE(SA(0, 0), a2);
            BAR; WAIT_L(0); MMA(1, 0, At, B0); BAR; SCHED;
            STAGE(SB(0, 1), b2 + 16384);
            WAIT_V(6); BAR; MMA(1, 1, At, B1); BAR;
            LDB(B0, 1, 0); SCHED; LDA(At, 1, 0); STAGE(SA(0, 1), a2 + 16384);
            WAIT_L(8); BAR; WAIT_L(0); MMA(0, 0, At, B0); BAR; SCHED;
            ensure(t + 4);
            LDB(B1, 1, 1); STAGE(SB(1, 0), b3);
            BAR; WAIT_L(0); MMA(0, 1, At, B1); BAR;
            LDA(At, 1, 1); STAGE(SA(1, 0), a3);
            BAR; WAIT_L(0); MMA(1, 0, At, B0); BAR; SCHED;
            STAGE(SB(1, 1), b3 + 16384);
            WAIT_V(6); BAR; MMA(1, 1, At, B1); BAR;
        }
        if (t >= nt && wr == 0) BAR;
        STAMP_AT(3);
        if (mode != 2) break;
        {
            int lz = lane; asm volatile("" : "+v"(lz));
            const unsigned nanm = bad ? 0x7fc00000u : 0u;
            const unsigned so = OFF_DIAG + (unsigned)(dest * 16 + tile) * 262144u + wid * 32768;
#pragma unroll
            for (int a = 0; a < 2; ++a)
#pragma unroll
                for (int m = 0; m < 4; ++m)
#pragma unroll
                    for (int b = 0; b < 2; ++b)
#pragma unroll
                        for (int n = 0; n < 2; ++n) {
                            u32x4 o = __builtin_bit_cast(u32x4, acc[a][b][m][n]); o.x |= nanm;
                            ST128(o, rs_me, lz * 16 + (b * 2 + n) * 1024, so + (a * 4 + m) * 4096, AUX_AGENT);
                        }
            WAIT_V(0);
            if (lz == 0) __builtin_amdgcn_raw_buffer_store_b32(call, rs_me, 0, F_DG + ((dest * 16 + tile) * 8 + wid) * 4, AUX_AGENT);
        }
        STAMP_AT(4);
        if (t >= nt) break;
#pragma unroll
        for (int a = 0; a < 2; ++a)
#pragma unroll
            for (int b = 0; b < 2; ++b)
#pragma unroll
                for (int m = 0; m < 4; ++m)
#pragma unroll
                    for (int n = 0; n < 2; ++n) acc[a][b][m][n] = (f32x4){0.f, 0.f, 0.f, 0.f};
        tile += 8;
    }
    if (mode != 2) {
        int lz = lane; asm volatile("" : "+v"(lz));
        if (mode == 0) {
            {
                const unsigned long long t0 = __builtin_amdgcn_s_memrealtime();
                while (!bad && (unsigned)rfl((int)__builtin_amdgcn_raw_buffer_load_b32(rs_me, 0, F_DG + ((rank * 16 + tile) * 8 + wid) * 4, AUX_AGENT)) < call)
                    if (give_up(rs_me, call, t0)) bad = true;
            }
            const unsigned nanm = bad ? 0x7fc07fc0u : 0u;
            const rsrc_t rs_out = mkrs(p.out, OUT_BYTES);
            const unsigned vo = (unsigned)(lz & 15) * (GN * 2) + (unsigned)(lz >> 4) * 16;
            const unsigned so = (unsigned)((tile >> 3) * 256 + wr * 64) * (GN * 2) + (unsigned)(rank * 2048 + (tile & 7) * 256 + wc * 64) * 2;
            const unsigned hs = OFF_DIAG + (unsigned)(rank * 16 + tile) * 262144u + wid * 32768;
#pragma unroll
            for (int a = 0; a < 2; ++a) {
                f32x4 h[4][2][2];
#pragma unroll
                for (int m = 0; m < 4; ++m)
#pragma unroll
                    for (int b = 0; b < 2; ++b)
#pragma unroll
                        for (int n = 0; n < 2; ++n)
                            h[m][b][n] = __builtin_bit_cast(f32x4, __builtin_amdgcn_raw_buffer_load_b128(rs_me, lz * 16 + (b * 2 + n) * 1024, hs + (a * 4 + m) * 4096, AUX_AGENT));
#pragma unroll
                for (int m = 0; m < 4; ++m)
#pragma unroll
                    for (int b = 0; b < 2; ++b) {
                        const f32x4 v0 = acc[a][b][m][0] + h[m][b][0], v1 = acc[a][b][m][1] + h[m][b][1];
                        u32x4 o; o.x = cvt_pk_bf16(silu(v0[0]), silu(v0[1])) | nanm; o.y = cvt_pk_bf16(silu(v0[2]), silu(v0[3]));
                        o.z = cvt_pk_bf16(silu(v1[0]), silu(v1[1])); o.w = cvt_pk_bf16(silu(v1[2]), silu(v1[3]));
                        ST128(o, rs_out, vo + b * 64, so + (unsigned)(a * 128 + m * 16) * (GN * 2), AUX_AGENT);
                    }
                asm volatile("" ::: "memory");
            }
        } else {
            const unsigned nanm = bad ? 0x7fc07fc0u : 0u;
            const unsigned so = yexp_off + (unsigned)(dest * 16 + tile) * 131072u + wid * 16384;
#pragma unroll
            for (int a = 0; a < 2; ++a)
#pragma unroll
                for (int m = 0; m < 4; ++m)
#pragma unroll
                    for (int b = 0; b < 2; ++b) {
                        const f32x4 v0 = acc[a][b][m][0], v1 = acc[a][b][m][1];
                        u32x4 o; o.x = cvt_pk_bf16(v0[0], v0[1]) | nanm; o.y = cvt_pk_bf16(v0[2], v0[3]);
                        o.z = cvt_pk_bf16(v1[0], v1[1]); o.w = cvt_pk_bf16(v1[2], v1[3]);
                        ST128(o, rs_me, lz * 16 + b * 1024, so + (a * 4 + m) * 2048, AUX_SYS);
                    }
            WAIT_V(0);
            if (lz == 0) __builtin_amdgcn_raw_buffer_store_b32(call, mkrs(p.ws[(dest - rank - 1) & 7], WS_NEED), 0, F_Y + ((rank * 16 + tile) * 8 + wid) * 4, AUX_SYS);
        }
        STAMP_AT(4);
    }
    WAIT_V(0);
}

}

extern "C" void kernel_launch(int rank, int nranks, ncclComm_t comm,
                              hipStream_t stream,
                              void* const* d_in, const int* in_sizes, int n_in,
                              void* d_out, int out_size,
                              const int* in_dtypes, int out_dtype,
                              void* const* d_out_all,
                              void* const* d_ws_all, size_t ws_bytes) {
    (void)comm; (void)in_dtypes; (void)out_dtype; (void)d_out_all;
    if (nranks != NR || (unsigned)rank >= (unsigned)NR || n_in < 2 || ws_bytes < (size_t)WS_NEED ||
        in_sizes[0] != GM * KS || in_sizes[1] != KS * GN || out_size != (GM / NR) * GN) {
        fprintf(stderr, "kernel_launch: unexpected problem shape\n");
        return;
    }
    Params p;
    p.x = d_in[0]; p.w = d_in[1]; p.out = d_out; p.me = d_ws_all[rank]; p.rank = rank; p.pad = 0;
    for (int i = 0; i < NR; ++i) p.ws[i] = d_ws_all[(rank + 1 + i) & 7];
    hipLaunchKernelGGL(fused_gemm_rs, dim3(256), dim3(512), 0, stream, p);
    HIP_CHECK(hipGetLastError());
}
```

```cpp
#include <hip/hip_runtime.h>
#include <rccl/rccl.h>
#include <cstdio>

#define HIP_CHECK(expr) do { hipError_t _e = (expr); if (_e != hipSuccess) { \
    fprintf(stderr, "HIP error %s at %s:%d\n", hipGetErrorString(_e), __FILE__, __LINE__); \
    return; } } while (0)

namespace {

constexpr int NR = 8;
constexpr int GM = 4096, GK = 4096, GN = 16384;
constexpr int KS = GK / NR;
constexpr unsigned MiB = 1u << 20;
constexpr unsigned UNIT = 32768;
constexpr unsigned X_BYTES = GM * KS * 2, W_BYTES = KS * GN * 2, OUT_BYTES = (GM / NR) * GN * 2;

constexpr unsigned EXP_SZ = 20 * MiB;
constexpr unsigned EXP_W = 4 * MiB;
constexpr unsigned OFF_EXP = 2 * MiB;
constexpr unsigned YEXP_SZ = 16 * MiB;
constexpr unsigned OFF_YEXP = OFF_EXP + 2 * EXP_SZ;
constexpr unsigned STG_SZ = 6 * MiB;
constexpr unsigned OFF_STG = OFF_YEXP + 2 * YEXP_SZ;
constexpr unsigned OFF_DIAG = OFF_STG + NR * STG_SZ;
constexpr unsigned OFF_PUSH = OFF_DIAG + 32 * MiB;
constexpr unsigned WS_NEED = OFF_PUSH + 4 * MiB;
constexpr unsigned F_CNT = 0;
constexpr unsigned F_TMO = 32768;
constexpr unsigned F_PUB = 36864;
constexpr unsigned F_Y = 49152;
constexpr unsigned F_DG = 57344;
constexpr unsigned F_Q = 33792;
constexpr unsigned F_A = 65536;
constexpr unsigned F_B = 131072;
constexpr unsigned F_STAMP = 524288;

constexpr unsigned long long DEADLINE = 100000000ull;

constexpr int NPX = 3;
constexpr int FR = 4;
constexpr int NPUSH = 5;
constexpr int LAG_O = 2;
constexpr int LAG_W = 6;

static_assert(NPUSH >= 1 && NPUSH <= 6 && 56 * NPUSH <= 8 * 25 * (8 - NPX), "push pieces");
static_assert(FR * 200 >= 640, "every unit is published");
constexpr int AUX_AGENT = 16, AUX_SYS = 17;

typedef short bf16x8 __attribute__((ext_vector_type(8)));
typedef short s16x4 __attribute__((ext_vector_type(4)));
typedef float f32x4 __attribute__((ext_vector_type(4)));
typedef unsigned u32x4 __attribute__((ext_vector_type(4)));
typedef unsigned u32x2 __attribute__((ext_vector_type(2)));
#define LAS __attribute__((address_space(3)))
typedef __amdgpu_buffer_rsrc_t rsrc_t;
typedef __attribute__((address_space(1))) unsigned gu32;

struct Params {
    const void* x; const void* w; void* out; void* me;
    int rank, pad;
    void* ws[NR];
};

__device__ __forceinline__ rsrc_t mkrs(const void* p, unsigned bytes) {
    return __builtin_amdgcn_make_buffer_rsrc((void*)p, 0, bytes, 0x00020000);
}
__device__ __forceinline__ unsigned cvt_pk_bf16(float lo, float hi) {
    typedef float f32x2 __attribute__((ext_vector_type(2)));
    typedef __bf16 bf16x2 __attribute__((ext_vector_type(2)));
    return __builtin_bit_cast(unsigned, __builtin_convertvector((f32x2){lo, hi}, bf16x2));
}
__device__ __forceinline__ float silu(float y) {
    const float c = __builtin_amdgcn_fmed3f(y, -60.0f, 60.0f);
    const float e = __builtin_amdgcn_exp2f(c * -1.44269504088896341f);
    return y * __builtin_amdgcn_rcpf(1.0f + e);
}
__device__ __forceinline__ int rfl(int v) { return __builtin_amdgcn_readfirstlane(v); }
__device__ __forceinline__ int blk_at(int me, int l, int pos) { int v = l + 1 + pos; if (v >= 7) v -= 7; return (me + 1 + v) & 7; }
#define ST128(v, rs, voff, soff, aux) __builtin_amdgcn_raw_buffer_store_b128((v), (rs), (unsigned)(voff) + (unsigned)(soff), 0, (aux))
#define WAIT_V(n) asm volatile("s_waitcnt vmcnt(" #n ")" ::: "memory")
#define WAIT_L(n) asm volatile("s_waitcnt lgkmcnt(" #n ")" ::: "memory")
#define BAR __builtin_amdgcn_s_barrier()
#define SCHED __builtin_amdgcn_sched_barrier(0)

#define STAMP_AT(slot) do { } while (0)
#define STAMP_T(t) do { } while (0)

__device__ __forceinline__ bool give_up(rsrc_t rs_me, unsigned call, unsigned long long t0) {
    const unsigned tm = __builtin_amdgcn_raw_buffer_load_b32(rs_me, 0, F_TMO, AUX_AGENT);
    if ((unsigned)rfl((int)tm) == call || __builtin_amdgcn_s_memrealtime() - t0 > DEADLINE) {
        __builtin_amdgcn_raw_buffer_store_b32(call, rs_me, 0, F_TMO, AUX_AGENT);
        return true;
    }
    __builtin_amdgcn_s_sleep(2);
    asm volatile("" ::: "memory");
    return false;
}

__global__ __launch_bounds__(512, 2) void fused_gemm_rs(const Params p) {
    __shared__ __attribute__((aligned(16))) unsigned char smem[131072];
    LAS unsigned char* lds = (LAS unsigned char*)smem;
    const int tid = threadIdx.x, lane = tid & 63, wid = rfl(tid >> 6);
    const int bid = blockIdx.x, xcd = bid & 7, slot = bid >> 3;
    const int rank = p.rank;
    const int po = xcd < NPX ? (slot - 7) * NPX + xcd : 25 * NPX + (slot - 7) * (8 - NPX) + (xcd - NPX);
    constexpr int PWV = (56 * NPUSH + 25 * (8 - NPX) - 1) / (25 * (8 - NPX));
    const bool pusher = slot >= 7 && xcd >= NPX && wid < PWV && (po - 25 * NPX) * PWV + wid < 56 * NPUSH;
    const int pi = pusher ? (po - 25 * NPX) * PWV + wid : 0, pe = pi / 7;
    void* const push_ws = p.ws[slot < 7 ? slot : pi - pe * 7];
    asm volatile("" :: "s"(p.me), "s"(p.x), "s"(p.w), "s"(rank), "s"(push_ws));
    const rsrc_t rs_me = mkrs(p.me, WS_NEED);

    STAMP_AT(0);
    const unsigned vcall = __builtin_amdgcn_raw_buffer_load_b32(rs_me, 0, F_CNT + bid * 128, AUX_AGENT);
    unsigned call = 0, exp_off = 0;

    if (slot < 7) {
        if (wid == 0) {
            unsigned tq = 0;
            if (lane < 2) tq = __builtin_amdgcn_raw_buffer_load_b32(mkrs(push_ws, WS_NEED), lane * (2 * MiB), OFF_PUSH + (unsigned)(((rank + 1 + slot) & 7) * 8 + 4) * UNIT + (rank * 8 + xcd) * 128, AUX_SYS);
            asm volatile("" :: "v"(tq));
        }
        call = (unsigned)rfl((int)vcall) + 1u;
        exp_off = OFF_EXP + (call & 1u) * EXP_SZ;
        __syncthreads();
        if (tid == 0) __builtin_amdgcn_raw_buffer_store_b32(call, rs_me, 0, F_CNT + bid * 128, AUX_AGENT);
    } else {
        const rsrc_t rs_x = mkrs(p.x, X_BYTES), rs_w = mkrs(p.w, W_BYTES);
        const unsigned vx = (unsigned)(lane & 15) * (KS * 2) + (unsigned)(lane >> 4) * 16;
        const int pdest = (rank + 1 + pi - pe * 7) & 7, pw = pe / NPUSH, pu = pe - pw * NPUSH;
        const int prt = 2 * blk_at(pdest, (rank - pdest - 1) & 7, pu);
        const rsrc_t rs_push = mkrs(push_ws, WS_NEED);
        if (pusher) {
            u32x4 pv[4];
#pragma unroll
            for (int i = 0; i < 4; ++i)
                pv[i] = __builtin_amdgcn_raw_buffer_load_b128(rs_x, vx, (unsigned)(prt * 256 + ((pw >> 1) + 4 * i) * 16) * (KS * 2) + (pw & 1) * 64, 0);
            SCHED;
            unsigned tw = 0;
            if (lane < 2) tw = __builtin_amdgcn_raw_buffer_load_b32(rs_push, lane * (2 * MiB), OFF_PUSH + (unsigned)(pdest * 8) * UNIT + (rank * 64 + pe) * 128, AUX_SYS);
            SCHED;
            const unsigned d = OFF_PUSH + (unsigned)((((rfl((int)vcall) + 1) & 1) * 8 + rank) * 8 + (prt >> 1)) * UNIT + pw * 1024;
#pragma unroll
            for (int i = 0; i < 4; ++i) ST128(pv[i], rs_push, lane * 16, d + i * 8192, AUX_SYS);
            SCHED;
            asm volatile("" :: "v"(tw));
        }
        for (int r = 0; r < FR; ++r) {
            const int u = xcd < NPX ? r * (25 * NPX) + po : (FR - 1) * (25 * NPX) + r * (25 * (8 - NPX)) + po;
            if (u >= 640) break;
            const int kc = u / 80, j = u - kc * 80;
            const bool isw = j < 64;
            const int dl = isw ? j >> 3 : (wid + 7) & 7, dest = (rank + 1 + dl) & 7;
            const rsrc_t rs_flag = mkrs(p.ws[dl], WS_NEED);
            const int ct = j & 7, rt = j - 64;
            u32x4 v[4];
            if (isw) {
                const unsigned so = (unsigned)(kc * 64) * (GN * 2) + (unsigned)(dest * 2048 + ct * 256) * 2;
                const unsigned vo = (unsigned)(tid >> 5) * (GN * 2) + (unsigned)(tid & 31) * 16;
#pragma unroll
                for (int i = 0; i < 4; ++i) v[i] = __builtin_amdgcn_raw_buffer_load_b128(rs_w, vo, so + i * 16 * (GN * 2), 0);
            } else {
#pragma unroll
                for (int i = 0; i < 4; ++i) {
                    const int F = wid * 4 + i;
                    v[i] = __builtin_amdgcn_raw_buffer_load_b128(rs_x, vx, (unsigned)(rt * 256 + (F >> 1) * 16) * (KS * 2) + kc * 128 + (F & 1) * 64, 0);
                }
            }
            if (r == 0) {
                SCHED;
                call = (unsigned)rfl((int)vcall) + 1u;
                exp_off = OFF_EXP + (call & 1u) * EXP_SZ;
            }
            if (isw) {
                const unsigned dst = exp_off + EXP_W + (unsigned)((dest * 8 + kc) * 8 + ct) * UNIT;
#pragma unroll
                for (int i = 0; i < 4; ++i) *(LAS u32x4*)(lds + (i * 16 + (tid >> 5)) * 576 + (tid & 31) * 16) = v[i];
                __syncthreads();
#pragma unroll
                for (int i = 0; i < 4; ++i) {
                    const int F = i * 8 + wid, bj = F >> 4, wcn = (F >> 1) & 7, kk = F & 1;
                    const int col = (wcn >> 1) * 64 + bj * 32 + (wcn & 1) * 4 + (lane & 3) * 8;
                    const int row = kk * 32 + (lane >> 4) * 8 + ((lane & 15) >> 2);
                    const s16x4 t0 = __builtin_amdgcn_ds_read_tr16_b64_v4i16((LAS s16x4*)(lds + row * 576 + col * 2));
                    const s16x4 t1 = __builtin_amdgcn_ds_read_tr16_b64_v4i16((LAS s16x4*)(lds + (row + 4) * 576 + col * 2));
                    const u32x2 a = __builtin_bit_cast(u32x2, t0), b = __builtin_bit_cast(u32x2, t1);
                    v[i] = (u32x4){a.x, a.y, b.x, b.y};
                }
#pragma unroll
                for (int i = 0; i < 4; ++i) ST128(v[i], rs_me, lane * 16, dst + (i * 8 + wid) * 1024, AUX_SYS);
            } else {
                const unsigned dst = exp_off + (unsigned)(kc * 16 + rt) * UNIT;
#pragma unroll
                for (int i = 0; i < 4; ++i) ST128(v[i], rs_me, lane * 16, dst + (wid * 4 + i) * 1024, AUX_SYS);
            }
            WAIT_V(0);
            if (r == 0 && pusher && lane == 0)
                __builtin_amdgcn_raw_buffer_store_b32(call, rs_push, 0, F_A + ((pw * 16 + prt) * 64 + rank * 8) * 4, AUX_SYS);
            __syncthreads();
            if (wid == 0) {
                if (isw) {
                    if (dest != rank && lane == 0)
                        __builtin_amdgcn_raw_buffer_store_b32(call, rs_flag, 0, F_PUB + (rank * 256 + kc * 24 + ct) * 4, AUX_SYS);
                    if (lane < 8)
                        __builtin_amdgcn_raw_buffer_store_b32(call, rs_me, lane * (64 * 64 * 4), F_B + ((dest * 8 + ct) * 64 + rank * 8 + kc) * 4, AUX_AGENT);
                } else {
                    if (lane < 8)
                        __builtin_amdgcn_raw_buffer_store_b32(call, rs_me, lane * (16 * 64 * 4), F_A + (rt * 64 + rank * 8 + kc) * 4, AUX_AGENT);
                }
                if (r == 0 && lane == 0) __builtin_amdgcn_raw_buffer_store_b32(call, rs_me, 0, F_CNT + bid * 128, AUX_AGENT);
            } else if (!isw && lane == 0) {
                if ((rt >> 1) != rank)
                    __builtin_amdgcn_raw_buffer_store_b32(call, rs_flag, 0, F_PUB + (rank * 256 + kc * 24 + 8 + rt) * 4, AUX_SYS);
            }
        }
    }
    const unsigned par = call & 1u;
    const unsigned yexp_off = OFF_YEXP + par * YEXP_SZ;
    STAMP_AT(1);

    if (slot < 7) {
        const int link = slot;
        const int src = (rank + 1 + link) & 7;
        const rsrc_t rs_src = mkrs(push_ws, WS_NEED);
        const rsrc_t rs_out = mkrs(p.out, OUT_BYTES);
        constexpr unsigned NIN = (176 - NPUSH) * 8, NTOT = NIN + 16 * 64, NDRAW = NTOT + 64;
        unsigned vz = 0; asm volatile("" : "+v"(vz));
        gu32* const qctr = (gu32*)((char*)p.me + F_Q + link * 128) + vz;
        const unsigned qbase = (call - 1u) * NDRAW;
        unsigned vtick = 0;
        if (lane == 0) vtick = __hip_atomic_fetch_add(qctr, 1u, __ATOMIC_RELAXED, __HIP_MEMORY_SCOPE_AGENT);
        int code[3];
#pragma unroll
        for (int c = 0; c < 3; ++c) {
            int pos = lane + 64 * c, cd = 0; bool found = false;
            for (int s = 0; s < 8 + (LAG_W > LAG_O ? LAG_W : LAG_O); ++s) {
                if (s < 8) {
                    if (!found && pos < 8) { cd = s | 8 | (pos << 4); found = true; } pos -= 8;
                    if (s == 0) pos += NPUSH;
                    if (!found && pos < 6) { cd = s | ((2 * blk_at(rank, link, pos)) << 4); found = true; } pos -= 6;
                }
                if (s >= LAG_O && s - LAG_O < 8) {
                    if (!found && pos < 6) { cd = (s - LAG_O) | ((2 * blk_at(rank, link, pos) + 1) << 4); found = true; } pos -= 6;
                }
                if (s >= LAG_W && s - LAG_W < 8) { if (!found && pos < 2) { cd = (s - LAG_W) | ((2 * rank + pos) << 4); found = true; } pos -= 2; }
            }
            code[c] = found ? cd : 8;
        }
        int pub_ready = 0; bool bad = false;
        unsigned long long yav0 = 0, yav1 = 0;
        const unsigned vl16 = (unsigned)lane * 16u, v_fy = (unsigned)(lane * 4 + src * 512);
#define DRAIN() do { WAIT_V(0); SCHED; __builtin_amdgcn_s_waitcnt(0x0F70); SCHED; } while (0)
#define YWAIT() do { WAIT_V(1); SCHED; __builtin_amdgcn_s_waitcnt(0x0F71); SCHED; } while (0)
#define DRAW() do { if (lane == 0) vtick = __hip_atomic_fetch_add(qctr, 1u, __ATOMIC_RELAXED, __HIP_MEMORY_SCOPE_AGENT); } while (0)
#define TICKET() ((unsigned)rfl((int)vtick) - qbase)
#define RLOAD(off) __builtin_amdgcn_raw_buffer_load_b128(rs_src, vl16, (off), AUX_SYS)
#define LLOAD(off) __builtin_amdgcn_raw_buffer_load_b128(rs_me, vl16, (off), AUX_AGENT)
        auto in_decode = [&](unsigned t, unsigned& so, unsigned& dofs, unsigned& fl) {
            const int u = t >> 3, w = t & 7;
            if (u >= pub_ready) {
                const unsigned long long t0 = __builtin_amdgcn_s_memrealtime();
                for (;;) {
                    unsigned f[3];
#pragma unroll
                    for (int c = 0; c < 3; ++c)
                        f[c] = __builtin_amdgcn_raw_buffer_load_b32(rs_me, ((code[c] & 7) * 24 + ((code[c] & 8) ? 0 : 8) + (code[c] >> 4)) * 4, F_PUB + src * 1024, AUX_SYS);
                    const unsigned long long n0 = ~__ballot(f[0] >= call), n1 = ~__ballot(f[1] >= call), n2 = ~__ballot(f[2] >= call);
                    pub_ready = n0 ? __builtin_ctzll(n0) : n1 ? 64 + __builtin_ctzll(n1) : n2 ? 128 + __builtin_ctzll(n2) : 192;
                    if (!(((u < 64 ? n0 : u < 128 ? n1 : n2) >> (u & 63)) & 1ull)) break;
                    if (give_up(rs_me, call, t0)) { bad = true; break; }
                }
            }
            const int c0 = __builtin_amdgcn_readlane(code[0], u & 63), c1 = __builtin_amdgcn_readlane(code[1], u & 63), c2 = __builtin_amdgcn_readlane(code[2], u & 63);
            const int cd = u < 64 ? c0 : u < 128 ? c1 : c2, kc = cd & 7, idx = cd >> 4;
            if (cd & 8) {
                so = exp_off + EXP_W + (unsigned)((rank * 8 + kc) * 8 + idx) * UNIT;
                dofs = OFF_STG + src * STG_SZ + EXP_W + (unsigned)(kc * 8 + idx) * UNIT;
                fl = F_B + ((w * 64 + rank * 8 + idx) * 64 + src * 8 + kc) * 4;
            } else {
                so = exp_off + (unsigned)(kc * 16 + idx) * UNIT;
                dofs = OFF_STG + src * STG_SZ + (unsigned)(kc * 16 + idx) * UNIT;
                fl = F_A + ((w * 16 + idx) * 64 + src * 8 + kc) * 4;
            }
            so += w * 1024; dofs += w * 1024;
            STAMP_T(t);
        };
        auto y_decode = [&](unsigned t, unsigned& ys, unsigned& ds, unsigned& os) {
            const int q = t - NIN, tl = q >> 6, g = (q >> 3) & 7, am = q & 7;
            const int bit = (tl & 7) * 8 + g;
            if (!((((tl & 8) ? yav1 : yav0) >> bit) & 1ull) && !bad) {
                const unsigned long long t0 = __builtin_amdgcn_s_memrealtime();
                for (;;) {
                    const unsigned y0 = __builtin_amdgcn_raw_buffer_load_b32(rs_me, v_fy + F_Y, 0, AUX_SYS);
                    const unsigned y1 = __builtin_amdgcn_raw_buffer_load_b32(rs_me, v_fy + F_Y + 256, 0, AUX_SYS);
                    const unsigned d0 = __builtin_amdgcn_raw_buffer_load_b32(rs_me, v_fy + F_DG, 0, AUX_AGENT);
                    const unsigned d1 = __builtin_amdgcn_raw_buffer_load_b32(rs_me, v_fy + F_DG + 256, 0, AUX_AGENT);
                    yav0 = __ballot((y0 < d0 ? y0 : d0) >= call); yav1 = __ballot((y1 < d1 ? y1 : d1) >= call);
                    if ((((tl & 8) ? yav1 : yav0) >> bit) & 1ull) break;
                    if (give_up(rs_me, call, t0)) { bad = true; break; }
                }
            }
            ys = yexp_off + (unsigned)(rank * 16 + tl) * 131072u + g * 16384 + am * 2048;
            ds = OFF_DIAG + (unsigned)(src * 16 + tl) * 262144u + g * 32768 + am * 4096;
            os = (unsigned)((tl >> 3) * 256 + (am >> 2) * 128 + (g >> 2) * 64 + (am & 3) * 16) * (GN * 2)
               + (unsigned)(src * 2048 + (tl & 7) * 256 + (g & 3) * 64) * 2;
            STAMP_T(t);
        };
        unsigned t = TICKET();
        if (t < NIN) {
            unsigned so, dofs, fl, prev = 0;
            in_decode(t, so, dofs, fl);
            STAMP_AT(5);
            if (!bad) {
                u32x4 A = RLOAD(so), B;
                for (;;) {
                    DRAW();
                    DRAIN(); B = RLOAD(so + 8192); SCHED; ST128(A, rs_me, vl16, dofs, AUX_AGENT);
                    if (prev && lane == 0) __builtin_amdgcn_raw_buffer_store_b32(call, rs_me, 0, prev, AUX_AGENT);
                    t = TICKET();
                    DRAIN(); A = RLOAD(so + 16384); SCHED; ST128(B, rs_me, vl16, dofs + 8192, AUX_AGENT);
                    DRAIN(); B = RLOAD(so + 24576); SCHED; ST128(A, rs_me, vl16, dofs + 16384, AUX_AGENT);
                    const unsigned dlast = dofs + 24576; prev = fl;
                    bool more = t < NIN;
                    if (more) { in_decode(t, so, dofs, fl); more = !bad; }
                    DRAIN();
                    if (more) A = RLOAD(so);
                    SCHED; ST128(B, rs_me, vl16, dlast, AUX_AGENT);
                    if (!more) break;
                }
                DRAIN();
                if (lane == 0) __builtin_amdgcn_raw_buffer_store_b32(call, rs_me, 0, prev, AUX_AGENT);
            }
            while (t < NIN) { DRAW(); t = TICKET(); }
        }
        STAMP_AT(2);
        if (t < NTOT) {
            const unsigned v_out = (unsigned)(lane & 15) * (GN * 2) + (unsigned)(lane >> 4) * 16;
            auto finish = [&](const u32x4 y, const u32x4 d0, const u32x4 d1, unsigned os) {
                u32x4 o;
                o.x = cvt_pk_bf16(silu(__uint_as_float(y.x << 16) + __uint_as_float(d0.x)), silu(__uint_as_float(y.x & 0xffff0000u) + __uint_as_float(d0.y))) | (bad ? 0x7fc07fc0u : 0u);
                o.y = cvt_pk_bf16(silu(__uint_as_float(y.y << 16) + __uint_as_float(d0.z)), silu(__uint_as_float(y.y & 0xffff0000u) + __uint_as_float(d0.w)));
                o.z = cvt_pk_bf16(silu(__uint_as_float(y.z << 16) + __uint_as_float(d1.x)), silu(__uint_as_float(y.z & 0xffff0000u) + __uint_as_float(d1.y)));
                o.w = cvt_pk_bf16(silu(__uint_as_float(y.w << 16) + __uint_as_float(d1.z)), silu(__uint_as_float(y.w & 0xffff0000u) + __uint_as_float(d1.w)));
                ST128(o, rs_out, v_out, os, AUX_AGENT);
            };
            unsigned ys, ds, os;
            y_decode(t, ys, ds, os);
            u32x4 A = RLOAD(ys), a0 = LLOAD(ds), a1 = LLOAD(ds + 1024);
            DRAW();
            DRAIN();
            for (;;) {
                t = TICKET();
                const u32x4 B = RLOAD(ys + 1024), b0 = LLOAD(ds + 2048), b1 = LLOAD(ds + 3072);
                SCHED;
                finish(A, a0, a1, os);
                const unsigned ol = os + 64;
                const bool more = t < NTOT;
                if (more) y_decode(t, ys, ds, os);
                YWAIT();
                if (more) { A = RLOAD(ys); a0 = LLOAD(ds); a1 = LLOAD(ds + 1024); DRAW(); }
                SCHED;
                finish(B, b0, b1, ol);
                if (!more) break;
                YWAIT();
            }
        }
        STAMP_AT(3);
        return;
    }

    const int wr = wid >> 2, wc = wid & 3;
    int nt, seglen, mode, tile0;
    unsigned v_a, v_b, v_fa, v_fb;
    int dest;
    if (slot < 23 || slot >= 30) {
        const bool helper = slot >= 30;
        const int i = slot - 7;
        const int j = helper ? rank : ((xcd >> 1) & 1) * 4 + (i & 3), rt2 = helper ? slot - 30 : xcd & 1;
        const int rt = 2 * j + rt2, ct = helper ? xcd : (xcd >> 2) * 4 + (i >> 2);
        const bool own = (j == rank);
        const int nsrc = own ? 7 : 6;
        nt = own ? 32 : 56; seglen = nt; mode = helper ? 2 : own ? 0 : 1; dest = j; tile0 = rt2 * 8 + ct;
        const int st = own ? 2 * (lane & 31) + (helper ? 0 : 1) : lane;
        int src, kc, si = 0;
        if (st < 8 || st >= 8 + 8 * nsrc) { src = rank; kc = st & 7; }
        else {
            const int q = st - 8; kc = q / nsrc; si = q - kc * nsrc;
            int l = ((j - rank - 1) & 7) - 1 - si; if (l < 0) l += 7;
            src = (rank + 1 + l) & 7;
        }
        v_a = (src == rank ? exp_off : OFF_STG + src * STG_SZ) + (unsigned)(kc * 16 + rt) * UNIT;
        if (src != rank && kc == 0 && !own && rt2 == 0 && si < NPUSH)
            v_a = OFF_PUSH + (unsigned)((par * 8 + src) * 8 + j) * UNIT;
        v_b = (src == rank ? exp_off + (unsigned)(rank * 64) * UNIT : OFF_STG + src * STG_SZ) + EXP_W + (unsigned)(kc * 8 + ct) * UNIT;
        v_fa = F_A + ((wid * 16 + rt) * 64 + src * 8 + kc) * 4;
        v_fb = F_B + ((wid * 64 + rank * 8 + ct) * 64 + src * 8 + kc) * 4;
    } else {
        const int slab = (rank + 1 + (slot - 23)) & 7, ct = xcd;
        nt = 16; seglen = 8; mode = 2; dest = slab; tile0 = ct;
        const int kc = lane & 7, rt = 2 * rank + ((lane >> 3) & 1);
        v_a = exp_off + (unsigned)(kc * 16 + rt) * UNIT;
        v_b = exp_off + EXP_W + (unsigned)((slab * 8 + kc) * 8 + ct) * UNIT;
        v_fa = F_A + ((wid * 16 + rt) * 64 + rank * 8 + kc) * 4;
        v_fb = F_B + ((wid * 64 + slab * 8 + ct) * 64 + rank * 8 + kc) * 4;
    }
    int ready = 0; bool bad = false;
    auto ensure = [&](int need) {
        if (need > nt) need = nt;
        if (ready >= need) return;
        const unsigned long long t0 = __builtin_amdgcn_s_memrealtime();
        __builtin_amdgcn_s_waitcnt(0xC07F);
        for (;;) {
            const unsigned fa = __builtin_amdgcn_raw_buffer_load_b32(rs_me, v_fa, 0, AUX_SYS);
            const unsigned fb = __builtin_amdgcn_raw_buffer_load_b32(rs_me, v_fb, 0, AUX_AGENT);
            const unsigned long long nm = ~__ballot((fa < fb ? fa : fb) >= call);
            ready = nm ? __builtin_ctzll(nm) : 64;
            if (ready >= need) break;
            if (give_up(rs_me, call, t0)) { bad = true; ready = 64; break; }
            __builtin_amdgcn_s_sleep(4);
        }
    };
    const unsigned vst = (unsigned)tid * 16u;
    const int ldsw = wid * 1024;
    const int aoff = wr * 8192 + lane * 16;
    int boff = 65536 + wc * 4096 + lane * 16; asm volatile("" : "+v"(boff));
#define SA(b, h) (((b) * 2 + (h)) * 16384)
#define SB(b, h) ((4 + (b) * 2 + (h)) * 16384)
#define STAGE(bufoff, goff) do { _Pragma("unroll") for (int _i = 0; _i < 2; ++_i) \
        __builtin_amdgcn_raw_ptr_buffer_load_lds(rs_me, (LAS void*)(lds + (bufoff) + ldsw + _i * 8192), 16, vst, (goff) + _i * 8192, 0, AUX_AGENT); } while (0)
#define LDA(dst, b, h) do { _Pragma("unroll") for (int m = 0; m < 4; ++m) _Pragma("unroll") for (int k = 0; k < 2; ++k) \
        dst[m][k] = *(const LAS bf16x8*)(lds + SA(b, h) + aoff + m * 2048 + k * 1024); } while (0)
#define LDB(dst, b, h) do { _Pragma("unroll") for (int n = 0; n < 2; ++n) _Pragma("unroll") for (int k = 0; k < 2; ++k) \
        dst[n][k] = *(const LAS bf16x8*)(lds + boff + (SB(b, h) - 65536 + n * 2048 + k * 1024)); } while (0)
#define MMA(ai, bj, At, Bt) do { __builtin_amdgcn_s_setprio(1); _Pragma("unroll") for (int m = 0; m < 4; ++m) _Pragma("unroll") for (int n = 0; n < 2; ++n) \
        _Pragma("unroll") for (int k = 0; k < 2; ++k) acc[ai][bj][m][n] = __builtin_amdgcn_mfma_f32_16x16x32_bf16(Bt[n][k], At[m][k], acc[ai][bj][m][n], 0, 0, 0); \
        __builtin_amdgcn_s_setprio(0); } while (0)
#define AOFF(t) ((unsigned)__builtin_amdgcn_readlane((int)v_a, (t)))
#define BOFF(t) ((unsigned)__builtin_amdgcn_readlane((int)v_b, (t)))
    f32x4 acc[2][2][4][2];
#pragma unroll
    for (int a = 0; a < 2; ++a)
#pragma unroll
        for (int b = 0; b < 2; ++b)
#pragma unroll
            for (int m = 0; m < 4; ++m)
#pragma unroll
                for (int n = 0; n < 2; ++n) acc[a][b][m][n] = (f32x4){0.f, 0.f, 0.f, 0.f};
    bf16x8 At[4][2], B0[2][2], B1[2][2];
    ensure(2);
    {
        const unsigned a0 = AOFF(0), b0 = BOFF(0), a1 = AOFF(1), b1 = BOFF(1);
        STAGE(SB(0, 0), b0); STAGE(SA(0, 0), a0); STAGE(SB(0, 1), b0 + 16384); STAGE(SA(0, 1), a0 + 16384);
        if (wr == 1) BAR;
        WAIT_V(4); BAR;
        STAGE(SB(1, 0), b1); STAGE(SA(1, 0), a1); STAGE(SB(1, 1), b1 + 16384);
        WAIT_V(6); BAR;
    }
    STAMP_AT(2);
    int tile = tile0;
    for (int t = 0;;) {
        const int seg_end = t + seglen;
        for (; t < seg_end; t += 2) {
            const int t2 = (t + 2 < nt) ? t + 2 : 0;
            const unsigned a1 = AOFF(t + 1), a2 = AOFF(t2), b2 = BOFF(t2), a3 = AOFF(t2 + 1), b3 = BOFF(t2 + 1);
            LDB(B0, 0, 0); SCHED; LDA(At, 0, 0); STAGE(SA(1, 1), a1 + 16384);
            WAIT_L(8); BAR; WAIT_L(0); MMA(0, 0, At, B0); BAR; SCHED;
            ensure(t + 3);
            LDB(B1, 0, 1); STAGE(SB(0, 0), b2);
            BAR; WAIT_L(0); MMA(0, 1, At, B1); BAR;
            LDA(At, 0, 1); STAGE(SA(0, 0), a2);
            BAR; WAIT_L(0); MMA(1, 0, At, B0); BAR; SCHED;
            STAGE(SB(0, 1), b2 + 16384);
            WAIT_V(6); BAR; MMA(1, 1, At, B1); BAR;
            LDB(B0, 1, 0); SCHED; LDA(At, 1, 0); STAGE(SA(0, 1), a2 + 16384);
            WAIT_L(8); BAR; WAIT_L(0); MMA(0, 0, At, B0); BAR; SCHED;
            ensure(t + 4);
            LDB(B1, 1, 1); STAGE(SB(1, 0), b3);
            BAR; WAIT_L(0); MMA(0, 1, At, B1); BAR;
            LDA(At, 1, 1); STAGE(SA(1, 0), a3);
            BAR; WAIT_L(0); MMA(1, 0, At, B0); BAR; SCHED;
            STAGE(SB(1, 1), b3 + 16384);
            WAIT_V(6); BAR; MMA(1, 1, At, B1); BAR;
        }
        if (t >= nt && wr == 0) BAR;
        STAMP_AT(3);
        if (mode != 2) break;
        {
            int lz = lane; asm volatile("" : "+v"(lz));
            const unsigned nanm = bad ? 0x7fc00000u : 0u;
            const unsigned so = OFF_DIAG + (unsigned)(dest * 16 + tile) * 262144u + wid * 32768;
#pragma unroll
            for (int a = 0; a < 2; ++a)
#pragma unroll
                for (int m = 0; m < 4; ++m)
#pragma unroll
                    for (int b = 0; b < 2; ++b)
#pragma unroll
                        for (int n = 0; n < 2; ++n) {
                            u32x4 o = __builtin_bit_cast(u32x4, acc[a][b][m][n]); o.x |= nanm;
                            ST128(o, rs_me, lz * 16 + (b * 2 + n) * 1024, so + (a * 4 + m) * 4096, AUX_AGENT);
                        }
            WAIT_V(0);
            if (lz == 0) __builtin_amdgcn_raw_buffer_store_b32(call, rs_me, 0, F_DG + ((dest * 16 + tile) * 8 + wid) * 4, AUX_AGENT);
        }
        STAMP_AT(4);
        if (t >= nt) break;
#pragma unroll
        for (int a = 0; a < 2; ++a)
#pragma unroll
            for (int b = 0; b < 2; ++b)
#pragma unroll
                for (int m = 0; m < 4; ++m)
#pragma unroll
                    for (int n = 0; n < 2; ++n) acc[a][b][m][n] = (f32x4){0.f, 0.f, 0.f, 0.f};
        tile += 8;
    }
    if (mode != 2) {
        int lz = lane; asm volatile("" : "+v"(lz));
        if (mode == 0) {
            {
                const unsigned long long t0 = __builtin_amdgcn_s_memrealtime();
                while (!bad && (unsigned)rfl((int)__builtin_amdgcn_raw_buffer_load_b32(rs_me, 0, F_DG + ((rank * 16 + tile) * 8 + wid) * 4, AUX_AGENT)) < call)
                    if (give_up(rs_me, call, t0)) bad = true;
            }
            const unsigned nanm = bad ? 0x7fc07fc0u : 0u;
            const rsrc_t rs_out = mkrs(p.out, OUT_BYTES);
            const unsigned vo = (unsigned)(lz & 15) * (GN * 2) + (unsigned)(lz >> 4) * 16;
            const unsigned so = (unsigned)((tile >> 3) * 256 + wr * 64) * (GN * 2) + (unsigned)(rank * 2048 + (tile & 7) * 256 + wc * 64) * 2;
            const unsigned hs = OFF_DIAG + (unsigned)(rank * 16 + tile) * 262144u + wid * 32768;
#pragma unroll
            for (int a = 0; a < 2; ++a) {
                f32x4 h[4][2][2];
#pragma unroll
                for (int m = 0; m < 4; ++m)
#pragma unroll
                    for (int b = 0; b < 2; ++b)
#pragma unroll
                        for (int n = 0; n < 2; ++n)
                            h[m][b][n] = __builtin_bit_cast(f32x4, __builtin_amdgcn_raw_buffer_load_b128(rs_me, lz * 16 + (b * 2 + n) * 1024, hs + (a * 4 + m) * 4096, AUX_AGENT));
#pragma unroll
                for (int m = 0; m < 4; ++m)
#pragma unroll
                    for (int b = 0; b < 2; ++b) {
                        const f32x4 v0 = acc[a][b][m][0] + h[m][b][0], v1 = acc[a][b][m][1] + h[m][b][1];
                        u32x4 o; o.x = cvt_pk_bf16(silu(v0[0]), silu(v0[1])) | nanm; o.y = cvt_pk_bf16(silu(v0[2]), silu(v0[3]));
                        o.z = cvt_pk_bf16(silu(v1[0]), silu(v1[1])); o.w = cvt_pk_bf16(silu(v1[2]), silu(v1[3]));
                        ST128(o, rs_out, vo + b * 64, so + (unsigned)(a * 128 + m * 16) * (GN * 2), AUX_AGENT);
                    }
                asm volatile("" ::: "memory");
            }
        } else {
            const unsigned nanm = bad ? 0x7fc07fc0u : 0u;
            const unsigned so = yexp_off + (unsigned)(dest * 16 + tile) * 131072u + wid * 16384;
#pragma unroll
            for (int a = 0; a < 2; ++a)
#pragma unroll
                for (int m = 0; m < 4; ++m)
#pragma unroll
                    for (int b = 0; b < 2; ++b) {
                        const f32x4 v0 = acc[a][b][m][0], v1 = acc[a][b][m][1];
                        u32x4 o; o.x = cvt_pk_bf16(v0[0], v0[1]) | nanm; o.y = cvt_pk_bf16(v0[2], v0[3]);
                        o.z = cvt_pk_bf16(v1[0], v1[1]); o.w = cvt_pk_bf16(v1[2], v1[3]);
                        ST128(o, rs_me, lz * 16 + b * 1024, so + (a * 4 + m) * 2048, AUX_SYS);
                    }
            WAIT_V(0);
            if (lz == 0) __builtin_amdgcn_raw_buffer_store_b32(call, mkrs(p.ws[(dest - rank - 1) & 7], WS_NEED), 0, F_Y + ((rank * 16 + tile) * 8 + wid) * 4, AUX_SYS);
        }
        STAMP_AT(4);
    }
    WAIT_V(0);
}

}

extern "C" void kernel_launch(int rank, int nranks, ncclComm_t comm,
                              hipStream_t stream,
                              void* const* d_in, const int* in_sizes, int n_in,
                              void* d_out, int out_size,
                              const int* in_dtypes, int out_dtype,
                              void* const* d_out_all,
                              void* const* d_ws_all, size_t ws_bytes) {
    (void)comm; (void)in_dtypes; (void)out_dtype; (void)d_out_all;
    if (nranks != NR || (unsigned)rank >= (unsigned)NR || n_in < 2 || ws_bytes < (size_t)WS_NEED ||
        in_sizes[0] != GM * KS || in_sizes[1] != KS * GN || out_size != (GM / NR) * GN) {
        fprintf(stderr, "kernel_launch: unexpected problem shape\n");
        return;
    }
    Params p;
    p.x = d_in[0]; p.w = d_in[1]; p.out = d_out; p.me = d_ws_all[rank]; p.rank = rank; p.pad = 0;
    for (int i = 0; i < NR; ++i) p.ws[i] = d_ws_all[(rank + 1 + i) & 7];
    hipLaunchKernelGGL(fused_gemm_rs, dim3(256), dim3(512), 0, stream, p);
    HIP_CHECK(hipGetLastError());
}
```
